```python
import numpy as np

kernel_name = "gpu_gemm_ar_m4096_k4096_n16384_bf16_qe4m3"


ABSMAX_RTOL = 0.09

COMPUTE_DTYPE = "bfloat16"
EPILOGUE = "quant_e4m3"
IN_QUANT = None
OUT_QUANT = "e4m3"
IN_DTYPES = {"x": "bfloat16", "w_mat": "bfloat16"}
WORLD_SIZE = 8

def _to_bf16(x):
    u = np.ascontiguousarray(x, dtype=np.float32).view(np.uint32)
    return ((u + 0x7FFF + ((u >> 16) & 1)) & 0xFFFF0000).view(np.float32)

def setup_inputs():

    x = _to_bf16(np.random.randn(4096, 4096).astype(np.float32))
    w_mat = _to_bf16((np.random.randn(4096, 16384) / np.sqrt(4096)).astype(np.float32))
    return {"x": x, "w_mat": w_mat}

def _snap_fp8(x, mant_bits, max_abs, min_normal):
    a = np.abs(x.astype(np.float64))
    _m, _e = np.frexp(a)
    q = np.where(a >= min_normal, np.ldexp(1.0, _e - 1 - mant_bits),
                 min_normal * 2.0 ** -mant_bits)
    snapped = np.minimum(np.round(a / q) * q, max_abs)
    return (np.sign(x) * snapped).astype(np.float32)
def _quant_dequant(y, amax):
    scale = amax / 448.0
    q = _snap_fp8(y / scale, 3, 448.0, 2.0 ** -6)
    return (q * scale).astype(np.float32)
def _epilogue(y):
    return _quant_dequant(y, np.abs(y).max())

def reference(x, w_mat):
    y = (x @ w_mat).astype(np.float32)
    return _epilogue(y)
```

```cpp
#include <hip/hip_runtime.h>
#include <rccl/rccl.h>
#include <cstdio>
#include <cstdint>

#define HIP_CHECK(expr) do { hipError_t _e = (expr); if (_e != hipSuccess) { \
    fprintf(stderr, "HIP error %s at %s:%d\n", hipGetErrorString(_e), __FILE__, __LINE__); \
    return; } } while (0)

namespace {

#define LAS __attribute__((address_space(3)))
typedef short bf16x8 __attribute__((ext_vector_type(8)));
typedef float f32x4 __attribute__((ext_vector_type(4)));
typedef unsigned u32x4 __attribute__((ext_vector_type(4)));
typedef unsigned u32x2 __attribute__((ext_vector_type(2)));

constexpr int NR = 8;
constexpr int M_ = 4096, K_ = 4096, N_ = 16384;
constexpr int KS = K_ / NR;
constexpr int NB = N_ / NR;
constexpr int NRB = 16, NCB = 8;
constexpr int NKC = 8;
constexpr int NT = 64;
constexpr unsigned TILE_B = 32768, HALF_B = 16384;

constexpr unsigned XS_OFF = 0;
constexpr unsigned WS_OFF = XS_OFF + 64u * NRB * TILE_B;
constexpr unsigned FX_OFF = WS_OFF + 64u * NCB * TILE_B;
constexpr unsigned FW_OFF = FX_OFF + NRB * 64 * 16;
constexpr unsigned AM_OFF = FW_OFF + NCB * 64 * 16;
constexpr unsigned DN_OFF = AM_OFF + NR * 128 * 8;
constexpr unsigned CT_OFF = DN_OFF + 128 * 128;
constexpr unsigned TM_OFF = CT_OFF + 256 * 8 * 4;
constexpr unsigned DB_OFF = TM_OFF + 256;
constexpr unsigned WS_USED = DB_OFF + 256 * 64 * 8;
constexpr unsigned OUT_BYTES = (unsigned)M_ * N_;

constexpr unsigned long long DEADLINE = 100000000ull;
constexpr int AUX_SYS = 17;

#define PUSH_WAVES 4
#define ROLE_SPLIT 0

constexpr int LDS_STAGE = 8 * HALF_B;
constexpr int LDS_BYTES = LDS_STAGE + 256;

struct Params {
    const unsigned short* x;
    const unsigned short* w;
    unsigned char* out[NR];
    unsigned char* ws[NR];
    unsigned char* myws;
    unsigned char* myout;
    int rank;
};

__device__ __forceinline__ unsigned char* uni(const void* p) {
    const unsigned long long v = (unsigned long long)p;
    const unsigned lo = __builtin_amdgcn_readfirstlane((unsigned)v), hi = __builtin_amdgcn_readfirstlane((unsigned)(v >> 32));
    return (unsigned char*)(((unsigned long long)hi << 32) | lo);
}
__device__ __forceinline__ __amdgpu_buffer_rsrc_t mk_rsrc(const void* p, unsigned bytes) {
    return __builtin_amdgcn_make_buffer_rsrc((void*)uni(p), 0, (int)bytes, 0x00020000);
}
__device__ __forceinline__ unsigned long long now() { return __builtin_amdgcn_s_memrealtime(); }

#define STAMP(i) do { } while (0)

__device__ __forceinline__ bool give_up(unsigned* tmo, unsigned call, unsigned long long t0) {
    bool g = __hip_atomic_load(tmo, __ATOMIC_RELAXED, __HIP_MEMORY_SCOPE_AGENT) == call || now() - t0 > DEADLINE;
    g = __builtin_amdgcn_readfirstlane(g ? 1 : 0) != 0;
    if (g) __hip_atomic_store(tmo, call, __ATOMIC_RELAXED, __HIP_MEMORY_SCOPE_AGENT);
    return g;
}

__device__ __forceinline__ void push_role(const Params& p, int q, int wgi, int wid, int lane, unsigned call) {
    const int rank = p.rank;
    const int d = (rank + q) & (NR - 1);
    unsigned char* dws = uni(p.ws[d]);
    const __amdgpu_buffer_rsrc_t rd = mk_rsrc(dws, WS_USED);
    const __amdgpu_buffer_rsrc_t rx = mk_rsrc(p.x, (unsigned)M_ * KS * 2);
    const __amdgpu_buffer_rsrc_t rw = mk_rsrc(p.w, (unsigned)KS * N_ * 2);
    constexpr int NW = 16 * PUSH_WAVES;
    const int widx = wid * 16 + wgi;
    const int fr = lane & 15, fq = lane >> 4;
    const unsigned xoff = (unsigned)(fr * KS + fq * 8) * 2;
    const unsigned woff = (unsigned)(fq * 8 * N_ + fr * 4) * 2;
    const unsigned soff = lane * 16;
    for (int u = widx; u < NKC * 96; u += NW) {
        const int kc = u / 96, j = u - kc * 96;
        unsigned flag_off;
        if (j < 32) {
            const int cb = j >> 2, wc = j & 3;
            const unsigned sbase = (unsigned)((kc * 64) * N_ + d * NB + cb * 256 + wc * 64) * 2;
            const unsigned dbase = WS_OFF + (unsigned)((rank * NKC + kc) * NCB + cb) * TILE_B + wc * 4096;
            u32x2 in[2][8];
#pragma unroll
            for (int kh = 0; kh < 2; ++kh)
#pragma unroll
                for (int jj = 0; jj < 8; ++jj)
                    in[kh][jj] = __builtin_amdgcn_raw_buffer_load_b64(rw, woff, sbase + (unsigned)((kh * 32 + jj) * N_) * 2, 0);
#pragma unroll
            for (int kh = 0; kh < 2; ++kh)
#pragma unroll
                for (int f = 0; f < 4; ++f) {
                    u32x4 o;
#pragma unroll
                    for (int dd = 0; dd < 4; ++dd) {
                        const unsigned lo = (f & 2) ? in[kh][2 * dd].y : in[kh][2 * dd].x;
                        const unsigned hi = (f & 2) ? in[kh][2 * dd + 1].y : in[kh][2 * dd + 1].x;
                        o[dd] = __builtin_amdgcn_perm(hi, lo, (f & 1) ? 0x07060302u : 0x05040100u);
                    }
                    __builtin_amdgcn_raw_buffer_store_b128(o, rd, soff, dbase + (f >> 1) * HALF_B + ((f & 1) * 2 + kh) * 1024, AUX_SYS);
                }
            flag_off = FW_OFF + (unsigned)((cb * 64 + kc * 8 + rank) * 4 + wc) * 4;
        } else {
            const int jj = j - 32, rb = ((jj >> 2) + 2 * q) & 15, uq = jj & 3;
            const unsigned sbase = (unsigned)((rb * 256 + uq * 64) * KS + kc * 64) * 2;
            const unsigned dbase = XS_OFF + (unsigned)((rank * NKC + kc) * NRB + rb) * TILE_B + uq * 8192;
            u32x4 in[8];
#pragma unroll
            for (int g = 0; g < 8; ++g)
                in[g] = __builtin_amdgcn_raw_buffer_load_b128(rx, xoff, sbase + (unsigned)((g >> 1) * 16 * KS + (g & 1) * 32) * 2, 0);
#pragma unroll
            for (int g = 0; g < 8; ++g)
                __builtin_amdgcn_raw_buffer_store_b128(in[g], rd, soff, dbase + g * 1024, AUX_SYS);
            flag_off = FX_OFF + (unsigned)((rb * 64 + kc * 8 + rank) * 4 + uq) * 4;
        }
        asm volatile("s_waitcnt vmcnt(0)" ::: "memory");
        if (lane == 0) __hip_atomic_store((unsigned*)(dws + flag_off), call, __ATOMIC_RELAXED, __HIP_MEMORY_SCOPE_SYSTEM);
    }
}

#define WAIT_V(n) asm volatile("s_waitcnt vmcnt(" #n ")" ::: "memory")
#define WAIT_L(n) asm volatile("s_waitcnt lgkmcnt(" #n ")" ::: "memory")
#define BAR __builtin_amdgcn_s_barrier()
#define SCHED __builtin_amdgcn_sched_barrier(0)

__device__ __forceinline__ void gemm_role(const Params& p, LAS unsigned char* lds, int rb, int cb, int wid, int lane, unsigned call) {
    const int rank = p.rank;
    unsigned char* myws = p.myws;
    unsigned* tmo = (unsigned*)(myws + TM_OFF);
    const __amdgpu_buffer_rsrc_t rws = mk_rsrc(myws, WS_USED);
    const int wr = wid >> 2, wc = wid & 3;
    const unsigned tid16 = (unsigned)(wid * 64 + lane) * 16;
    volatile LAS int* sync_word = (volatile LAS int*)(lds + LDS_STAGE);

    int tiv;
    {
        int s = rank, kc = lane;
        if (lane >= 8) {
            const int j = lane - 8, i = j % 7, q0 = rb >> 1;
            kc = j / 7;
            const int q = (i < q0) ? q0 - i : 7 - (i - q0);
            s = (rank - q) & (NR - 1);
        }
        tiv = s * 8 + kc;
    }
    const unsigned xtile0 = XS_OFF + rb * TILE_B, wtile0 = WS_OFF + cb * TILE_B;
    int ready = 0;

#define NEED(n) do { if (ready < (n)) { \
        if (wid == 0) { \
            const unsigned fi = (unsigned)((tiv & 7) * 8 + (tiv >> 3)) * 16; \
            const unsigned long long t0 = now(); int r; \
            for (;;) { \
                const u32x4 fx = __builtin_amdgcn_raw_buffer_load_b128(rws, FX_OFF + rb * 1024 + fi, 0, AUX_SYS); \
                const u32x4 fw = __builtin_amdgcn_raw_buffer_load_b128(rws, FW_OFF + cb * 1024 + fi, 0, AUX_SYS); \
                const bool ok = fx.x == call && fx.y == call && fx.z == call && fx.w == call && fw.x == call && fw.y == call && fw.z == call && fw.w == call; \
                const unsigned long long nm = ~__ballot(ok); \
                r = nm ? __builtin_ctzll(nm) : 64; \
                if (r >= (n)) break; \
                if (give_up(tmo, call, t0)) { r = 64; break; } \
                __builtin_amdgcn_s_sleep(4); \
            } \
            __builtin_amdgcn_fence(__ATOMIC_ACQUIRE, ""); \
            if (lane == 0) *sync_word = r; \
            asm volatile("s_waitcnt vmcnt(0) lgkmcnt(0)" ::: "memory"); \
        } \
        asm volatile("" ::: "memory"); BAR; asm volatile("" ::: "memory"); \
        ready = __builtin_amdgcn_readfirstlane(*sync_word); \
        asm volatile("s_waitcnt lgkmcnt(0)" ::: "memory"); \
    } } while (0)

#define SA(b, h) (((b) * 2 + (h)) * HALF_B)
#define SB(b, h) ((4 + (b) * 2 + (h)) * HALF_B)
#define XOFF(t) (xtile0 + (unsigned)__builtin_amdgcn_readlane(tiv, (t)) * (NRB * TILE_B))
#define WOFF(t) (wtile0 + (unsigned)__builtin_amdgcn_readlane(tiv, (t)) * (NCB * TILE_B))
#define STAGE(P, SRC) do { \
        __builtin_amdgcn_raw_ptr_buffer_load_lds(rws, (LAS void*)(lds + (P) + wid * 1024), 16, tid16, (SRC), 0, 0); \
        __builtin_amdgcn_raw_ptr_buffer_load_lds(rws, (LAS void*)(lds + (P) + 8192 + wid * 1024), 16, tid16, (SRC) + 8192, 0, 0); } while (0)
#define LDA(dst, b, h) _Pragma("unroll") for (int m = 0; m < 4; ++m) _Pragma("unroll") for (int k = 0; k < 2; ++k) \
        dst[m][k] = *(const LAS bf16x8*)(abase + SA(b, h) + (m * 2 + k) * 1024)
#define LDB(dst, b, h) _Pragma("unroll") for (int n = 0; n < 2; ++n) _Pragma("unroll") for (int k = 0; k < 2; ++k) \
        dst[n][k] = *(const LAS bf16x8*)(bbase + SB(b, h) - 4 * HALF_B + (n * 2 + k) * 1024)
#define MMA(ai, bj, At, Bt) do { __builtin_amdgcn_s_setprio(1); \
        _Pragma("unroll") for (int m = 0; m < 4; ++m) _Pragma("unroll") for (int n = 0; n < 2; ++n) _Pragma("unroll") for (int k = 0; k < 2; ++k) \
            acc[ai][bj][m][n] = __builtin_amdgcn_mfma_f32_16x16x32_bf16(Bt[n][k], At[m][k], acc[ai][bj][m][n], 0, 0, 0); \
        __builtin_amdgcn_s_setprio(0); } while (0)

    const LAS unsigned char* abase = lds + wr * 8192 + lane * 16;
    const LAS unsigned char* bbase = lds + 4 * HALF_B + wc * 4096 + lane * 16;
    f32x4 acc[2][2][4][2] = {};
    bf16x8 At[4][2], B0[2][2], B1[2][2];

    STAMP(0);
    NEED(2);
    STAMP(1);
    {
        const unsigned x0 = XOFF(0), w0 = WOFF(0), x1 = XOFF(1), w1 = WOFF(1);
        STAGE(SB(0, 0), w0); STAGE(SA(0, 0), x0);
        STAGE(SB(0, 1), w0 + HALF_B); STAGE(SA(0, 1), x0 + HALF_B);
        if (wr == 1) BAR;
        WAIT_V(4); BAR;
        STAGE(SB(1, 0), w1); STAGE(SA(1, 0), x1); STAGE(SB(1, 1), w1 + HALF_B);
        WAIT_V(6); BAR;
    }
    for (int t = 0; t < NT - 2; t += 2) {
        const unsigned x1 = XOFF(t + 1), x2 = XOFF(t + 2), w2 = WOFF(t + 2), x3 = XOFF(t + 3), w3 = WOFF(t + 3);
        LDB(B0, 0, 0); SCHED; LDA(At, 0, 0); STAGE(SA(1, 1), x1 + HALF_B);
        WAIT_L(8); BAR; WAIT_L(0); MMA(0, 0, At, B0); BAR; SCHED;
        NEED(t + 3);
        LDB(B1, 0, 1); STAGE(SB(0, 0), w2);
        BAR; WAIT_L(0); MMA(0, 1, At, B1); BAR;
        LDA(At, 0, 1); STAGE(SA(0, 0), x2);
        BAR; WAIT_L(0); MMA(1, 0, At, B0); BAR; SCHED;
        STAGE(SB(0, 1), w2 + HALF_B);
        WAIT_V(6); BAR; MMA(1, 1, At, B1); BAR;
        LDB(B0, 1, 0); SCHED; LDA(At, 1, 0); STAGE(SA(0, 1), x2 + HALF_B);
        WAIT_L(8); BAR; WAIT_L(0); MMA(0, 0, At, B0); BAR; SCHED;
        NEED(t + 4);
        LDB(B1, 1, 1); STAGE(SB(1, 0), w3);
        BAR; WAIT_L(0); MMA(0, 1, At, B1); BAR;
        LDA(At, 1, 1); STAGE(SA(1, 0), x3);
        BAR; WAIT_L(0); MMA(1, 0, At, B0); BAR; SCHED;
        STAGE(SB(1, 1), w3 + HALF_B);
        WAIT_V(6); BAR; MMA(1, 1, At, B1); BAR;
    }
    STAMP(2);
    {
        const unsigned xl = XOFF(NT - 1);
        LDB(B0, 0, 0); WAIT_V(0); LDA(At, 0, 0); STAGE(SA(1, 1), xl + HALF_B);
        BAR; WAIT_L(0); MMA(0, 0, At, B0); BAR;
        LDB(B1, 0, 1); BAR; WAIT_L(0); MMA(0, 1, At, B1); BAR;
        LDA(At, 0, 1); WAIT_V(4); BAR; WAIT_L(0); MMA(1, 0, At, B0); MMA(1, 1, At, B1); BAR;
    }
    {
        LDB(B0, 1, 0); LDA(At, 1, 0); WAIT_V(2); BAR; WAIT_L(0); MMA(0, 0, At, B0); BAR;
        LDB(B1, 1, 1); WAIT_V(0); BAR; WAIT_L(0); MMA(0, 1, At, B1); BAR;
        LDA(At, 1, 1); BAR; WAIT_L(0); MMA(1, 0, At, B0); MMA(1, 1, At, B1); BAR;
    }
    if (wr == 0) BAR;
    STAMP(3);

    const int gid = rb * NCB + cb;
    volatile LAS float* red = (volatile LAS float*)(lds + LDS_STAGE + 64);
    float am = 0.f;
#pragma unroll
    for (int ai = 0; ai < 2; ++ai)
#pragma unroll
        for (int bj = 0; bj < 2; ++bj)
#pragma unroll
            for (int m = 0; m < 4; ++m)
#pragma unroll
                for (int n = 0; n < 2; ++n) {
                    const f32x4 v = acc[ai][bj][m][n];
                    am = fmaxf(am, fmaxf(fmaxf(fabsf(v[0]), fabsf(v[1])), fmaxf(fabsf(v[2]), fabsf(v[3]))));
                }
#pragma unroll
    for (int o = 32; o >= 1; o >>= 1) am = fmaxf(am, __shfl_xor(am, o));
    if (lane == 0) red[wid] = am;
    __syncthreads();
    if (wid == 0 && lane < NR) {
        float a = red[0];
#pragma unroll
        for (int i = 1; i < 8; ++i) a = fmaxf(a, red[i]);
        unsigned char* dws = p.ws[lane];
        __hip_atomic_store((unsigned long long*)(dws + AM_OFF) + rank * 128 + gid,
                           ((unsigned long long)call << 32) | __float_as_uint(a), __ATOMIC_RELAXED, __HIP_MEMORY_SCOPE_SYSTEM);
    }
    {
        const unsigned long long t0 = now();
        float g;
        for (;;) {
            const u32x4 v = __builtin_amdgcn_raw_buffer_load_b128(rws, AM_OFF + tid16, 0, AUX_SYS);
            g = fmaxf(__uint_as_float(v.x), __uint_as_float(v.z));
            if (__all(v.y == call && v.w == call)) break;
            if (give_up(tmo, call, t0)) break;
            __builtin_amdgcn_s_sleep(2);
        }
#pragma unroll
        for (int o = 32; o >= 1; o >>= 1) g = fmaxf(g, __shfl_xor(g, o));
        if (lane == 0) red[8 + wid] = g;
    }
    __syncthreads();
    float gmax = red[8];
#pragma unroll
    for (int i = 1; i < 8; ++i) gmax = fmaxf(gmax, red[8 + i]);
    const bool bad = __hip_atomic_load(tmo, __ATOMIC_RELAXED, __HIP_MEMORY_SCOPE_AGENT) == call;
    STAMP(4);

    const float scale = gmax / 448.0f;
    const float inv = 1.0f / scale;
    u32x4 qv[2][4];
#pragma unroll
    for (int ai = 0; ai < 2; ++ai)
#pragma unroll
        for (int m = 0; m < 4; ++m)
#pragma unroll
            for (int r = 0; r < 4; ++r) {
                const float a = __builtin_amdgcn_fmed3f(acc[ai][0][m][0][r] * inv, -448.f, 448.f);
                const float b = __builtin_amdgcn_fmed3f(acc[ai][0][m][1][r] * inv, -448.f, 448.f);
                const float c = __builtin_amdgcn_fmed3f(acc[ai][1][m][0][r] * inv, -448.f, 448.f);
                const float e = __builtin_amdgcn_fmed3f(acc[ai][1][m][1][r] * inv, -448.f, 448.f);
                int pk = __builtin_amdgcn_cvt_pk_fp8_f32(a, b, 0, false);
                pk = __builtin_amdgcn_cvt_pk_fp8_f32(c, e, pk, true);
                qv[ai][m][r] = bad ? 0x7f7f7f7fu : (unsigned)pk;
            }
    const unsigned ooff = (unsigned)(rb * 256 + wr * 64 + (lane & 15)) * N_ + rank * NB + cb * 256 + wc * 64 + (lane >> 4) * 16;
    const int rot = gid * 8 + wid;
    for (int j = 0; j < NR; ++j) {
        const int q = (j < NR - 1) ? 1 + (rot + j) % 7 : 0;
        const int d = (rank + q) & (NR - 1);
        const __amdgpu_buffer_rsrc_t ro = mk_rsrc(p.out[d], OUT_BYTES);
#pragma unroll
        for (int ai = 0; ai < 2; ++ai)
#pragma unroll
            for (int m = 0; m < 4; ++m)
                __builtin_amdgcn_raw_buffer_store_b128(qv[ai][m], ro, ooff, (unsigned)(ai * 128 + m * 16) * N_, AUX_SYS);
    }
    asm volatile("s_waitcnt vmcnt(0)" ::: "memory");
    __syncthreads();
    STAMP(5);
    if (wid == 0) {
        if (lane < NR) {
            unsigned char* dws = p.ws[lane];
            __hip_atomic_store((unsigned*)(dws + DN_OFF) + gid * 32 + rank, call, __ATOMIC_RELAXED, __HIP_MEMORY_SCOPE_SYSTEM);
        }
        const unsigned long long t0 = now();
        for (;;) {
            const unsigned v = __hip_atomic_load((unsigned*)(myws + DN_OFF) + gid * 32 + (lane & 7), __ATOMIC_RELAXED, __HIP_MEMORY_SCOPE_SYSTEM);
            if (__all(v == call)) break;
            if (give_up(tmo, call, t0)) {
                if (lane == 0) p.myout[ooff] = 0x7f;
                break;
            }
            __builtin_amdgcn_s_sleep(2);
        }
        __builtin_amdgcn_fence(__ATOMIC_ACQUIRE, "");
    }
    STAMP(6);
}

__global__ void __launch_bounds__(512, 2) fused_kernel(const Params p) {
    extern __shared__ __attribute__((aligned(16))) unsigned char smem[];
    const int b = blockIdx.x, xcd = b & 7, slot = b >> 3;
    const int wid = __builtin_amdgcn_readfirstlane(threadIdx.x >> 6), lane = threadIdx.x & 63;
    const bool is_push = slot < 16;
    const int q = slot & 7, wgi = xcd * 2 + (slot >> 3);
    const int g = slot - 16, rb = (xcd >> 1) * 4 + (g >> 2), cb = (xcd & 1) * 4 + (g & 3);
    if (is_push && wid >= PUSH_WAVES) return;
    unsigned char* myws = p.myws;
    unsigned* ctr = (unsigned*)(myws + CT_OFF) + b * 8 + wid;
    const unsigned call = __builtin_amdgcn_readfirstlane(__hip_atomic_load(ctr, __ATOMIC_RELAXED, __HIP_MEMORY_SCOPE_AGENT)) + 1;
    if (lane == 0) __hip_atomic_store(ctr, call, __ATOMIC_RELAXED, __HIP_MEMORY_SCOPE_AGENT);
    if (is_push) {
        STAMP(0);
        push_role(p, q, wgi, wid, lane, call);
        STAMP(1);
    } else {
        gemm_role(p, (LAS unsigned char*)smem, rb, cb, wid, lane, call);
    }
}

}

extern "C" void kernel_launch(int rank, int nranks, ncclComm_t comm, hipStream_t stream,
                              void* const* d_in, const int* in_sizes, int n_in,
                              void* d_out, int out_size,
                              const int* in_dtypes, int out_dtype,
                              void* const* d_out_all,
                              void* const* d_ws_all, size_t ws_bytes) {
    if (nranks != NR || (unsigned)rank >= (unsigned)NR || n_in < 2 || ws_bytes < WS_USED ||
        in_sizes[0] != M_ * KS || in_sizes[1] != KS * N_ || out_size != M_ * N_) {
        fprintf(stderr, "kernel_launch: unexpected problem shape\n");
        return;
    }
    static bool attr_set = false;
    if (!attr_set) {
        HIP_CHECK(hipFuncSetAttribute((const void*)fused_kernel, hipFuncAttributeMaxDynamicSharedMemorySize, LDS_BYTES));
        attr_set = true;
    }
    Params p;
    p.x = (const unsigned short*)d_in[0];
    p.w = (const unsigned short*)d_in[1];
    for (int i = 0; i < NR; ++i) {
        p.out[i] = (unsigned char*)d_out_all[i];
        p.ws[i] = (unsigned char*)d_ws_all[i];
    }
    p.myws = p.ws[rank];
    p.myout = (unsigned char*)d_out;
    p.rank = rank;
    hipLaunchKernelGGL(fused_kernel, dim3(256), dim3(512), LDS_BYTES, stream, p);
    HIP_CHECK(hipGetLastError());
}
```

```cpp
#include <hip/hip_runtime.h>
#include <rccl/rccl.h>
#include <cstdio>
#include <cstdint>

#define HIP_CHECK(expr) do { hipError_t _e = (expr); if (_e != hipSuccess) { \
    fprintf(stderr, "HIP error %s at %s:%d\n", hipGetErrorString(_e), __FILE__, __LINE__); \
    return; } } while (0)

namespace {

#define LAS __attribute__((address_space(3)))
typedef short bf16x8 __attribute__((ext_vector_type(8)));
typedef float f32x4 __attribute__((ext_vector_type(4)));
typedef unsigned u32x4 __attribute__((ext_vector_type(4)));
typedef unsigned u32x2 __attribute__((ext_vector_type(2)));

constexpr int NR = 8;
constexpr int M_ = 4096, K_ = 4096, N_ = 16384;
constexpr int KS = K_ / NR;
constexpr int NB = N_ / NR;
constexpr int NRB = 16, NCB = 8;
constexpr int NKC = 8;
constexpr int NT = 64;
constexpr unsigned TILE_B = 32768, HALF_B = 16384;

constexpr unsigned XS_OFF = 0;
constexpr unsigned WS_OFF = XS_OFF + 64u * NRB * TILE_B;
constexpr unsigned FX_OFF = WS_OFF + 64u * NCB * TILE_B;
constexpr unsigned FW_OFF = FX_OFF + NRB * 64 * 16;
constexpr unsigned AM_OFF = FW_OFF + NCB * 64 * 16;
constexpr unsigned DN_OFF = AM_OFF + NR * 128 * 8;
constexpr unsigned CT_OFF = DN_OFF + 1024 * 32;
constexpr unsigned TK_OFF = CT_OFF + 256 * 8 * 4;
constexpr unsigned TM_OFF = TK_OFF + NR * 128;
constexpr unsigned DB_OFF = TM_OFF + 256;
constexpr unsigned UL_OFF = DB_OFF + 512 * 64 * 8;
constexpr unsigned WS_USED = UL_OFF + 8 * 768 * 16;
constexpr unsigned OUT_BYTES = (unsigned)M_ * N_;

constexpr unsigned long long DEADLINE = 100000000ull;
constexpr int AUX_SYS = 17;

#define PUSH_WAVES 2
#define PACE_OUT -1
#define PACE_FINE 1800
#define LOCAL_AT 1
#define PACE_LEAD 20
#define PACE_IN 1450
#define NTAIL 0
#define TIMETABLE 1440
#define TT_START 100
#define JIT_FROM 640
#define SNAPSHOT 1
#define LOCAL_AUX 17
#define ACQ 0
#define STAGE_AUX 16
#define POLL_AUX 17
#define XCDMAP 1
#define ROLE_SPLIT 0

#define COLMAP 1
__host__ __device__ __forceinline__ constexpr unsigned col0(int r, int cb) {
    return COLMAP ? (unsigned)(cb * 8 + ((r + cb) & 7)) * 256u : (unsigned)(r * 8 + cb) * 256u;
}

constexpr int LDS_STAGE = 8 * HALF_B;
constexpr int LDS_BYTES = LDS_STAGE + 256;

struct Params {
    const unsigned short* x;
    const unsigned short* w;
    unsigned char* out[NR];
    unsigned char* ws[NR];
    unsigned char* myws;
    unsigned char* myout;
    int rank;
};

__device__ __forceinline__ unsigned char* uni(const void* p) {
    const unsigned long long v = (unsigned long long)p;
    const unsigned lo = __builtin_amdgcn_readfirstlane((unsigned)v), hi = __builtin_amdgcn_readfirstlane((unsigned)(v >> 32));
    return (unsigned char*)(((unsigned long long)hi << 32) | lo);
}
__device__ __forceinline__ __amdgpu_buffer_rsrc_t mk_rsrc(const void* p, unsigned bytes) {
    return __builtin_amdgcn_make_buffer_rsrc((void*)uni(p), 0, (int)bytes, 0x00020000);
}
__device__ __forceinline__ float wave_max(float v) {
#pragma unroll
    for (int o = 32; o >= 1; o >>= 1) v = fmaxf(v, __shfl_xor(v, o));
    return v;
}
__device__ __forceinline__ unsigned long long now() { return __builtin_amdgcn_s_memrealtime(); }

#define STAMP(i) do { } while (0)
#define STAMPMAX(i) do { } while (0)

#define ARR(r) do { } while (0)

__device__ __forceinline__ unsigned ld_tmo(unsigned* tmo) { return __hip_atomic_load(tmo, __ATOMIC_RELAXED, __HIP_MEMORY_SCOPE_AGENT); }
__device__ __forceinline__ bool give_up(unsigned* tmo, unsigned tmv, unsigned call, unsigned long long t0) {
    bool g = tmv == call || now() - t0 > DEADLINE;
    g = __builtin_amdgcn_readfirstlane(g ? 1 : 0) != 0;
    if (g) __hip_atomic_store(tmo, call, __ATOMIC_RELAXED, __HIP_MEMORY_SCOPE_AGENT);
    return g;
}

constexpr int NUNIT = NKC * 96;
constexpr int NW = 16 * PUSH_WAVES;
__device__ __forceinline__ void push_role(const Params& p, int q, int wgi, int wid, int lane, unsigned* ctr) {
    const int rank = p.rank;
    const int d = (rank + q) & (NR - 1);
    unsigned char* myws = p.myws;
    const __amdgpu_buffer_rsrc_t rd = mk_rsrc(p.ws[d], WS_USED);
    const __amdgpu_buffer_rsrc_t rx = mk_rsrc(p.x, (unsigned)M_ * KS * 2);
    const __amdgpu_buffer_rsrc_t rw = mk_rsrc(p.w, (unsigned)KS * N_ * 2);
    const unsigned char* tk = myws + TK_OFF + q * 128;
    const unsigned l0 = lane == 0 ? 0u : 0x80000000u;
    const int fr = lane & 15, fq = lane >> 4;
    const unsigned xoff = (unsigned)(fr * KS + fq * 8) * 2;
    const unsigned woff = (unsigned)(fq * 8 * N_ + fr * 4) * 2;
    const unsigned soff = lane * 16;
    u32x4 buf[8];
#define DRAW(T) asm volatile("s_mov_b64 exec, 1\n\tglobal_atomic_add %0, %1, %2, %3 sc0\n\ts_mov_b64 exec, -1" : "=&v"(T) : "v"(0u), "v"(1u), "s"(tk) : "memory")
#define FETCH(U) do { const int kc_ = (U) / 96, j_ = (U) - kc_ * 96; \
        if (j_ < 32) { \
            const unsigned sb_ = (unsigned)((kc_ * 64) * N_ + col0(d, j_ >> 2) + (j_ & 3) * 64) * 2; \
            _Pragma("unroll") for (int i_ = 0; i_ < 16; ++i_) { \
                const u32x2 v_ = __builtin_amdgcn_raw_buffer_load_b64(rw, woff, sb_ + (unsigned)(((i_ >> 3) * 32 + (i_ & 7)) * N_) * 2, 0); \
                buf[i_ >> 1][(i_ & 1) * 2] = v_.x; buf[i_ >> 1][(i_ & 1) * 2 + 1] = v_.y; } \
        } else { \
            const int jj_ = j_ - 32, rb_ = ((jj_ >> 2) + 2 * q) & 15; \
            const unsigned sb_ = (unsigned)((rb_ * 256 + (jj_ & 3) * 64) * KS + kc_ * 64) * 2; \
            _Pragma("unroll") for (int g_ = 0; g_ < 8; ++g_) \
                buf[g_] = __builtin_amdgcn_raw_buffer_load_b128(rx, xoff, sb_ + (unsigned)((g_ >> 1) * 16 * KS + (g_ & 1) * 32) * 2, 0); \
        } } while (0)
    const unsigned long long tstart = now();
    const unsigned cprev = __hip_atomic_load(ctr, __ATOMIC_RELAXED, __HIP_MEMORY_SCOPE_AGENT);
    unsigned u = (unsigned)(wid * 16 + wgi);
    unsigned tnext;
    FETCH((int)u);
    const unsigned call = __builtin_amdgcn_readfirstlane(cprev) + 1;
    if (lane == 0) __hip_atomic_store(ctr, call, __ATOMIC_RELAXED, __HIP_MEMORY_SCOPE_AGENT);
    const unsigned tbase = (call - 1) * (unsigned)NUNIT - NW;
    if (TIMETABLE == 0 && PACE_IN > 0 && q != 0) { const unsigned dl = u * PACE_IN / 100; while ((unsigned)(now() - tstart) < dl) __builtin_amdgcn_s_sleep(1); }
    for (;;) {
        const int kc = (int)u / 96, j = (int)u - kc * 96;
        unsigned flag_off;
        const bool jit = u >= (unsigned)JIT_FROM;
        if (TIMETABLE > 0) {
            if (q != 0) {
                const unsigned dl = TT_START + u * TIMETABLE / 100;
                for (;;) {
                    const int left = (int)(dl - (unsigned)(now() - tstart));
                    if (left <= 0) break;
                    if (left > 30) __builtin_amdgcn_s_sleep(8); else __builtin_amdgcn_s_sleep(1);
                }
            }
        } else if (!jit) DRAW(tnext);
        if (j < 32) {
            const int cb = j >> 2, wc = j & 3;
            const unsigned dbase = WS_OFF + (unsigned)((rank * NKC + kc) * NCB + cb) * TILE_B + wc * 4096;
#pragma unroll
            for (int kh = 0; kh < 2; ++kh)
#pragma unroll
                for (int f = 0; f < 4; ++f) {
                    u32x4 o;
#pragma unroll
                    for (int dd = 0; dd < 4; ++dd)
                        o[dd] = __builtin_amdgcn_perm(buf[kh * 4 + dd][2 + (f >> 1)], buf[kh * 4 + dd][f >> 1], (f & 1) ? 0x07060302u : 0x05040100u);
                    __builtin_amdgcn_raw_buffer_store_b128(o, rd, soff, dbase + (f >> 1) * HALF_B + ((f & 1) * 2 + kh) * 1024, AUX_SYS);
                }
            flag_off = FW_OFF + (unsigned)((cb * 64 + kc * 8 + rank) * 4 + wc) * 4;
        } else {
            const int jj = j - 32, rb = ((jj >> 2) + 2 * q) & 15, uq = jj & 3;
            const unsigned dbase = XS_OFF + (unsigned)((rank * NKC + kc) * NRB + rb) * TILE_B + uq * 8192;
#pragma unroll
            for (int g = 0; g < 8; ++g)
                __builtin_amdgcn_raw_buffer_store_b128(buf[g], rd, soff, dbase + g * 1024, AUX_SYS);
            flag_off = FX_OFF + (unsigned)((rb * 64 + kc * 8 + rank) * 4 + uq) * 4;
        }
        unsigned un;
        if (TIMETABLE > 0) {
            un = u + NW;
            asm volatile("" ::: "memory");
            if (un < (unsigned)NUNIT) {
                FETCH((int)un);
                asm volatile("" ::: "memory");
                if ((int)un % 96 < 32) asm volatile("s_waitcnt vmcnt(16)" ::: "memory");
                else asm volatile("s_waitcnt vmcnt(8)" ::: "memory");
            } else {
                asm volatile("s_waitcnt vmcnt(0)" ::: "memory");
            }
            __builtin_amdgcn_raw_buffer_store_b32(call, rd, l0, flag_off, AUX_SYS);
        } else if (!jit) {
            asm volatile("s_waitcnt vmcnt(8)" : "+v"(tnext) :: "memory");
            un = __builtin_amdgcn_readfirstlane(tnext) - tbase;
            if (un < (unsigned)NUNIT) {
                FETCH((int)un);
                asm volatile("" ::: "memory");
                if ((int)un % 96 < 32) asm volatile("s_waitcnt vmcnt(16)" ::: "memory");
                else asm volatile("s_waitcnt vmcnt(8)" ::: "memory");
            } else {
                asm volatile("s_waitcnt vmcnt(0)" ::: "memory");
            }
            __builtin_amdgcn_raw_buffer_store_b32(call, rd, l0, flag_off, AUX_SYS);
        } else {
            asm volatile("s_waitcnt vmcnt(0)" ::: "memory");
            DRAW(tnext);
            __builtin_amdgcn_raw_buffer_store_b32(call, rd, l0, flag_off, AUX_SYS);
            asm volatile("s_waitcnt vmcnt(1)" : "+v"(tnext) :: "memory");
            un = __builtin_amdgcn_readfirstlane(tnext) - tbase;
            if (un < (unsigned)NUNIT) FETCH((int)un);
        }
        const bool more = un < (unsigned)NUNIT;
        if (!more) break;
        u = un;
    }
    STAMPMAX(1);
#undef DRAW
#undef FETCH
}

#define WAIT_V(n) asm volatile("s_waitcnt vmcnt(" #n ")" ::: "memory")
#define WAIT_L(n) asm volatile("s_waitcnt lgkmcnt(" #n ")" ::: "memory")
#define BAR __builtin_amdgcn_s_barrier()
#define SCHED __builtin_amdgcn_sched_barrier(0)

__device__ __forceinline__ void gemm_role(const Params& p, LAS unsigned char* lds, int rb, int cb, int wid, int lane, unsigned call) {
    const int rank = p.rank;
    unsigned char* myws = p.myws;
    unsigned* tmo = (unsigned*)(myws + TM_OFF);
    const __amdgpu_buffer_rsrc_t rws = mk_rsrc(myws, WS_USED);
    const int wr = wid >> 2, wc = wid & 3;
    const unsigned tid16 = (unsigned)(wid * 64 + lane) * 16;
    volatile LAS int* sync_word = (volatile LAS int*)(lds + LDS_STAGE);
    unsigned char* peer_ws = p.ws[lane & 7];

    int tiv;
    {
        int s = rank, kc = lane;
        if (lane >= 8) {
            const int j = lane - 8, i = j % 7, q0 = rb >> 1;
            kc = j / 7;
            const int q = (i < q0) ? q0 - i : 7 - (i - q0);
            s = (rank - q) & (NR - 1);
        }
        tiv = s * 8 + kc;
    }
    const unsigned xtile0 = XS_OFF + rb * TILE_B, wtile0 = WS_OFF + cb * TILE_B;
    unsigned long long avail = 0, taken = 0;

    u32x4 sfx = {0u, 0u, 0u, 0u}, sfw = {0u, 0u, 0u, 0u};
    const unsigned char* wsu = uni(myws);
    const unsigned fi = (unsigned)((tiv & 7) * 8 + (tiv >> 3)) * 16;
    const unsigned fxo = FX_OFF + rb * 1024 + fi, fwo = FW_OFF + cb * 1024 + fi;
#define FLAGS_OK(fx, fw) ((((fx.x ^ call) | (fx.y ^ call) | (fx.z ^ call) | (fx.w ^ call)) | ((fw.x ^ call) | (fw.y ^ call) | (fw.z ^ call) | (fw.w ^ call))) == 0)
#define PICK(T, SLOT, SNAP) do { if (avail == 0) { \
        if (wid == 0) { \
            asm volatile("" : "+v"(sfx), "+v"(sfw)); \
            unsigned long long r = __ballot(FLAGS_OK(sfx, sfw)) & ~taken; \
            if (!r) { \
                const unsigned long long t0 = now(); \
                for (;;) { \
                    const unsigned tmv = ld_tmo(tmo); \
                    const u32x4 fx = __builtin_amdgcn_raw_buffer_load_b128(rws, fxo, 0, POLL_AUX); \
                    const u32x4 fw = __builtin_amdgcn_raw_buffer_load_b128(rws, fwo, 0, POLL_AUX); \
                    r = __ballot(FLAGS_OK(fx, fw)) & ~taken; \
                    if (r) break; \
                    if (give_up(tmo, tmv, call, t0)) { r = ~taken; break; } \
                    __builtin_amdgcn_s_sleep(2); \
                } \
            } \
            ARR(r); \
            if (ACQ) { __builtin_amdgcn_fence(__ATOMIC_ACQUIRE, ""); asm volatile("s_waitcnt vmcnt(0)" ::: "memory"); } \
            if (lane == 0) { sync_word[2 * (SLOT)] = (int)(unsigned)r; sync_word[2 * (SLOT) + 1] = (int)(unsigned)(r >> 32); } \
            asm volatile("s_waitcnt lgkmcnt(0)" ::: "memory"); \
        } \
        asm volatile("" ::: "memory"); BAR; asm volatile("" ::: "memory"); \
        const unsigned lo_ = __builtin_amdgcn_readfirstlane(sync_word[2 * (SLOT)]), hi_ = __builtin_amdgcn_readfirstlane(sync_word[2 * (SLOT) + 1]); \
        avail = ((unsigned long long)hi_ << 32) | lo_; \
        asm volatile("s_waitcnt lgkmcnt(0)" ::: "memory"); \
    } \
    { const int i_ = __builtin_ctzll(avail); const unsigned long long b_ = 1ull << i_; avail &= ~b_; taken |= b_; T = (unsigned)__builtin_amdgcn_readlane(tiv, i_); } \
    if ((SNAP) && avail == 0 && wid == 0) \
        asm volatile("global_load_dwordx4 %0, %2, %4 sc0 sc1\n\tglobal_load_dwordx4 %1, %3, %4 sc0 sc1" : "+v"(sfx), "+v"(sfw) : "v"(fxo), "v"(fwo), "s"(wsu) : "memory"); \
    } while (0)

#define SA(b, h) (((b) * 2 + (h)) * HALF_B)
#define SB(b, h) ((4 + (b) * 2 + (h)) * HALF_B)
#define XOFF(T) (xtile0 + (T) * (NRB * TILE_B))
#define WOFF(T) (wtile0 + (T) * (NCB * TILE_B))
#define STAGE(P, SRC) do { \
        __builtin_amdgcn_raw_ptr_buffer_load_lds(rws, (LAS void*)(lds + (P) + wid * 1024), 16, tid16, (SRC), 0, STAGE_AUX); \
        __builtin_amdgcn_raw_ptr_buffer_load_lds(rws, (LAS void*)(lds + (P) + 8192 + wid * 1024), 16, tid16, (SRC) + 8192, 0, STAGE_AUX); } while (0)
#define LDA(dst, b, h) _Pragma("unroll") for (int m = 0; m < 4; ++m) _Pragma("unroll") for (int k = 0; k < 2; ++k) \
        dst[m][k] = *(const LAS bf16x8*)(abase + SA(b, h) + (m * 2 + k) * 1024)
#define LDB(dst, b, h) _Pragma("unroll") for (int n = 0; n < 2; ++n) _Pragma("unroll") for (int k = 0; k < 2; ++k) \
        dst[n][k] = *(const LAS bf16x8*)(bbase + SB(b, h) - 4 * HALF_B + (n * 2 + k) * 1024)
#define MMA(ai, bj, At, Bt) do { __builtin_amdgcn_s_setprio(1); \
        _Pragma("unroll") for (int m = 0; m < 4; ++m) _Pragma("unroll") for (int n = 0; n < 2; ++n) _Pragma("unroll") for (int k = 0; k < 2; ++k) \
            acc[ai][bj][m][n] = __builtin_amdgcn_mfma_f32_16x16x32_bf16(Bt[n][k], At[m][k], acc[ai][bj][m][n], 0, 0, 0); \
        __builtin_amdgcn_s_setprio(0); } while (0)

    const LAS unsigned char* abase = lds + wr * 8192 + lane * 16;
    const LAS unsigned char* bbase = lds + 4 * HALF_B + wc * 4096 + lane * 16;
    f32x4 acc[2][2][4][2] = {};
    bf16x8 At[4][2], B0[2][2], B1[2][2];

    unsigned ta, tb;
    PICK(ta, 0, 0);
    STAMP(1);
    {
        const unsigned x0 = XOFF(ta), w0 = WOFF(ta);
        STAGE(SB(0, 0), w0); STAGE(SA(0, 0), x0);
        STAGE(SB(0, 1), w0 + HALF_B); STAGE(SA(0, 1), x0 + HALF_B);
        PICK(tb, 1, SNAPSHOT);
        const unsigned x1 = XOFF(tb), w1 = WOFF(tb);
        if (wr == 1) BAR;
        WAIT_V(4); BAR;
        STAGE(SB(1, 0), w1); STAGE(SA(1, 0), x1); STAGE(SB(1, 1), w1 + HALF_B);
        WAIT_V(6); BAR;
    }
    for (int t = 0; t < NT - NTAIL - 2; t += 2) {
        STAMP(8 + (t >> 1));
        LDB(B0, 0, 0); SCHED; LDA(At, 0, 0); STAGE(SA(1, 1), XOFF(tb) + HALF_B);
        WAIT_L(8); BAR; WAIT_L(0); MMA(0, 0, At, B0); BAR; SCHED;
        PICK(ta, 2, SNAPSHOT);
        const unsigned x2 = XOFF(ta), w2 = WOFF(ta);
        LDB(B1, 0, 1); STAGE(SB(0, 0), w2);
        BAR; WAIT_L(0); MMA(0, 1, At, B1); BAR;
        LDA(At, 0, 1); STAGE(SA(0, 0), x2);
        BAR; WAIT_L(0); MMA(1, 0, At, B0); BAR; SCHED;
        STAGE(SB(0, 1), w2 + HALF_B);
        WAIT_V(6); BAR; MMA(1, 1, At, B1); BAR;
        LDB(B0, 1, 0); SCHED; LDA(At, 1, 0); STAGE(SA(0, 1), x2 + HALF_B);
        WAIT_L(8); BAR; WAIT_L(0); MMA(0, 0, At, B0); BAR; SCHED;
        PICK(tb, 3, SNAPSHOT);
        const unsigned x3 = XOFF(tb), w3 = WOFF(tb);
        LDB(B1, 1, 1); STAGE(SB(1, 0), w3);
        BAR; WAIT_L(0); MMA(0, 1, At, B1); BAR;
        LDA(At, 1, 1); STAGE(SA(1, 0), x3);
        BAR; WAIT_L(0); MMA(1, 0, At, B0); BAR; SCHED;
        STAGE(SB(1, 1), w3 + HALF_B);
        WAIT_V(6); BAR; MMA(1, 1, At, B1); BAR;
    }
    STAMP(2);
    {
        const unsigned xl = XOFF(tb);
        LDB(B0, 0, 0); WAIT_V(0); LDA(At, 0, 0); STAGE(SA(1, 1), xl + HALF_B);
        BAR; WAIT_L(0); MMA(0, 0, At, B0); BAR;
        LDB(B1, 0, 1); BAR; WAIT_L(0); MMA(0, 1, At, B1); BAR;
        LDA(At, 0, 1); WAIT_V(4); BAR; WAIT_L(0); MMA(1, 0, At, B0); MMA(1, 1, At, B1); BAR;
    }
    {
        LDB(B0, 1, 0); LDA(At, 1, 0); WAIT_V(2); BAR; WAIT_L(0); MMA(0, 0, At, B0); BAR;
        LDB(B1, 1, 1); WAIT_V(0); BAR; WAIT_L(0); MMA(0, 1, At, B1); BAR;
        LDA(At, 1, 1); BAR; WAIT_L(0); MMA(1, 0, At, B0); MMA(1, 1, At, B1); BAR;
    }
    if (wr == 0) BAR;
    STAMP(7);
    if (NTAIL > 0) {
#define STAGE_TILE(BO, T) do { const unsigned x_ = XOFF(T), w_ = WOFF(T); \
        STAGE(SB(0, 0) + (BO), w_); STAGE(SA(0, 0) + (BO), x_); STAGE(SB(0, 1) + (BO), w_ + HALF_B); STAGE(SA(0, 1) + (BO), x_ + HALF_B); } while (0)
        int bo = 0;
        PICK(ta, 0, 0);
        STAGE_TILE(0, ta);
        for (int i = 0; i < NTAIL; ++i) {
            const bool pre = i + 1 < NTAIL && avail != 0;
            STAMP(40 + 3 * i);
            if (pre) { PICK(ta, 1, 0); STAGE_TILE(bo ^ (2 * HALF_B), ta); WAIT_V(8); } else { WAIT_V(0); }
            BAR; asm volatile("" ::: "memory");
            STAMP(41 + 3 * i);
            {
                const LAS unsigned char* abase0 = abase; const LAS unsigned char* bbase0 = bbase;
                const LAS unsigned char* abase = abase0 + bo; const LAS unsigned char* bbase = bbase0 + bo;
                LDB(B0, 0, 0); LDA(At, 0, 0); LDB(B1, 0, 1);
                MMA(0, 0, At, B0); MMA(0, 1, At, B1);
                LDA(At, 0, 1);
                MMA(1, 0, At, B0); MMA(1, 1, At, B1);
            }
            asm volatile("" ::: "memory"); BAR; asm volatile("" ::: "memory");
            STAMP(42 + 3 * i);
            if (!pre && i + 1 < NTAIL) { PICK(ta, 2 + (i & 1), 0); STAGE_TILE(bo ^ (2 * HALF_B), ta); }
            bo ^= 2 * HALF_B;
        }
    }
    STAMP(3);

    const int gid = rb * NCB + cb;
    volatile LAS float* red = (volatile LAS float*)(lds + LDS_STAGE + 64);
    float am = 0.f;
#pragma unroll
    for (int ai = 0; ai < 2; ++ai)
#pragma unroll
        for (int bj = 0; bj < 2; ++bj)
#pragma unroll
            for (int m = 0; m < 4; ++m)
#pragma unroll
                for (int n = 0; n < 2; ++n) {
                    const f32x4 v = acc[ai][bj][m][n];
                    am = fmaxf(fmaxf(am, fabsf(v[0])), fabsf(v[1]));
                    am = fmaxf(fmaxf(am, fabsf(v[2])), fabsf(v[3]));
                }
    am = wave_max(am);
    if (lane == 0) red[wid] = am;
    WAIT_L(0); BAR; asm volatile("" ::: "memory");
    if (wid == 0) {
        float a = red[lane & 7];
        a = fmaxf(a, __shfl_xor(a, 1)); a = fmaxf(a, __shfl_xor(a, 2)); a = fmaxf(a, __shfl_xor(a, 4));
        if (lane < NR)
            __hip_atomic_store((unsigned long long*)(peer_ws + AM_OFF) + rank * 128 + gid,
                               ((unsigned long long)call << 32) | __float_as_uint(a), __ATOMIC_RELAXED, __HIP_MEMORY_SCOPE_SYSTEM);
    }
    {
        const unsigned long long t0 = now();
        float g; bool bd;
        for (;;) {
            const unsigned tmv = ld_tmo(tmo);
            const u32x4 v = __builtin_amdgcn_raw_buffer_load_b128(rws, AM_OFF + tid16, 0, AUX_SYS);
            g = fmaxf(__uint_as_float(v.x), __uint_as_float(v.z));
            bd = tmv == call;
            if (__all(v.y == call && v.w == call)) break;
            if (give_up(tmo, tmv, call, t0)) { bd = true; break; }
            __builtin_amdgcn_s_sleep(1);
        }
        g = wave_max(g);
        if (lane == 0) red[8 + wid] = bd ? __builtin_nanf("") : g;
    }
    WAIT_L(0); BAR; asm volatile("" ::: "memory");
    const unsigned long long tout = now();
    float gmax = 0.f; bool bad = false;
#pragma unroll
    for (int i = 0; i < 8; ++i) { const float r = red[8 + i]; bad |= r != r; gmax = fmaxf(gmax, r); }
    STAMP(4);

    const float scale = gmax / 448.0f;
    const float inv = 1.0f / scale;
    u32x4 qv[2][4];
    {
        const int fr = lane & 15, fq = lane >> 4;
#pragma unroll
        for (int ai = 0; ai < 2; ++ai)
#pragma unroll
            for (int m = 0; m < 4; ++m) {
                u32x4 o;
                const f32x4 a = acc[ai][0][m][0] * inv, b = acc[ai][0][m][1] * inv, c = acc[ai][1][m][0] * inv, e = acc[ai][1][m][1] * inv;
#pragma unroll
                for (int r = 0; r < 4; ++r) {
                    int pk = __builtin_amdgcn_cvt_pk_fp8_f32(a[r], b[r], 0, false);
                    pk = __builtin_amdgcn_cvt_pk_fp8_f32(c[r], e[r], pk, true);
                    o[r] = bad ? 0x7f7f7f7fu : (unsigned)pk;
                }
                *(LAS u32x4*)(lds + (ai * 128 + wr * 64 + m * 16 + fr) * 256 + (((wc * 4 + fq) ^ fr) * 16)) = o;
            }
        WAIT_L(0); BAR; asm volatile("" ::: "memory");
#pragma unroll
        for (int i = 0; i < 8; ++i) {
            const int row = wid * 32 + i * 4 + fq;
            qv[i >> 2][i & 3] = *(const LAS u32x4*)(lds + row * 256 + ((fr ^ (row & 15)) * 16));
        }
    }
    const unsigned ooff = (unsigned)(rb * 256 + wid * 32 + (lane >> 4)) * N_ + col0(rank, cb) + (lane & 15) * 16;
    const unsigned l0 = lane == 0 ? 0u : 0x80000000u;
    const int wix = wid * 128 + gid;
    const unsigned dn = DN_OFF + (unsigned)(gid * 8 + wid) * 32;
    if (PACE_OUT < 0) {
        const int rot = gid * 8 + wid;
        __amdgpu_buffer_rsrc_t ro[NR - 1];
#pragma unroll
        for (int j = 0; j < NR - 1; ++j) ro[j] = mk_rsrc(p.out[(rank + 1 + (rot + j) % 7) & (NR - 1)], OUT_BYTES);
        const __amdgpu_buffer_rsrc_t rl = mk_rsrc(p.myout, OUT_BYTES);
#pragma unroll
        for (int i = 0; i < 8; ++i) {
            if (PACE_FINE > 0) {
                const unsigned dl = (unsigned)(i * 1024 + wix) * PACE_FINE / 1000;
                for (;;) {
                    const int left = (int)(dl - (unsigned)(now() - tout));
                    if (left <= 0) break;
                    if (left > 40) __builtin_amdgcn_s_sleep(10); else __builtin_amdgcn_s_sleep(1);
                }
            }
#pragma unroll
            for (int j = 0; j < NR - 1; ++j)
                __builtin_amdgcn_raw_buffer_store_b128(qv[i >> 2][i & 3], ro[j], ooff, (unsigned)(i * 4) * N_, AUX_SYS);
            if (LOCAL_AT == 0) __builtin_amdgcn_raw_buffer_store_b128(qv[i >> 2][i & 3], rl, ooff, (unsigned)(i * 4) * N_, LOCAL_AUX);
            if (LOCAL_AT == 2 && i == 3) {
#pragma unroll
                for (int k = 0; k < 8; ++k)
                    __builtin_amdgcn_raw_buffer_store_b128(qv[k >> 2][k & 3], rl, ooff, (unsigned)(k * 4) * N_, LOCAL_AUX);
            }
        }
        if (LOCAL_AT == 1) {
#pragma unroll
            for (int i = 0; i < 8; ++i)
                __builtin_amdgcn_raw_buffer_store_b128(qv[i >> 2][i & 3], rl, ooff, (unsigned)(i * 4) * N_, LOCAL_AUX);
        }
        asm volatile("s_waitcnt vmcnt(0)" ::: "memory");
        if (lane < NR) __hip_atomic_store((unsigned*)(peer_ws + dn) + rank, call, __ATOMIC_RELAXED, __HIP_MEMORY_SCOPE_SYSTEM);
    } else {
        const int qw = (1024 - wix + 145) / 146;
        for (int j = 0; j < NR; ++j) {
            if (j == 1) {
                const __amdgpu_buffer_rsrc_t rl = mk_rsrc(p.myout, OUT_BYTES);
#pragma unroll
                for (int i = 0; i < 8; ++i)
                    __builtin_amdgcn_raw_buffer_store_b128(qv[i >> 2][i & 3], rl, ooff, (unsigned)(i * 4) * N_, 0);
                continue;
            }
            const int jj = j == 0 ? 0 : j - 1;
            const int q = 1 + ((qw <= 7 ? qw - 1 : 0) + jj) % 7;
            const int d = (rank + q) & (NR - 1);
            const int slot = (wix + 146 * q) & 1023;
            const __amdgpu_buffer_rsrc_t ro = mk_rsrc(p.out[d], OUT_BYTES);
            const __amdgpu_buffer_rsrc_t rf = mk_rsrc(p.ws[d], WS_USED);
            if (PACE_OUT > 0 && slot > PACE_LEAD) {
                const unsigned dl = (unsigned)(slot - PACE_LEAD) * PACE_OUT / 100;
                for (;;) {
                    const int left = (int)(dl - (unsigned)(now() - tout));
                    if (left <= 0) break;
                    if (left > 40) __builtin_amdgcn_s_sleep(10); else __builtin_amdgcn_s_sleep(1);
                }
            }
#pragma unroll
            for (int i = 0; i < 8; ++i)
                __builtin_amdgcn_raw_buffer_store_b128(qv[i >> 2][i & 3], ro, ooff, (unsigned)(i * 4) * N_, AUX_SYS);
            asm volatile("s_waitcnt vmcnt(0)" ::: "memory");
            __builtin_amdgcn_raw_buffer_store_b32(call, rf, l0, dn + rank * 4, AUX_SYS);
        }
    }
    STAMPMAX(5);
    {
        const unsigned long long t0 = now();
        for (;;) {
            const unsigned tmv = ld_tmo(tmo);
            const unsigned v = __builtin_amdgcn_raw_buffer_load_b32(rws, (lane & 7) * 4, dn, AUX_SYS);
            if (__all(v == call || (lane & 7) == rank)) break;
            if (give_up(tmo, tmv, call, t0)) { if (lane == 0) p.myout[ooff] = 0x7f; break; }
            __builtin_amdgcn_s_sleep(2);
        }
    }
    STAMPMAX(6);
}

__global__ void __launch_bounds__(512, 2) fused_kernel(const Params p) {
    extern __shared__ __attribute__((aligned(16))) unsigned char smem[];
    const int b = blockIdx.x, xcd = b & 7, slot = b >> 3;
    const int wid = __builtin_amdgcn_readfirstlane(threadIdx.x >> 6), lane = threadIdx.x & 63;
    const bool is_push = slot < 16;
    const int q = slot & 7, wgi = xcd * 2 + (slot >> 3);
    const int g = slot - 16;
    const int rb = xcd * 2 + (g >> 3), cb = g & 7;
    if (is_push && wid >= PUSH_WAVES) return;
    unsigned char* myws = p.myws;
    unsigned* ctr = (unsigned*)(myws + CT_OFF) + b * 8 + wid;
    STAMP(0);
    if (is_push) {
        push_role(p, q, wgi, wid, lane, ctr);
    } else {
        const unsigned call = __builtin_amdgcn_readfirstlane(__hip_atomic_load(ctr, __ATOMIC_RELAXED, __HIP_MEMORY_SCOPE_AGENT)) + 1;
        if (lane == 0) __hip_atomic_store(ctr, call, __ATOMIC_RELAXED, __HIP_MEMORY_SCOPE_AGENT);
        gemm_role(p, (LAS unsigned char*)smem, rb, cb, wid, lane, call);
    }
}

}

extern "C" void kernel_launch(int rank, int nranks, ncclComm_t comm, hipStream_t stream,
                              void* const* d_in, const int* in_sizes, int n_in,
                              void* d_out, int out_size,
                              const int* in_dtypes, int out_dtype,
                              void* const* d_out_all,
                              void* const* d_ws_all, size_t ws_bytes) {
    if (nranks != NR || (unsigned)rank >= (unsigned)NR || n_in < 2 || ws_bytes < WS_USED ||
        in_sizes[0] != M_ * KS || in_sizes[1] != KS * N_ || out_size != M_ * N_) {
        fprintf(stderr, "kernel_launch: unexpected problem shape\n");
        return;
    }
    static bool attr_set = false;
    if (!attr_set) {
        HIP_CHECK(hipFuncSetAttribute((const void*)fused_kernel, hipFuncAttributeMaxDynamicSharedMemorySize, LDS_BYTES));
        attr_set = true;
    }
    Params p;
    p.x = (const unsigned short*)d_in[0];
    p.w = (const unsigned short*)d_in[1];
    for (int i = 0; i < NR; ++i) {
        p.out[i] = (unsigned char*)d_out_all[i];
        p.ws[i] = (unsigned char*)d_ws_all[i];
    }
    p.myws = p.ws[rank];
    p.myout = (unsigned char*)d_out;
    p.rank = rank;
    hipLaunchKernelGGL(fused_kernel, dim3(256), dim3(512), LDS_BYTES, stream, p);
    HIP_CHECK(hipGetLastError());
}
```

```cpp
#include <hip/hip_runtime.h>
#include <rccl/rccl.h>
#include <cstdio>
#include <cstdint>

#define HIP_CHECK(expr) do { hipError_t _e = (expr); if (_e != hipSuccess) { \
    fprintf(stderr, "HIP error %s at %s:%d\n", hipGetErrorString(_e), __FILE__, __LINE__); \
    return; } } while (0)

namespace {

#define LAS __attribute__((address_space(3)))
typedef short bf16x8 __attribute__((ext_vector_type(8)));
typedef float f32x4 __attribute__((ext_vector_type(4)));
typedef unsigned u32x4 __attribute__((ext_vector_type(4)));
typedef unsigned u32x2 __attribute__((ext_vector_type(2)));

constexpr int NR = 8;
constexpr int M_ = 4096, K_ = 4096, N_ = 16384;
constexpr int KS = K_ / NR;
constexpr int NB = N_ / NR;
constexpr int NRB = 16, NCB = 8;
constexpr int NKC = 8;
constexpr int NT = 64;
constexpr unsigned TILE_B = 32768, HALF_B = 16384;
constexpr int NUNIT = NKC * 96;
constexpr int NOUT = 1024;

constexpr unsigned XS_OFF = 0;
constexpr unsigned WS_OFF = XS_OFF + 64u * NRB * TILE_B;
constexpr unsigned EW_OFF = WS_OFF + 64u * NCB * TILE_B;
constexpr unsigned FX_OFF = EW_OFF + 64u * NCB * TILE_B;
constexpr unsigned FW_OFF = FX_OFF + NRB * 64 * 16;
constexpr unsigned EF_OFF = FW_OFF + NCB * 64 * 16;
constexpr unsigned OF_OFF = EF_OFF + NR * NUNIT * 4;
constexpr unsigned AM_OFF = OF_OFF + 8 * NR * 128 * 4;
constexpr unsigned DN_OFF = AM_OFF + NR * 128 * 8;
constexpr unsigned CT_OFF = DN_OFF + 1024 * 32;
constexpr unsigned TK_OFF = CT_OFF + 256 * 8 * 4;
constexpr unsigned TM_OFF = TK_OFF + 2 * NR * 128;
constexpr unsigned DB_OFF = TM_OFF + 256;
constexpr unsigned UL_OFF = DB_OFF + 512 * 64 * 8;
constexpr unsigned WS_USED = UL_OFF + 8 * 2048 * 16;
constexpr unsigned OUT_BYTES = (unsigned)M_ * N_;

constexpr unsigned long long DEADLINE = 100000000ull;
constexpr int AUX_SYS = 17, AUX_DEV = 16;

#define NW 18
#define NWO 16
#define STAGE2_DELAY 450
#define STAGE2_STEP 60

#define PACE_IN 1340
#define PACE_OUT 1340
#define TT_START 100
#define NPUSH0 16
#define NPUSH1 16
#define PACE_FINE 1800
#define EXP_LEAD 900
#define FLAG_NOW 128
#define POLL_AUX AUX_DEV
#define XCDMAP 0
#define STAGE2_AT 100
#define RB_MIN 115
#define RB_TGT 160
#define LST_AUX AUX_DEV
#define PICK_SLEEP 8
#define RLD_AUX AUX_SYS
#define SNAPSHOT 0
constexpr int LINK_WAVES = 2;
constexpr int NEXP = 112 * (8 - LINK_WAVES) + 128;
static_assert(NW >= 8 && NW <= 32 && NWO >= 8 && NWO <= 32 && NPUSH0 <= 16 && NPUSH1 <= 112, "");

constexpr int LDS_STAGE = 8 * HALF_B;
constexpr int LDS_BYTES = LDS_STAGE + 256;

struct Params {
    const unsigned short* x;
    const unsigned short* w;
    unsigned char* out[NR];
    unsigned char* ws[NR];
    unsigned char* myws;
    unsigned char* myout;
    int rank;
};

__device__ __forceinline__ unsigned char* uni(const void* p) {
    const unsigned long long v = (unsigned long long)p;
    const unsigned lo = __builtin_amdgcn_readfirstlane((unsigned)v), hi = __builtin_amdgcn_readfirstlane((unsigned)(v >> 32));
    return (unsigned char*)(((unsigned long long)hi << 32) | lo);
}
__device__ __forceinline__ __amdgpu_buffer_rsrc_t mk_rsrc(const void* p, unsigned bytes) {
    return __builtin_amdgcn_make_buffer_rsrc((void*)uni(p), 0, (int)bytes, 0x00020000);
}
#define ST128(V, R, VO, SO, AUX) do { const u32x4 st_v_ = (V); __builtin_amdgcn_raw_buffer_store_b128(st_v_, (R), (VO), (SO), (AUX)); \
        asm volatile("s_nop 1" :: "v"(st_v_)); } while (0)
__device__ __forceinline__ float wave_max(float v) {
#pragma unroll
    for (int o = 32; o >= 1; o >>= 1) v = fmaxf(v, __shfl_xor(v, o));
    return v;
}
__device__ __forceinline__ unsigned long long now() { return __builtin_amdgcn_s_memrealtime(); }

#define STAMP(i) do { } while (0)
#define STAMPMAX(i) do { } while (0)

#define ARR(r) do { } while (0)

__device__ __forceinline__ unsigned ld_tmo(unsigned* tmo) { return __hip_atomic_load(tmo, __ATOMIC_RELAXED, __HIP_MEMORY_SCOPE_AGENT); }
__device__ __forceinline__ bool give_up(unsigned* tmo, unsigned tmv, unsigned call, unsigned long long t0) {
    bool g = tmv == call || now() - t0 > DEADLINE;
    g = __builtin_amdgcn_readfirstlane(g ? 1 : 0) != 0;
    if (g) __hip_atomic_store(tmo, call, __ATOMIC_RELAXED, __HIP_MEMORY_SCOPE_AGENT);
    return g;
}

__device__ __forceinline__ void wait_until(unsigned long long t0, unsigned dl) {
    for (;;) {
        const int left = (int)(dl - (unsigned)(now() - t0));
        if (left <= 0) break;
        if (left > 30) __builtin_amdgcn_s_sleep(8); else __builtin_amdgcn_s_sleep(1);
    }
}

__device__ __forceinline__ void load_w(u32x4 (&buf)[8], __amdgpu_buffer_rsrc_t rw, int kc, unsigned c0, int lane) {
    const unsigned woff = (unsigned)((lane >> 4) * 8 * N_ + (lane & 15) * 4) * 2, sb = ((unsigned)(kc * 64) * N_ + c0) * 2;
#pragma unroll
    for (int i = 0; i < 16; ++i) {
        const u32x2 v = __builtin_amdgcn_raw_buffer_load_b64(rw, woff, sb + (unsigned)(((i >> 3) * 32 + (i & 7)) * N_) * 2, 0);
        buf[i >> 1][(i & 1) * 2] = v.x; buf[i >> 1][(i & 1) * 2 + 1] = v.y;
    }
}
__device__ __forceinline__ void store_w(const u32x4 (&buf)[8], __amdgpu_buffer_rsrc_t rd, unsigned dbase, int lane) {
#pragma unroll
    for (int kh = 0; kh < 2; ++kh)
#pragma unroll
        for (int f = 0; f < 4; ++f) {
            u32x4 o;
#pragma unroll
            for (int dd = 0; dd < 4; ++dd)
                o[dd] = __builtin_amdgcn_perm(buf[kh * 4 + dd][2 + (f >> 1)], buf[kh * 4 + dd][f >> 1], (f & 1) ? 0x07060302u : 0x05040100u);
            ST128(o, rd, lane * 16, dbase + (f >> 1) * HALF_B + ((f & 1) * 2 + kh) * 1024, AUX_SYS);
        }
}

__device__ __forceinline__ void export_role(const Params& p, int eidx, int push_q, int push_u, int wid, int lane, unsigned* ctr) {
    const unsigned long long tstart = now();
    const int rank = p.rank;
    unsigned char* myws = p.myws;
    const unsigned cprev = __hip_atomic_load(ctr, __ATOMIC_RELAXED, __HIP_MEMORY_SCOPE_AGENT);
    unsigned char* peer = p.ws[(rank + lane) & (NR - 1)];
    const __amdgpu_buffer_rsrc_t rme = mk_rsrc(myws, WS_USED);
    const __amdgpu_buffer_rsrc_t rx = mk_rsrc(p.x, (unsigned)M_ * KS * 2);
    const __amdgpu_buffer_rsrc_t rw = mk_rsrc(p.w, (unsigned)KS * N_ * 2);
    u32x4 buf[8];
    if (push_q != 0) load_w(buf, rw, 0, (unsigned)(((rank + push_q) & (NR - 1)) * NB + push_u * 64), lane);
    const unsigned call = __builtin_amdgcn_readfirstlane(cprev) + 1;
    if (lane == 0) __hip_atomic_store(ctr, call, __ATOMIC_RELAXED, __HIP_MEMORY_SCOPE_AGENT);
    if (push_q != 0) {
        const __amdgpu_buffer_rsrc_t rd = mk_rsrc(p.ws[(rank + push_q) & (NR - 1)], WS_USED);
        wait_until(tstart, TT_START + (unsigned)push_u * 144 / 10);
        store_w(buf, rd, WS_OFF + (unsigned)(rank * NKC * NCB + (push_u >> 2)) * TILE_B + (push_u & 3) * 4096, lane);
        asm volatile("s_waitcnt vmcnt(0)" ::: "memory");
        __builtin_amdgcn_raw_buffer_store_b32((call & 0xffffu) * 0x10001u, rd, lane == 0 ? 0u : 0x80000000u,
                                              FW_OFF + (unsigned)(((push_u >> 2) * 64 + rank) * 4 + (push_u & 3)) * 4, AUX_SYS);
    }
    for (int e = eidx; e < NKC * 320; e += NEXP) {
        const int kc = e / 320, r = e - kc * 320;
        unsigned foff; int flane;
        if (kc > 0) wait_until(tstart, (unsigned)(kc * 96 * PACE_IN / 100 - EXP_LEAD + r * 2));
        else if (r >= 256) wait_until(tstart, 150u);
        if (r < 256) {
            const int j = r >> 3, q = r & 7, cb = j >> 2, wc = j & 3;
            if (kc == 0 && j < NPUSH0 && q != 0) continue;
            load_w(buf, rw, kc, (unsigned)(((rank + q) & (NR - 1)) * NB + j * 64), lane);
            store_w(buf, rme, (q == 0 ? WS_OFF + (unsigned)(rank * NKC * NCB) * TILE_B : EW_OFF + (unsigned)(q * NKC * NCB) * TILE_B)
                              + (unsigned)(kc * NCB + cb) * TILE_B + wc * 4096, lane);
            flane = q;
            foff = q == 0 ? FW_OFF + (unsigned)((cb * 64 + kc * 8 + rank) * 4 + wc) * 4 : EF_OFF + (unsigned)(q * NUNIT + kc * 96 + j) * 4;
        } else {
            const int rb = (r - 256) >> 2, uq = r & 3;
            const unsigned xoff = (unsigned)((lane & 15) * KS + (lane >> 4) * 8) * 2, sb = (unsigned)((rb * 256 + uq * 64) * KS + kc * 64) * 2;
            const unsigned dbase = XS_OFF + (unsigned)((rank * NKC + kc) * NRB + rb) * TILE_B + uq * 8192;
#pragma unroll
            for (int g = 0; g < 8; ++g)
                buf[g] = __builtin_amdgcn_raw_buffer_load_b128(rx, xoff, sb + (unsigned)((g >> 1) * 16 * KS + (g & 1) * 32) * 2, 0);
#pragma unroll
            for (int g = 0; g < 8; ++g)
                ST128(buf[g], rme, lane * 16, dbase + g * 1024, AUX_SYS);
            flane = -1;
            foff = lane == 0 ? FX_OFF + (unsigned)((rb * 64 + kc * 8 + rank) * 4 + uq) * 4
                             : EF_OFF + (unsigned)(lane * NUNIT + kc * 96 + 32 + ((rb - 2 * lane) & 15) * 4 + uq) * 4;
        }
        asm volatile("s_waitcnt vmcnt(0)" ::: "memory");
        if (flane < 0 ? lane < NR : lane == flane)
            __hip_atomic_store((unsigned*)(peer + foff), lane == 0 ? (call & 0xffffu) * 0x10001u : call, __ATOMIC_RELAXED, __HIP_MEMORY_SCOPE_SYSTEM);
    }
    STAMPMAX(3);
}

__device__ __forceinline__ void link_role(const Params& p, int q, int widx, int wid, int lane, unsigned* ctr) {
    unsigned long long tstart = now();
    const int rank = p.rank;
    const int src = (rank - q) & (NR - 1);
    unsigned char* myws = p.myws;
    unsigned* tmo = (unsigned*)(myws + TM_OFF);
    const unsigned cprev = __hip_atomic_load(ctr, __ATOMIC_RELAXED, __HIP_MEMORY_SCOPE_AGENT);
    const __amdgpu_buffer_rsrc_t rme = mk_rsrc(myws, WS_USED);
    const unsigned l0 = lane == 0 ? 0u : 0x80000000u;
    const unsigned soff = lane * 16;
    const unsigned call = __builtin_amdgcn_readfirstlane(cprev) + 1;
    if (lane == 0) __hip_atomic_store(ctr, call, __ATOMIC_RELAXED, __HIP_MEMORY_SCOPE_AGENT);
    u32x4 bufa[4], bufb[4];
    const unsigned char* tk = uni(myws + TK_OFF + q * 128);
    unsigned tick;
#define DRAW() asm volatile("s_mov_b64 exec, 1\n\tglobal_atomic_add %0, %1, %2, %3 sc0\n\ts_mov_b64 exec, -1" : "=&v"(tick) : "v"(0u), "v"(1u), "s"(tk) : "memory")
#define DUE(T0, DL) ((int)((DL) - (unsigned)(now() - (T0))) <= 0)

    if (widx < NW) {
        const __amdgpu_buffer_rsrc_t rs = mk_rsrc(p.ws[src], WS_USED);
#define WAIT_EXPORTED(FO, COND) do { const unsigned long long t0_ = now(); \
            for (;;) { \
                const unsigned tmv = ld_tmo(tmo); \
                const u32x2 v = __builtin_amdgcn_raw_buffer_load_b64(rme, (FO), EF_OFF + (unsigned)(q * NUNIT) * 4, AUX_SYS); \
                if (__all(COND)) break; \
                if (give_up(tmo, tmv, call, t0_)) break; \
                __builtin_amdgcn_s_sleep(2); \
            } } while (0)
#define IN_ADDR(H, SB, DB, FO) do { const int u_ = (H) >> 1, hf_ = (H) & 1, kc_ = u_ / 96, j_ = u_ - kc_ * 96; \
            if (j_ < 32) { \
                const unsigned o_ = (unsigned)(kc_ * NCB + (j_ >> 2)) * TILE_B + (j_ & 3) * 4096 + hf_ * HALF_B; \
                SB = EW_OFF + (unsigned)(q * NKC * NCB) * TILE_B + o_; DB = WS_OFF + (unsigned)(src * NKC * NCB) * TILE_B + o_; \
                FO = FW_OFF + (unsigned)(((j_ >> 2) * 64 + kc_ * 8 + src) * 8 + (j_ & 3) * 2 + hf_) * 2; \
            } else { \
                const int rb_ = (((j_ - 32) >> 2) + 2 * q) & 15; \
                SB = DB = XS_OFF + (unsigned)((src * NKC + kc_) * NRB + rb_) * TILE_B + (j_ & 3) * 8192 + hf_ * 4096; \
                FO = FX_OFF + (unsigned)((rb_ * 64 + kc_ * 8 + src) * 8 + (j_ & 3) * 2 + hf_) * 2; \
            } } while (0)
#define IN_LOAD(BUF, SB) do { _Pragma("unroll") for (int g = 0; g < 4; ++g) \
            BUF[g] = __builtin_amdgcn_raw_buffer_load_b128(rs, soff, (SB) + g * 1024, AUX_SYS); } while (0)
#define FLAG(FO) __builtin_amdgcn_raw_buffer_store_b16((unsigned short)call, rme, l0, (FO), AUX_DEV)
        DRAW();
        WAIT_EXPORTED((unsigned)(NPUSH0 + (lane < (32 - NPUSH0) / 2 ? lane * 2 : 0)) * 4, v.x == call && v.y == call);
        {
            const int late = (int)((unsigned)(now() - tstart) - (TT_START + (unsigned)NPUSH0 * PACE_IN / 100));
            if (late > 0) tstart += (unsigned)late;
        }
        const unsigned tb = (call - 1) * (unsigned)(2 * (NUNIT - NPUSH0)) - (2 * NPUSH0 + NW);
        int u_ok = 32;
        unsigned fprev = 0;
        unsigned sb, db, fo, dbn, fon;
        IN_ADDR(2 * NPUSH0 + widx, sb, db, fo);
        wait_until(tstart, TT_START + (unsigned)(2 * NPUSH0 + widx) * PACE_IN / 200);
        IN_LOAD(bufa, sb);
#define IN_STEP(CUR, NXT) { \
              \
            asm volatile("s_waitcnt vmcnt(0)" : "+v"(tick), "+v"(CUR[0]), "+v"(CUR[1]), "+v"(CUR[2]), "+v"(CUR[3]) :: "memory"); \
            const int hn = (int)(__builtin_amdgcn_readfirstlane(tick) - tb); \
            const bool valid = hn < 2 * NUNIT; \
            while ((hn >> 1) >= u_ok && u_ok < NUNIT) {                      \
                const int n2 = (u_ok == 32 ? 64 : 96) / 2; \
                WAIT_EXPORTED((unsigned)(u_ok + (lane < n2 ? lane * 2 : 0)) * 4, v.x == call && v.y == call); \
                u_ok += 2 * n2; \
            } \
            IN_ADDR(hn, sb, dbn, fon); \
            const unsigned dl = TT_START + (unsigned)hn * PACE_IN / 200; \
            const bool early = valid && DUE(tstart, dl); \
            if (early) IN_LOAD(NXT, sb); \
            asm volatile("" ::: "memory"); \
            fprev = __builtin_amdgcn_readfirstlane(fprev); db = __builtin_amdgcn_readfirstlane(db); fo = __builtin_amdgcn_readfirstlane(fo); \
            if (fprev) FLAG(fprev); \
            _Pragma("unroll") for (int g = 0; g < 4; ++g) ST128(CUR[g], rme, soff, db + g * 1024, AUX_DEV); \
            fprev = fo; \
            if (valid) DRAW(); \
            if (hn >= 2 * NUNIT - FLAG_NOW && !early) {                      \
                asm volatile("s_waitcnt vmcnt(0)" : "+v"(tick) :: "memory"); \
                FLAG(fprev); fprev = 0; \
            } \
            if (!valid) break; \
            if (!early) { wait_until(tstart, dl); IN_LOAD(NXT, sb); } \
            db = dbn; fo = fon; }
        for (;;) {
            IN_STEP(bufa, bufb)
            IN_STEP(bufb, bufa)
        }
#undef WAIT_EXPORTED
    }
    STAMPMAX(1);

    if (widx < NWO) {
        const __amdgpu_buffer_rsrc_t rd = mk_rsrc(p.myout, OUT_BYTES);
        bool bad;
#define WAIT_TILES(STAGE) do { const unsigned long long t0_ = now(); \
            for (;;) { \
                const unsigned tmv = ld_tmo(tmo); \
                const u32x2 v = __builtin_amdgcn_raw_buffer_load_b64(rme, lane * 8, OF_OFF + ((STAGE) * NR + q) * 512, AUX_SYS); \
                bad = __builtin_amdgcn_readfirstlane(tmv) == call; \
                if (__all(v.x == call && v.y == call)) break; \
                if (give_up(tmo, tmv, call, t0_)) { bad = true; break; } \
                __builtin_amdgcn_s_sleep(3); \
            } \
              \
            bm = __builtin_amdgcn_readfirstlane(bad ? 0x7f7f7f7f : 0); \
            rs = mk_rsrc(p.out[src], bm ? 0u : OUT_BYTES); } while (0)
#define OUT_OFF(K) ((unsigned)(((NPUSH1 + ((K) >> 1)) & 15) * 256 + (((NPUSH1 + ((K) >> 1)) >> 4) & 7) * 32 + ((NPUSH1 + ((K) >> 1)) >> 7) * 4 + ((K) & 1) * 2) * N_ + src * NB)
#define OUT_LOAD(BUF, OB) do { _Pragma("unroll") for (int g = 0; g < 4; ++g) \
            BUF[g] = __builtin_amdgcn_raw_buffer_load_b128(rs, soff, (OB) + (g >> 1) * N_ + (g & 1) * 1024, AUX_SYS); } while (0)
        unsigned bm;
        __amdgpu_buffer_rsrc_t rs;
        tk += NR * 128;
        DRAW();
        WAIT_TILES(0);
        const unsigned long long tf = now();
        STAMPMAX(2);
        const unsigned tb = (call - 1) * (unsigned)(2 * (NOUT - NPUSH1)) - NWO;
        bool stage2 = false;
        unsigned ob = OUT_OFF(widx);
        wait_until(tf, (unsigned)widx * PACE_OUT / 200);
        OUT_LOAD(bufa, ob);
#define OUT_STEP(CUR, NXT) { \
            asm volatile("s_waitcnt vmcnt(0)" : "+v"(tick), "+v"(CUR[0]), "+v"(CUR[1]), "+v"(CUR[2]), "+v"(CUR[3]) :: "memory"); \
            const int kn = (int)(__builtin_amdgcn_readfirstlane(tick) - tb); \
            const bool valid = kn < 2 * (NOUT - NPUSH1); \
            const unsigned bmc = bm; \
            if (!stage2 && NPUSH1 + (kn >> 1) >= STAGE2_AT) { WAIT_TILES(1); stage2 = true; } \
            const unsigned obn = OUT_OFF(kn), dl = (unsigned)kn * PACE_OUT / 200; \
            const bool early = valid && DUE(tf, dl); \
            if (early) OUT_LOAD(NXT, obn); \
            asm volatile("" ::: "memory"); \
            ob = __builtin_amdgcn_readfirstlane(ob); \
            _Pragma("unroll") for (int g = 0; g < 4; ++g) ST128(CUR[g] | bmc, rd, soff, ob + (g >> 1) * N_ + (g & 1) * 1024, AUX_DEV); \
            if (!valid) break; \
            DRAW(); \
            if (!early) { wait_until(tf, dl); OUT_LOAD(NXT, obn); } \
            ob = obn; }
        for (;;) {
            OUT_STEP(bufa, bufb)
            OUT_STEP(bufb, bufa)
        }
        asm volatile("s_waitcnt vmcnt(0)" ::: "memory");
    }
    STAMPMAX(6);
}

#define WAIT_V(n) asm volatile("s_waitcnt vmcnt(" #n ")" ::: "memory")
#define WAIT_L(n) asm volatile("s_waitcnt lgkmcnt(" #n ")" ::: "memory")
#define BAR __builtin_amdgcn_s_barrier()
#define SCHED __builtin_amdgcn_sched_barrier(0)

__device__ __forceinline__ void gemm_role(const Params& p, LAS unsigned char* lds, int rb, int cb, int wid, int lane, unsigned call) {
    const int rank = p.rank;
    unsigned char* myws = p.myws;
    unsigned* tmo = (unsigned*)(myws + TM_OFF);
    const __amdgpu_buffer_rsrc_t rws = mk_rsrc(myws, WS_USED);
    const int wr = wid >> 2, wc = wid & 3;
    const unsigned tid16 = (unsigned)(wid * 64 + lane) * 16;
    volatile LAS int* sync_word = (volatile LAS int*)(lds + LDS_STAGE);
    unsigned char* peer_ws = p.ws[lane & 7];

    int tiv;
    {
        int s = rank, kc = lane;
        if (lane >= 8) {
            const int j = lane - 8, i = j % 7, q0 = rb >> 1;
            kc = j / 7;
            const int q = (i < q0) ? q0 - i : 7 - (i - q0);
            s = (rank - q) & (NR - 1);
        }
        tiv = s * 8 + kc;
    }
    const unsigned xtile0 = XS_OFF + rb * TILE_B, wtile0 = WS_OFF + cb * TILE_B;
    unsigned long long avail = 0, taken = 0;

    u32x4 sfx = {~call, ~call, ~call, ~call}, sfw = sfx;
    const unsigned char* wsu = uni(myws);
    const unsigned fi = (unsigned)((tiv & 7) * 8 + (tiv >> 3)) * 16;
    const unsigned fxo = FX_OFF + rb * 1024 + fi, fwo = FW_OFF + cb * 1024 + fi;
    const unsigned cc = (call & 0xffffu) * 0x10001u;
#define FLAGS_OK(fx, fw) ((((fx.x ^ cc) | (fx.y ^ cc) | (fx.z ^ cc) | (fx.w ^ cc)) | ((fw.x ^ cc) | (fw.y ^ cc) | (fw.z ^ cc) | (fw.w ^ cc))) == 0)
#define PICK(T, SLOT, SNAP) do { if (avail == 0) { \
        if (wid == 0) { \
            asm volatile("" : "+v"(sfx), "+v"(sfw)); \
            unsigned long long r = __ballot(FLAGS_OK(sfx, sfw)) & ~taken; \
            if (!r) { \
                const unsigned long long t0 = now(); \
                for (;;) { \
                    const unsigned tmv = ld_tmo(tmo); \
                    const u32x4 fx = __builtin_amdgcn_raw_buffer_load_b128(rws, fxo, 0, POLL_AUX); \
                    const u32x4 fw = __builtin_amdgcn_raw_buffer_load_b128(rws, fwo, 0, POLL_AUX); \
                    r = __ballot(FLAGS_OK(fx, fw)) & ~taken; \
                    if (r) break; \
                    if (give_up(tmo, tmv, call, t0)) { r = ~taken; break; } \
                    __builtin_amdgcn_s_sleep(PICK_SLEEP); \
                } \
            } \
            ARR(r); \
            if (lane == 0) { sync_word[2 * (SLOT)] = (int)(unsigned)r; sync_word[2 * (SLOT) + 1] = (int)(unsigned)(r >> 32); } \
            asm volatile("s_waitcnt lgkmcnt(0)" ::: "memory"); \
        } \
        asm volatile("" ::: "memory"); BAR; asm volatile("" ::: "memory"); \
        const unsigned lo_ = __builtin_amdgcn_readfirstlane(sync_word[2 * (SLOT)]), hi_ = __builtin_amdgcn_readfirstlane(sync_word[2 * (SLOT) + 1]); \
        avail = ((unsigned long long)hi_ << 32) | lo_; \
        asm volatile("s_waitcnt lgkmcnt(0)" ::: "memory"); \
    } \
    { const int i_ = __builtin_ctzll(avail); const unsigned long long b_ = 1ull << i_; avail &= ~b_; taken |= b_; T = (unsigned)__builtin_amdgcn_readlane(tiv, i_); } \
    if ((SNAP) && avail == 0 && wid == 0) \
        asm volatile("global_load_dwordx4 %0, %2, %4 sc1\n\tglobal_load_dwordx4 %1, %3, %4 sc1" : "+v"(sfx), "+v"(sfw) : "v"(fxo), "v"(fwo), "s"(wsu) : "memory"); \
    } while (0)

#define SA(b, h) (((b) * 2 + (h)) * HALF_B)
#define SB(b, h) ((4 + (b) * 2 + (h)) * HALF_B)
#define XOFF(T) (xtile0 + (T) * (NRB * TILE_B))
#define WOFF(T) (wtile0 + (T) * (NCB * TILE_B))
#define STAGE(P, SRC) do { \
        __builtin_amdgcn_raw_ptr_buffer_load_lds(rws, (LAS void*)(lds + (P) + wid * 1024), 16, tid16, (SRC), 0, AUX_DEV); \
        __builtin_amdgcn_raw_ptr_buffer_load_lds(rws, (LAS void*)(lds + (P) + 8192 + wid * 1024), 16, tid16, (SRC) + 8192, 0, AUX_DEV); } while (0)
#define LDA(dst, b, h) _Pragma("unroll") for (int m = 0; m < 4; ++m) _Pragma("unroll") for (int k = 0; k < 2; ++k) \
        dst[m][k] = *(const LAS bf16x8*)(abase + SA(b, h) + (m * 2 + k) * 1024)
#define LDB(dst, b, h) _Pragma("unroll") for (int n = 0; n < 2; ++n) _Pragma("unroll") for (int k = 0; k < 2; ++k) \
        dst[n][k] = *(const LAS bf16x8*)(bbase + SB(b, h) - 4 * HALF_B + (n * 2 + k) * 1024)
#define MMA(ai, bj, At, Bt) do { __builtin_amdgcn_s_setprio(1); \
        _Pragma("unroll") for (int m = 0; m < 4; ++m) _Pragma("unroll") for (int n = 0; n < 2; ++n) _Pragma("unroll") for (int k = 0; k < 2; ++k) \
            acc[ai][bj][m][n] = __builtin_amdgcn_mfma_f32_16x16x32_bf16(Bt[n][k], At[m][k], acc[ai][bj][m][n], 0, 0, 0); \
        __builtin_amdgcn_s_setprio(0); } while (0)

    const LAS unsigned char* abase = lds + wr * 8192 + lane * 16;
    const LAS unsigned char* bbase = lds + 4 * HALF_B + wc * 4096 + lane * 16;
    f32x4 acc[2][2][4][2] = {};
    bf16x8 At[4][2], B0[2][2], B1[2][2];

    unsigned ta, tb;
    PICK(ta, 0, 0);
    STAMP(1);
    {
        const unsigned x0 = XOFF(ta), w0 = WOFF(ta);
        STAGE(SB(0, 0), w0); STAGE(SA(0, 0), x0);
        STAGE(SB(0, 1), w0 + HALF_B); STAGE(SA(0, 1), x0 + HALF_B);
        PICK(tb, 1, SNAPSHOT);
        const unsigned x1 = XOFF(tb), w1 = WOFF(tb);
        if (wr == 1) BAR;
        WAIT_V(4); BAR;
        STAGE(SB(1, 0), w1); STAGE(SA(1, 0), x1); STAGE(SB(1, 1), w1 + HALF_B);
        WAIT_V(6); BAR;
    }
    for (int t = 0; t < NT - 2; t += 2) {
        STAMP(8 + (t >> 1));
        LDB(B0, 0, 0); SCHED; LDA(At, 0, 0); STAGE(SA(1, 1), XOFF(tb) + HALF_B);
        WAIT_L(8); BAR; WAIT_L(0); MMA(0, 0, At, B0); BAR; SCHED;
        PICK(ta, 2, SNAPSHOT);
        const unsigned x2 = XOFF(ta), w2 = WOFF(ta);
        LDB(B1, 0, 1); STAGE(SB(0, 0), w2);
        BAR; WAIT_L(0); MMA(0, 1, At, B1); BAR;
        LDA(At, 0, 1); STAGE(SA(0, 0), x2);
        BAR; WAIT_L(0); MMA(1, 0, At, B0); BAR; SCHED;
        STAGE(SB(0, 1), w2 + HALF_B);
        WAIT_V(6); BAR; MMA(1, 1, At, B1); BAR;
        LDB(B0, 1, 0); SCHED; LDA(At, 1, 0); STAGE(SA(0, 1), x2 + HALF_B);
        WAIT_L(8); BAR; WAIT_L(0); MMA(0, 0, At, B0); BAR; SCHED;
        PICK(tb, 3, SNAPSHOT);
        const unsigned x3 = XOFF(tb), w3 = WOFF(tb);
        LDB(B1, 1, 1); STAGE(SB(1, 0), w3);
        BAR; WAIT_L(0); MMA(0, 1, At, B1); BAR;
        LDA(At, 1, 1); STAGE(SA(1, 0), x3);
        BAR; WAIT_L(0); MMA(1, 0, At, B0); BAR; SCHED;
        STAGE(SB(1, 1), w3 + HALF_B);
        WAIT_V(6); BAR; MMA(1, 1, At, B1); BAR;
    }
    STAMP(2);
    {
        const unsigned xl = XOFF(tb);
        LDB(B0, 0, 0); WAIT_V(0); LDA(At, 0, 0); STAGE(SA(1, 1), xl + HALF_B);
        BAR; WAIT_L(0); MMA(0, 0, At, B0); BAR;
        LDB(B1, 0, 1); BAR; WAIT_L(0); MMA(0, 1, At, B1); BAR;
        LDA(At, 0, 1); WAIT_V(4); BAR; WAIT_L(0); MMA(1, 0, At, B0); MMA(1, 1, At, B1); BAR;
    }
    {
        LDB(B0, 1, 0); LDA(At, 1, 0); WAIT_V(2); BAR; WAIT_L(0); MMA(0, 0, At, B0); BAR;
        LDB(B1, 1, 1); WAIT_V(0); BAR; WAIT_L(0); MMA(0, 1, At, B1); BAR;
        LDA(At, 1, 1); BAR; WAIT_L(0); MMA(1, 0, At, B0); MMA(1, 1, At, B1); BAR;
    }
    if (wr == 0) BAR;
    STAMP(3);

    const int gid = rb * NCB + cb;
    volatile LAS float* red = (volatile LAS float*)(lds + LDS_STAGE + 64);
    float am = 0.f;
#pragma unroll
    for (int ai = 0; ai < 2; ++ai)
#pragma unroll
        for (int bj = 0; bj < 2; ++bj)
#pragma unroll
            for (int m = 0; m < 4; ++m)
#pragma unroll
                for (int n = 0; n < 2; ++n) {
                    const f32x4 v = acc[ai][bj][m][n];
                    am = fmaxf(fmaxf(am, fabsf(v[0])), fabsf(v[1]));
                    am = fmaxf(fmaxf(am, fabsf(v[2])), fabsf(v[3]));
                }
    am = wave_max(am);
    if (lane == 0) red[wid] = am;
    WAIT_L(0); BAR; asm volatile("" ::: "memory");
    if (wid == 0) {
        float a = red[lane & 7];
        a = fmaxf(a, __shfl_xor(a, 1)); a = fmaxf(a, __shfl_xor(a, 2)); a = fmaxf(a, __shfl_xor(a, 4));
        if (lane < NR)
            __hip_atomic_store((unsigned long long*)(peer_ws + AM_OFF) + rank * 128 + gid,
                               ((unsigned long long)call << 32) | __float_as_uint(a), __ATOMIC_RELAXED, __HIP_MEMORY_SCOPE_SYSTEM);
    }
    {
        const unsigned long long t0 = now();
        float g; bool bd;
        for (;;) {
            const unsigned tmv = ld_tmo(tmo);
            const u32x4 v = __builtin_amdgcn_raw_buffer_load_b128(rws, AM_OFF + tid16, 0, AUX_SYS);
            g = fmaxf(__uint_as_float(v.x), __uint_as_float(v.z));
            bd = tmv == call;
            if (__all(v.y == call && v.w == call)) break;
            if (give_up(tmo, tmv, call, t0)) { bd = true; break; }
            __builtin_amdgcn_s_sleep(1);
        }
        g = wave_max(g);
        if (lane == 0) red[8 + wid] = bd ? __builtin_nanf("") : g;
    }
    WAIT_L(0); BAR; asm volatile("" ::: "memory");
    const unsigned long long tout = now();
    float gmax = 0.f; bool bad = false;
#pragma unroll
    for (int i = 0; i < 8; ++i) { const float r = red[8 + i]; bad |= r != r; gmax = fmaxf(gmax, r); }
    STAMP(4);

    const float scale = gmax / 448.0f;
    const float inv = 1.0f / scale;
    u32x4 qv[2][4];
    {
        const int fr = lane & 15, fq = lane >> 4;
#pragma unroll
        for (int ai = 0; ai < 2; ++ai)
#pragma unroll
            for (int m = 0; m < 4; ++m) {
                u32x4 o;
                const f32x4 a = acc[ai][0][m][0] * inv, b = acc[ai][0][m][1] * inv, c = acc[ai][1][m][0] * inv, e = acc[ai][1][m][1] * inv;
#pragma unroll
                for (int r = 0; r < 4; ++r) {
                    int pk = __builtin_amdgcn_cvt_pk_fp8_f32(a[r], b[r], 0, false);
                    pk = __builtin_amdgcn_cvt_pk_fp8_f32(c[r], e[r], pk, true);
                    o[r] = bad ? 0x7f7f7f7fu : (unsigned)pk;
                }
                *(LAS u32x4*)(lds + (ai * 128 + wr * 64 + m * 16 + fr) * 256 + (((wc * 4 + fq) ^ fr) * 16)) = o;
            }
        WAIT_L(0); BAR; asm volatile("" ::: "memory");
#pragma unroll
        for (int i = 0; i < 8; ++i) {
            const int row = wid * 32 + i * 4 + fq;
            qv[i >> 2][i & 3] = *(const LAS u32x4*)(lds + row * 256 + ((fr ^ (row & 15)) * 16));
        }
    }
    const unsigned ooff = (unsigned)(rb * 256 + wid * 32 + (lane >> 4)) * N_ + rank * NB + cb * 256 + (lane & 15) * 16;
    const int wix = wid * 128 + gid;
    const bool pusher = (wix >> 3) < NPUSH1;
    const unsigned dl = (unsigned)wix * PACE_FINE / 1000;
    const bool early = pusher && (unsigned)(now() - tout) >= dl;
    const __amdgpu_buffer_rsrc_t rl = mk_rsrc(p.myout, OUT_BYTES);
    ST128(qv[0][0], rl, ooff, 0, AUX_SYS);
    asm volatile("" ::: "memory");
#define PUSH_CHUNK0() do { _Pragma("unroll") for (int j = 0; j < NR - 1; ++j) \
        ST128(qv[0][0], mk_rsrc(p.out[(rank + 1 + (gid + j) % 7) & (NR - 1)], OUT_BYTES), ooff, 0, AUX_SYS); } while (0)
    if (early) { PUSH_CHUNK0(); WAIT_V(7); } else { WAIT_V(0); }
    BAR; asm volatile("" ::: "memory");
    const int lq = ((lane & 7) - rank) & (NR - 1);
    if (wid == 7 && lane < NR && lq != 0)
        __hip_atomic_store((unsigned*)(peer_ws + OF_OFF) + lq * 128 + gid, call, __ATOMIC_RELAXED, __HIP_MEMORY_SCOPE_SYSTEM);
    STAMP(7);
    if (pusher && !early) { wait_until(tout, dl); PUSH_CHUNK0(); }
    wait_until(tout, (unsigned)(STAGE2_DELAY + wid * STAGE2_STEP));
#pragma unroll
    for (int i = 1; i < 8; ++i)
        ST128(qv[i >> 2][i & 3], rl, ooff, (unsigned)(i * 4) * N_, AUX_SYS);
    WAIT_V(0);
    if (pusher && lane < NR)
        __hip_atomic_store((unsigned*)(peer_ws + DN_OFF + (unsigned)(gid * 8 + wid) * 32) + lq, call, __ATOMIC_RELAXED, __HIP_MEMORY_SCOPE_SYSTEM);
    BAR; asm volatile("" ::: "memory");
    if (wid == 7 && lane < NR && lq != 0)
        __hip_atomic_store((unsigned*)(peer_ws + OF_OFF) + (NR + lq) * 128 + gid, call, __ATOMIC_RELAXED, __HIP_MEMORY_SCOPE_SYSTEM);
    STAMP(5);
    if (pusher) {
        const unsigned dn = DN_OFF + (unsigned)(gid * 8 + wid) * 32;
        const unsigned long long t0 = now();
        for (;;) {
            const unsigned tmv = ld_tmo(tmo);
            const unsigned v = __builtin_amdgcn_raw_buffer_load_b32(rws, (lane & 7) * 4, dn, AUX_SYS);
            if (__all(v == call)) break;
            if (give_up(tmo, tmv, call, t0)) { if (lane == 0) p.myout[ooff] = 0x7f; break; }
            __builtin_amdgcn_s_sleep(2);
        }
    }
    STAMPMAX(6);
}

__global__ void __launch_bounds__(512, 2) fused_kernel(const Params p) {
    extern __shared__ __attribute__((aligned(16))) unsigned char smem[];
    const int b = blockIdx.x, xcd = b & 7, slot = b >> 3;
    const int wid = __builtin_amdgcn_readfirstlane(threadIdx.x >> 6), lane = threadIdx.x & 63;
    unsigned char* myws = p.myws;
    unsigned* ctr = (unsigned*)(myws + CT_OFF) + b * 8 + wid;
    STAMP(0);
    if (slot < 16) {
        const int q = slot & 7, wgi = xcd * 2 + (slot >> 3);
        if (q != 0 && wid < LINK_WAVES) { if (wid * 16 + wgi < (NW > NWO ? NW : NWO)) link_role(p, q, wid * 16 + wgi, wid, lane, ctr); }
        else export_role(p, q == 0 ? wid * 16 + wgi : 128 + (wid - LINK_WAVES) * 112 + (q - 1) * 16 + wgi,
                         (wid == 7 && wgi < NPUSH0) ? q : 0, wgi, wid, lane, ctr);
    } else {
        const int g = slot - 16;
        const unsigned call = __builtin_amdgcn_readfirstlane(__hip_atomic_load(ctr, __ATOMIC_RELAXED, __HIP_MEMORY_SCOPE_AGENT)) + 1;
        if (lane == 0) __hip_atomic_store(ctr, call, __ATOMIC_RELAXED, __HIP_MEMORY_SCOPE_AGENT);
        if (XCDMAP) gemm_role(p, (LAS unsigned char*)smem, xcd * 2 + (g >> 3), g & 7, wid, lane, call);
        else gemm_role(p, (LAS unsigned char*)smem, (xcd >> 1) * 4 + (g >> 2), (xcd & 1) * 4 + (g & 3), wid, lane, call);
    }
}

}

extern "C" void kernel_launch(int rank, int nranks, ncclComm_t comm, hipStream_t stream,
                              void* const* d_in, const int* in_sizes, int n_in,
                              void* d_out, int out_size,
                              const int* in_dtypes, int out_dtype,
                              void* const* d_out_all,
                              void* const* d_ws_all, size_t ws_bytes) {
    if (nranks != NR || (unsigned)rank >= (unsigned)NR || n_in < 2 || ws_bytes < WS_USED ||
        in_sizes[0] != M_ * KS || in_sizes[1] != KS * N_ || out_size != M_ * N_) {
        fprintf(stderr, "kernel_launch: unexpected problem shape\n");
        return;
    }
    static bool attr_set = false;
    if (!attr_set) {
        HIP_CHECK(hipFuncSetAttribute((const void*)fused_kernel, hipFuncAttributeMaxDynamicSharedMemorySize, LDS_BYTES));
        attr_set = true;
    }
    Params p;
    p.x = (const unsigned short*)d_in[0];
    p.w = (const unsigned short*)d_in[1];
    for (int i = 0; i < NR; ++i) {
        p.out[i] = (unsigned char*)d_out_all[i];
        p.ws[i] = (unsigned char*)d_ws_all[i];
    }
    p.myws = p.ws[rank];
    p.myout = (unsigned char*)d_out;
    p.rank = rank;
    hipLaunchKernelGGL(fused_kernel, dim3(256), dim3(512), LDS_BYTES, stream, p);
    HIP_CHECK(hipGetLastError());
}
```

```cpp
#include <hip/hip_runtime.h>
#include <rccl/rccl.h>
#include <cstdio>
#include <cstdint>

#define HIP_CHECK(expr) do { hipError_t _e = (expr); if (_e != hipSuccess) { \
    fprintf(stderr, "HIP error %s at %s:%d\n", hipGetErrorString(_e), __FILE__, __LINE__); \
    return; } } while (0)

namespace {

#define LAS __attribute__((address_space(3)))
typedef short bf16x8 __attribute__((ext_vector_type(8)));
typedef float f32x4 __attribute__((ext_vector_type(4)));
typedef unsigned u32x4 __attribute__((ext_vector_type(4)));
typedef unsigned u32x2 __attribute__((ext_vector_type(2)));

constexpr int NR = 8;
constexpr int M_ = 4096, K_ = 4096, N_ = 16384;
constexpr int KS = K_ / NR;
constexpr int NB = N_ / NR;
constexpr int NRB = 16, NCB = 8;
constexpr int NKC = 8;
constexpr int NT = 64;
constexpr unsigned TILE_B = 32768, HALF_B = 16384;
constexpr int NUNIT = NKC * 96;
constexpr int NOUT = 1024;

constexpr unsigned XS_OFF = 0;
constexpr unsigned WS_OFF = XS_OFF + 64u * NRB * TILE_B;
constexpr unsigned EW_OFF = WS_OFF + 64u * NCB * TILE_B;
constexpr unsigned FX_OFF = EW_OFF + 64u * NCB * TILE_B;
constexpr unsigned FW_OFF = FX_OFF + NRB * 64 * 16;
constexpr unsigned EF_OFF = FW_OFF + NCB * 64 * 16;
constexpr unsigned OF_OFF = EF_OFF + NR * NUNIT * 4;
constexpr unsigned AM_OFF = OF_OFF + 8 * NR * 128 * 4;
constexpr unsigned DN_OFF = AM_OFF + NR * 128 * 8;
constexpr unsigned CT_OFF = DN_OFF + 1024 * 32;
constexpr unsigned TK_OFF = CT_OFF + 256 * 8 * 4;
constexpr unsigned EC_OFF = TK_OFF + 2 * NR * 128;
constexpr unsigned OC_OFF = EC_OFF + NKC * 128;
constexpr unsigned TS_OFF = OC_OFF + 17 * 128;
constexpr unsigned TM_OFF = TS_OFF + 128;
constexpr unsigned DB_OFF = TM_OFF + 256;
constexpr unsigned UL_OFF = DB_OFF + 512 * 64 * 8;
constexpr unsigned WS_USED = UL_OFF + 8 * 2048 * 16;
constexpr unsigned OUT_BYTES = (unsigned)M_ * N_;

constexpr unsigned long long DEADLINE = 100000000ull;
constexpr int AUX_SYS = 17, AUX_DEV = 16;

#define NW 18
#define NWO 14
#define STAGE2_DELAY 450
#define STAGE2_STEP 60

#define PACE_IN 1340
#define PACE_OUT 1340
#define TT_START 100
#define NPUSH0 16
#define NPUSH1 12
#define PACE_FINE 1800
#define EXP_LEAD 900
#define FLAG_NOW 128
#define POLL_AUX AUX_DEV
#define XCDMAP 0
#define STAGE2_AT 100
#define RB_MIN 115
#define RB_TGT 160
#define OFP_AUX AUX_SYS
#define OFP_SLEEP 3
#define AMP_AUX AUX_SYS
#define AMP_SLEEP 1
#define WEARLY 0
#define OUT_BASE 0
#define OUT_K0 200
#define OUT_EXTRA 0
#define DEEP_WAIT 1
#define GROUP_FLAGS 1
#define GROUP_OF1 1
#define LST_AUX AUX_DEV
#define PICK_SLEEP 20
#define FAST_FROM 54
#define RLD_AUX AUX_SYS
#define SNAPSHOT 1
constexpr int LINK_WAVES = 2;
constexpr int NEXP = 112 * (8 - LINK_WAVES) + 128;
static_assert(!(GROUP_FLAGS && WEARLY) && NPUSH1 % 4 == 0 && NPUSH1 <= 64 && NW >= 8 && NW <= 32 && NWO >= 8 && NWO <= 32 && NPUSH0 <= 16 && NPUSH1 <= 112, "");

constexpr int LDS_STAGE = 8 * HALF_B;
constexpr int LDS_BYTES = LDS_STAGE + 256;

struct Params {
    const unsigned short* x;
    const unsigned short* w;
    unsigned char* out[NR];
    unsigned char* ws[NR];
    unsigned char* myws;
    unsigned char* myout;
    int rank;
};

__device__ __forceinline__ unsigned char* uni(const void* p) {
    const unsigned long long v = (unsigned long long)p;
    const unsigned lo = __builtin_amdgcn_readfirstlane((unsigned)v), hi = __builtin_amdgcn_readfirstlane((unsigned)(v >> 32));
    return (unsigned char*)(((unsigned long long)hi << 32) | lo);
}
__device__ __forceinline__ __amdgpu_buffer_rsrc_t mk_rsrc(const void* p, unsigned bytes) {
    return __builtin_amdgcn_make_buffer_rsrc((void*)uni(p), 0, (int)bytes, 0x00020000);
}
#define ST128(V, R, VO, SO, AUX) do { const u32x4 st_v_ = (V); __builtin_amdgcn_raw_buffer_store_b128(st_v_, (R), (VO), (SO), (AUX)); \
        asm volatile("s_nop 1" :: "v"(st_v_)); } while (0)
__device__ __forceinline__ float wave_max(float f) {
    unsigned v = __float_as_uint(f);
    v = max(v, (unsigned)__builtin_amdgcn_update_dpp(0, (int)v, 0x111, 0xf, 0xf, false));
    v = max(v, (unsigned)__builtin_amdgcn_update_dpp(0, (int)v, 0x112, 0xf, 0xf, false));
    v = max(v, (unsigned)__builtin_amdgcn_update_dpp(0, (int)v, 0x114, 0xf, 0xf, false));
    v = max(v, (unsigned)__builtin_amdgcn_update_dpp(0, (int)v, 0x118, 0xf, 0xf, false));
    v = max(v, (unsigned)__builtin_amdgcn_update_dpp(0, (int)v, 0x142, 0xa, 0xf, false));
    v = max(v, (unsigned)__builtin_amdgcn_update_dpp(0, (int)v, 0x143, 0xc, 0xf, false));
    return __uint_as_float((unsigned)__builtin_amdgcn_readlane((int)v, 63));
}
__device__ __forceinline__ unsigned long long now() { return __builtin_amdgcn_s_memrealtime(); }

#define STAMP(i) do { } while (0)
#define STAMPMAX(i) do { } while (0)

#define ARR(r) do { } while (0)

__device__ __forceinline__ unsigned ld_tmo(unsigned* tmo) { return __hip_atomic_load(tmo, __ATOMIC_RELAXED, __HIP_MEMORY_SCOPE_AGENT); }
__device__ __forceinline__ bool give_up(unsigned* tmo, unsigned tmv, unsigned call, unsigned long long t0) {
    bool g = tmv == call || now() - t0 > DEADLINE;
    g = __builtin_amdgcn_readfirstlane(g ? 1 : 0) != 0;
    if (g) __hip_atomic_store(tmo, call, __ATOMIC_RELAXED, __HIP_MEMORY_SCOPE_AGENT);
    return g;
}

__device__ __forceinline__ void wait_until(unsigned long long t0, unsigned dl) {
    for (;;) {
        const int left = (int)(dl + (unsigned)t0 - (unsigned)now());
        if (left <= 0) break;
        if (left > 30) __builtin_amdgcn_s_sleep(8); else __builtin_amdgcn_s_sleep(1);
    }
}

__device__ __forceinline__ int list_pos(int kc, int j) {
    if (WEARLY && kc == 7 && j < 32) return 608 + j;
    if (WEARLY && kc == 6 && j >= 32) return 608 + j;
    return kc * 96 + j;
}
__device__ __forceinline__ void list_unit(int u, int& kc, int& j) {
    kc = u / 96; j = u - kc * 96;
    if (WEARLY && u >= 608 && u < 704) { kc = u < 640 ? 7 : 6; j = u - 608; }
}
__device__ __forceinline__ void load_w(u32x4 (&buf)[8], __amdgpu_buffer_rsrc_t rw, int kc, unsigned c0, int lane) {
    const unsigned woff = (unsigned)((lane >> 4) * 8 * N_ + (lane & 15) * 4) * 2, sb = ((unsigned)(kc * 64) * N_ + c0) * 2;
#pragma unroll
    for (int i = 0; i < 16; ++i) {
        const u32x2 v = __builtin_amdgcn_raw_buffer_load_b64(rw, woff, sb + (unsigned)(((i >> 3) * 32 + (i & 7)) * N_) * 2, 0);
        buf[i >> 1][(i & 1) * 2] = v.x; buf[i >> 1][(i & 1) * 2 + 1] = v.y;
    }
}
__device__ __forceinline__ void store_w(const u32x4 (&buf)[8], __amdgpu_buffer_rsrc_t rd, unsigned dbase, int lane) {
#pragma unroll
    for (int kh = 0; kh < 2; ++kh)
#pragma unroll
        for (int f = 0; f < 4; ++f) {
            u32x4 o;
#pragma unroll
            for (int dd = 0; dd < 4; ++dd)
                o[dd] = __builtin_amdgcn_perm(buf[kh * 4 + dd][2 + (f >> 1)], buf[kh * 4 + dd][f >> 1], (f & 1) ? 0x07060302u : 0x05040100u);
            ST128(o, rd, lane * 16, dbase + (f >> 1) * HALF_B + ((f & 1) * 2 + kh) * 1024, AUX_SYS);
        }
}

__device__ __forceinline__ void export_role(const Params& p, int eidx, int push_q, int push_u, int wid, int lane, unsigned* ctr) {
    const unsigned long long tstart = now();
    const int rank = p.rank;
    unsigned char* myws = p.myws;
    const unsigned cprev = __hip_atomic_load(ctr, __ATOMIC_RELAXED, __HIP_MEMORY_SCOPE_AGENT);
    unsigned char* peer = p.ws[(rank + lane) & (NR - 1)];
    const __amdgpu_buffer_rsrc_t rme = mk_rsrc(myws, WS_USED);
    const __amdgpu_buffer_rsrc_t rx = mk_rsrc(p.x, (unsigned)M_ * KS * 2);
    const __amdgpu_buffer_rsrc_t rw = mk_rsrc(p.w, (unsigned)KS * N_ * 2);
    u32x4 buf[8];
    if (push_q != 0) load_w(buf, rw, 0, (unsigned)(((rank + push_q) & (NR - 1)) * NB + push_u * 64), lane);
    const unsigned call = __builtin_amdgcn_readfirstlane(cprev) + 1;
    if (lane == 0) __hip_atomic_store(ctr, call, __ATOMIC_RELAXED, __HIP_MEMORY_SCOPE_AGENT);
    if (push_q != 0) {
        const __amdgpu_buffer_rsrc_t rd = mk_rsrc(p.ws[(rank + push_q) & (NR - 1)], WS_USED);
        wait_until(tstart, TT_START + (unsigned)push_u * 144 / 10);
        store_w(buf, rd, WS_OFF + (unsigned)(rank * NKC * NCB + (push_u >> 2)) * TILE_B + (push_u & 3) * 4096, lane);
        asm volatile("s_waitcnt vmcnt(0)" ::: "memory");
        __builtin_amdgcn_raw_buffer_store_b32((call & 0xffffu) * 0x10001u, rd, lane == 0 ? 0u : 0x80000000u,
                                              FW_OFF + (unsigned)(((push_u >> 2) * 64 + rank) * 4 + (push_u & 3)) * 4, AUX_SYS);
    }
    for (int e = eidx; e < NKC * 320; e += NEXP) {
        const int kc = e / 320, r = e - kc * 320;
        unsigned foff; int flane;
        if (kc > 0) wait_until(tstart, (unsigned)((kc - (WEARLY && kc == 7 && r < 256 ? 1 : 0)) * 96 * PACE_IN / 100 - EXP_LEAD + r * 2));
        else if (r >= 256) wait_until(tstart, 150u);
        if (r < 256) {
            const int j = r >> 3, q = r & 7, cb = j >> 2, wc = j & 3;
            if (kc == 0 && j < NPUSH0 && q != 0) continue;
            load_w(buf, rw, kc, (unsigned)(((rank + q) & (NR - 1)) * NB + j * 64), lane);
            store_w(buf, rme, (q == 0 ? WS_OFF + (unsigned)(rank * NKC * NCB) * TILE_B : EW_OFF + (unsigned)(q * NKC * NCB) * TILE_B)
                              + (unsigned)(kc * NCB + cb) * TILE_B + wc * 4096, lane);
            flane = q;
            foff = q == 0 ? FW_OFF + (unsigned)((cb * 64 + kc * 8 + rank) * 4 + wc) * 4 : EF_OFF + (unsigned)(q * NUNIT + list_pos(kc, j)) * 4;
        } else {
            const int rb = (r - 256) >> 2, uq = r & 3;
            const unsigned xoff = (unsigned)((lane & 15) * KS + (lane >> 4) * 8) * 2, sb = (unsigned)((rb * 256 + uq * 64) * KS + kc * 64) * 2;
            const unsigned dbase = XS_OFF + (unsigned)((rank * NKC + kc) * NRB + rb) * TILE_B + uq * 8192;
#pragma unroll
            for (int g = 0; g < 8; ++g)
                buf[g] = __builtin_amdgcn_raw_buffer_load_b128(rx, xoff, sb + (unsigned)((g >> 1) * 16 * KS + (g & 1) * 32) * 2, 0);
#pragma unroll
            for (int g = 0; g < 8; ++g)
                ST128(buf[g], rme, lane * 16, dbase + g * 1024, AUX_SYS);
            flane = -1;
            foff = lane == 0 ? FX_OFF + (unsigned)((rb * 64 + kc * 8 + rank) * 4 + uq) * 4
                             : EF_OFF + (unsigned)(lane * NUNIT + list_pos(kc, 32) + ((rb - 2 * lane) & 15) * 4 + uq) * 4;
        }
        asm volatile("s_waitcnt vmcnt(0)" ::: "memory");
        if ((flane < 0 ? lane < NR : lane == flane) && (lane == 0 || kc == 0 || !GROUP_FLAGS))
            __hip_atomic_store((unsigned*)(peer + foff), lane == 0 ? (call & 0xffffu) * 0x10001u : call, __ATOMIC_RELAXED, __HIP_MEMORY_SCOPE_SYSTEM);
        if (GROUP_FLAGS && kc > 0) {
            unsigned n = 0;
            if (lane == 0) n = __hip_atomic_fetch_add((unsigned*)(myws + EC_OFF + kc * 128), 1u, __ATOMIC_RELAXED, __HIP_MEMORY_SCOPE_AGENT);
            if (__builtin_amdgcn_readfirstlane(n) + 1 == call * 320u && lane > 0 && lane < NR)
                __hip_atomic_store((unsigned*)(peer + EF_OFF) + lane * NUNIT + kc * 96, call, __ATOMIC_RELAXED, __HIP_MEMORY_SCOPE_SYSTEM);
        }
    }
    STAMPMAX(3);
}

__device__ __forceinline__ void link_role(const Params& p, int q, int widx, int wid, int lane, unsigned* ctr) {
    unsigned long long tstart = now();
    const int rank = p.rank;
    const int src = (rank - q) & (NR - 1);
    unsigned char* myws = p.myws;
    unsigned* tmo = (unsigned*)(myws + TM_OFF);
    const unsigned cprev = __hip_atomic_load(ctr, __ATOMIC_RELAXED, __HIP_MEMORY_SCOPE_AGENT);
    const __amdgpu_buffer_rsrc_t rme = mk_rsrc(myws, WS_USED);
    const unsigned l0 = lane == 0 ? 0u : 0x80000000u;
    const unsigned soff = lane * 16;
    const unsigned call = __builtin_amdgcn_readfirstlane(cprev) + 1;
    if (lane == 0) __hip_atomic_store(ctr, call, __ATOMIC_RELAXED, __HIP_MEMORY_SCOPE_AGENT);
    u32x4 bufa[4], bufb[4];
    const unsigned char* tk = uni(myws + TK_OFF + q * 128);
    unsigned tick;
#define DRAW() asm volatile("s_mov_b64 exec, 1\n\tglobal_atomic_add %0, %1, %2, %3 sc0\n\ts_mov_b64 exec, -1" : "=&v"(tick) : "v"(0u), "v"(1u), "s"(tk) : "memory")
#define DUE(T0, DL) ((int)((DL) + (unsigned)(T0) - (unsigned)now()) <= 0)

    if (widx < NW) {
        const __amdgpu_buffer_rsrc_t rs = mk_rsrc(p.ws[src], WS_USED);
#define WAIT_EXPORTED(FO, COND) do { const unsigned long long t0_ = now(); \
            for (;;) { \
                const unsigned tmv = ld_tmo(tmo); \
                const u32x2 v = __builtin_amdgcn_raw_buffer_load_b64(rme, (FO), EF_OFF + (unsigned)(q * NUNIT) * 4, AUX_SYS); \
                if (__all(COND)) break; \
                if (give_up(tmo, tmv, call, t0_)) break; \
                __builtin_amdgcn_s_sleep(2); \
            } } while (0)
#define IN_ADDR(H, SB, DB, FO) do { const int hf_ = (H) & 1; int kc_, j_; list_unit((H) >> 1, kc_, j_); \
            if (j_ < 32) { \
                const unsigned o_ = (unsigned)(kc_ * NCB + (j_ >> 2)) * TILE_B + (j_ & 3) * 4096 + hf_ * HALF_B; \
                SB = EW_OFF + (unsigned)(q * NKC * NCB) * TILE_B + o_; DB = WS_OFF + (unsigned)(src * NKC * NCB) * TILE_B + o_; \
                FO = FW_OFF + (unsigned)(((j_ >> 2) * 64 + kc_ * 8 + src) * 8 + (j_ & 3) * 2 + hf_) * 2; \
            } else { \
                const int rb_ = (((j_ - 32) >> 2) + 2 * q) & 15; \
                SB = DB = XS_OFF + (unsigned)((src * NKC + kc_) * NRB + rb_) * TILE_B + (j_ & 3) * 8192 + hf_ * 4096; \
                FO = FX_OFF + (unsigned)((rb_ * 64 + kc_ * 8 + src) * 8 + (j_ & 3) * 2 + hf_) * 2; \
            } } while (0)
#define IN_LOAD(BUF, SB) do { _Pragma("unroll") for (int g = 0; g < 4; ++g) \
            BUF[g] = __builtin_amdgcn_raw_buffer_load_b128(rs, soff, (SB) + g * 1024, AUX_SYS); } while (0)
#define FLAG(FO) __builtin_amdgcn_raw_buffer_store_b16((unsigned short)call, rme, l0, (FO), AUX_DEV)
        DRAW();
        WAIT_EXPORTED((unsigned)(NPUSH0 + (lane < (32 - NPUSH0) / 2 ? lane * 2 : 0)) * 4, v.x == call && v.y == call);
        {
            const int late = (int)((unsigned)(now() - tstart) - (TT_START + (unsigned)NPUSH0 * PACE_IN / 100));
            if (late > 0) tstart += (unsigned)late;
        }
        const unsigned tb = (call - 1) * (unsigned)(2 * (NUNIT - NPUSH0)) - (2 * NPUSH0 + NW);
        int u_ok = 32;
        unsigned fprev = 0;
        unsigned sb, db, fo, dbn, fon;
        IN_ADDR(2 * NPUSH0 + widx, sb, db, fo);
        wait_until(tstart, TT_START + (unsigned)(2 * NPUSH0 + widx) * PACE_IN / 200);
        IN_LOAD(bufa, sb);
#define IN_STEP(CUR, NXT) { \
              \
            asm volatile("s_waitcnt vmcnt(0)" : "+v"(tick), "+v"(CUR[0]), "+v"(CUR[1]), "+v"(CUR[2]), "+v"(CUR[3]) :: "memory"); \
            const int hn = (int)(__builtin_amdgcn_readfirstlane(tick) - tb); \
            const bool valid = hn < 2 * NUNIT; \
            while ((hn >> 1) >= u_ok && u_ok < NUNIT) {                      \
                const int n2 = (u_ok == 32 ? 64 : 96) / 2; \
                if (GROUP_FLAGS && u_ok > 32) WAIT_EXPORTED((unsigned)u_ok * 4, v.x == call); \
                else WAIT_EXPORTED((unsigned)(u_ok + (lane < n2 ? lane * 2 : 0)) * 4, v.x == call && v.y == call); \
                u_ok += 2 * n2; \
            } \
            IN_ADDR(hn, sb, dbn, fon); \
            const unsigned dl = TT_START + (unsigned)hn * PACE_IN / 200; \
            const bool early = valid && DUE(tstart, dl); \
            if (early) IN_LOAD(NXT, sb); \
            asm volatile("" ::: "memory"); \
            fprev = __builtin_amdgcn_readfirstlane(fprev); db = __builtin_amdgcn_readfirstlane(db); fo = __builtin_amdgcn_readfirstlane(fo); \
            if (fprev) FLAG(fprev); \
            _Pragma("unroll") for (int g = 0; g < 4; ++g) ST128(CUR[g], rme, soff, db + g * 1024, AUX_DEV); \
            fprev = fo; \
            if (valid) DRAW(); \
            if (hn >= 2 * NUNIT - FLAG_NOW && !early) {                      \
                asm volatile("s_waitcnt vmcnt(0)" : "+v"(tick) :: "memory"); \
                FLAG(fprev); fprev = 0; \
            } \
            if (!valid) break; \
            if (!early) { wait_until(tstart, dl); IN_LOAD(NXT, sb); } \
            db = dbn; fo = fon; }
        for (;;) {
            IN_STEP(bufa, bufb)
            IN_STEP(bufb, bufa)
        }
#undef WAIT_EXPORTED
    }
    STAMPMAX(1);

    if (widx < NWO) {
        const __amdgpu_buffer_rsrc_t rd = mk_rsrc(p.myout, OUT_BYTES);
        bool bad;
#define WAIT_TILES(STAGE, MASK) do { const unsigned long long t0_ = now(); \
            for (;;) { \
                const unsigned tmv = ld_tmo(tmo); \
                const u32x2 v = __builtin_amdgcn_raw_buffer_load_b64(rme, lane * 8, OF_OFF + ((STAGE) * NR + q) * 512, OFP_AUX); \
                bad = __builtin_amdgcn_readfirstlane(tmv) == call; \
                seen = __ballot(v.x == call && v.y == call); \
                if ((seen & (MASK)) == (MASK)) break; \
                if (give_up(tmo, tmv, call, t0_)) { bad = true; seen = ~0ull; break; } \
                __builtin_amdgcn_s_sleep(OFP_SLEEP); \
            } \
              \
            bm = __builtin_amdgcn_readfirstlane(bad ? 0x7f7f7f7f : 0); \
            rs = mk_rsrc(p.out[src], bm ? 0u : OUT_BYTES); } while (0)
#define RB_MASK(K) ((GROUP_OF1 ? 1ull : 0xfull) << (((((2 * NPUSH1 + (K)) >> 3) & 3) * 4 + (((2 * NPUSH1 + (K)) >> 5) & 3)) * 4))
#define OUT_ROW(H) (((((H) >> 3) & 3) * 4 + (((H) >> 5) & 3)) * 256 + ((H) >> 10) * 128 + (((H) >> 7) & 1) * 64 + (((H) >> 8) & 3) * 16 + ((H) & 7) * 2)
#define OUT_OFF(K) ((unsigned)OUT_ROW(2 * NPUSH1 + (K)) * N_ + src * NB)
#define OUT_LOAD(BUF, OB) do { _Pragma("unroll") for (int g = 0; g < 4; ++g) \
            BUF[g] = __builtin_amdgcn_raw_buffer_load_b128(rs, soff, (OB) + (g >> 1) * N_ + (g & 1) * 1024, AUX_SYS); } while (0)
        unsigned bm;
        unsigned long long seen;
        __amdgpu_buffer_rsrc_t rs;
        tk += NR * 128;
        DRAW();
        WAIT_TILES(0, RB_MASK(widx));
        unsigned long long tf = __hip_atomic_load((unsigned long long*)(myws + TS_OFF), __ATOMIC_RELAXED, __HIP_MEMORY_SCOPE_AGENT);
        tf = ((unsigned long long)(unsigned)__builtin_amdgcn_readfirstlane((unsigned)(tf >> 32)) << 32) | (unsigned)__builtin_amdgcn_readfirstlane((unsigned)tf);
        tf = (OUT_BASE > 0 && tf > tstart) ? tf + OUT_BASE : now();
        STAMPMAX(2);
        const unsigned tb = (call - 1) * (unsigned)(2 * (NOUT - NPUSH1)) - NWO;
        bool stage2 = false;
        unsigned ob = OUT_OFF(widx);
#define OUT_SLOT(K) ((unsigned)(K) * PACE_OUT / 200 + (unsigned)((K) < OUT_K0 ? (K) : OUT_K0) * OUT_EXTRA / 200)
        wait_until(tf, OUT_SLOT(widx));
        OUT_LOAD(bufa, ob);
#define OUT_STEP(CUR, NXT) { \
            asm volatile("s_waitcnt vmcnt(0)" : "+v"(tick), "+v"(CUR[0]), "+v"(CUR[1]), "+v"(CUR[2]), "+v"(CUR[3]) :: "memory"); \
            const int kn = (int)(__builtin_amdgcn_readfirstlane(tick) - tb); \
            const bool valid = kn < 2 * (NOUT - NPUSH1); \
            const unsigned bmc = bm; \
            if (!stage2) { \
                if (NPUSH1 + (kn >> 1) >= STAGE2_AT) { WAIT_TILES(1, 1ull); stage2 = true; } \
                else if ((seen & RB_MASK(kn)) != RB_MASK(kn)) WAIT_TILES(0, RB_MASK(kn)); \
            } \
            const unsigned obn = OUT_OFF(kn), dl = OUT_SLOT(kn); \
            const bool early = valid && DUE(tf, dl); \
            if (early) OUT_LOAD(NXT, obn); \
            asm volatile("" ::: "memory"); \
            ob = __builtin_amdgcn_readfirstlane(ob); \
            _Pragma("unroll") for (int g = 0; g < 4; ++g) ST128(CUR[g] | bmc, rd, soff, ob + (g >> 1) * N_ + (g & 1) * 1024, AUX_DEV); \
            if (!valid) break; \
            DRAW(); \
            if (!early) { wait_until(tf, dl); OUT_LOAD(NXT, obn); } \
            ob = obn; }
        for (;;) {
            OUT_STEP(bufa, bufb)
            OUT_STEP(bufb, bufa)
        }
        asm volatile("s_waitcnt vmcnt(0)" ::: "memory");
    }
    STAMPMAX(6);
}

#define WAIT_V(n) asm volatile("s_waitcnt vmcnt(" #n ")" ::: "memory")
#define WAIT_L(n) asm volatile("s_waitcnt lgkmcnt(" #n ")" ::: "memory")
#define BAR __builtin_amdgcn_s_barrier()
#define SCHED __builtin_amdgcn_sched_barrier(0)

__device__ __forceinline__ void gemm_role(const Params& p, LAS unsigned char* lds, int rb, int cb, int wid, int lane, unsigned call) {
    const int rank = p.rank;
    unsigned char* myws = p.myws;
    unsigned* tmo = (unsigned*)(myws + TM_OFF);
    const __amdgpu_buffer_rsrc_t rws = mk_rsrc(myws, WS_USED);
    const int wr = wid >> 2, wc = wid & 3;
    const unsigned tid16 = (unsigned)(wid * 64 + lane) * 16;
    volatile LAS int* sync_word = (volatile LAS int*)(lds + LDS_STAGE);
    unsigned char* peer_ws = p.ws[lane & 7];

    int tiv;
    {
        int s = rank, kc = lane;
        if (lane >= 8) {
            const int j = lane - 8, i = j % 7, q0 = rb >> 1;
            kc = j / 7;
            const int q = (i < q0) ? q0 - i : 7 - (i - q0);
            s = (rank - q) & (NR - 1);
        }
        tiv = s * 8 + kc;
    }
    const unsigned xtile0 = XS_OFF + rb * TILE_B, wtile0 = WS_OFF + cb * TILE_B;
    unsigned long long avail = 0, taken = 0;

    const unsigned fi = (unsigned)((tiv & 7) * 8 + (tiv >> 3)) * 16;
    const unsigned fxo = FX_OFF + rb * 1024 + fi, fwo = FW_OFF + cb * 1024 + fi;
    const unsigned cc = (call & 0xffffu) * 0x10001u;
#define FLAGS_OK(fx, fw) ((((fx.x ^ cc) | (fx.y ^ cc) | (fx.z ^ cc) | (fx.w ^ cc)) | ((fw.x ^ cc) | (fw.y ^ cc) | (fw.z ^ cc) | (fw.w ^ cc))) == 0)
#define PICK(T, SLOT, FAST) do { \
    if (avail == 0) { \
        if (wid == 0) { \
            unsigned long long r; \
            const unsigned long long t0 = now(); \
            for (;;) { \
                const unsigned tmv = ld_tmo(tmo); \
                const u32x4 fx = __builtin_amdgcn_raw_buffer_load_b128(rws, fxo, 0, POLL_AUX); \
                const u32x4 fw = __builtin_amdgcn_raw_buffer_load_b128(rws, fwo, 0, POLL_AUX); \
                r = __ballot(FLAGS_OK(fx, fw)) & ~taken; \
                if (r) break; \
                if (give_up(tmo, tmv, call, t0)) { r = ~taken; break; } \
                if (FAST) __builtin_amdgcn_s_sleep(2); else __builtin_amdgcn_s_sleep(PICK_SLEEP); \
            } \
            ARR(r); \
            if (lane == 0) { sync_word[2 * (SLOT)] = (int)(unsigned)r; sync_word[2 * (SLOT) + 1] = (int)(unsigned)(r >> 32); } \
            asm volatile("s_waitcnt lgkmcnt(0)" ::: "memory"); \
        } \
        asm volatile("" ::: "memory"); BAR; asm volatile("" ::: "memory"); \
        const unsigned lo_ = __builtin_amdgcn_readfirstlane(sync_word[2 * (SLOT)]), hi_ = __builtin_amdgcn_readfirstlane(sync_word[2 * (SLOT) + 1]); \
        avail = ((unsigned long long)hi_ << 32) | lo_; \
        asm volatile("s_waitcnt lgkmcnt(0)" ::: "memory"); \
    } \
    { const int i_ = __builtin_ctzll(avail); const unsigned long long b_ = 1ull << i_; avail &= ~b_; taken |= b_; T = (unsigned)__builtin_amdgcn_readlane(tiv, i_); } \
    } while (0)

#define SA(b, h) (((b) * 2 + (h)) * HALF_B)
#define SB(b, h) ((4 + (b) * 2 + (h)) * HALF_B)
#define XOFF(T) (xtile0 + (T) * (NRB * TILE_B))
#define WOFF(T) (wtile0 + (T) * (NCB * TILE_B))
#define STAGE(P, SRC) do { \
        __builtin_amdgcn_raw_ptr_buffer_load_lds(rws, (LAS void*)(lds + (P) + wid * 1024), 16, tid16, (SRC), 0, AUX_DEV); \
        __builtin_amdgcn_raw_ptr_buffer_load_lds(rws, (LAS void*)(lds + (P) + 8192 + wid * 1024), 16, tid16, (SRC) + 8192, 0, AUX_DEV); } while (0)
#define LDA(dst, b, h) _Pragma("unroll") for (int m = 0; m < 4; ++m) _Pragma("unroll") for (int k = 0; k < 2; ++k) \
        dst[m][k] = *(const LAS bf16x8*)(abase + SA(b, h) + (m * 2 + k) * 1024)
#define LDB(dst, b, h) _Pragma("unroll") for (int n = 0; n < 2; ++n) _Pragma("unroll") for (int k = 0; k < 2; ++k) \
        dst[n][k] = *(const LAS bf16x8*)(bbase + SB(b, h) - 4 * HALF_B + (n * 2 + k) * 1024)
#define MMA(ai, bj, At, Bt) do { __builtin_amdgcn_s_setprio(1); \
        _Pragma("unroll") for (int m = 0; m < 4; ++m) _Pragma("unroll") for (int n = 0; n < 2; ++n) _Pragma("unroll") for (int k = 0; k < 2; ++k) \
            acc[ai][bj][m][n] = __builtin_amdgcn_mfma_f32_16x16x32_bf16(Bt[n][k], At[m][k], acc[ai][bj][m][n], 0, 0, 0); \
        __builtin_amdgcn_s_setprio(0); } while (0)

#define HALF_MAX(ai, A, B) do { _Pragma("unroll") for (int bj = 0; bj < 2; ++bj) _Pragma("unroll") for (int m = 0; m < 4; ++m) { \
        const f32x4 u_ = acc[ai][bj][m][0], v_ = acc[ai][bj][m][1]; \
        A = fmaxf(fmaxf(A, fabsf(u_[0])), fabsf(u_[1])); B = fmaxf(fmaxf(B, fabsf(v_[0])), fabsf(v_[1])); \
        A = fmaxf(fmaxf(A, fabsf(u_[2])), fabsf(u_[3])); B = fmaxf(fmaxf(B, fabsf(v_[2])), fabsf(v_[3])); } } while (0)
    float am0 = 0.f, am1 = 0.f;
    const LAS unsigned char* abase = lds + wr * 8192 + lane * 16;
    const LAS unsigned char* bbase = lds + 4 * HALF_B + wc * 4096 + lane * 16;
    f32x4 acc[2][2][4][2] = {};
    bf16x8 At[4][2], B0[2][2], B1[2][2];

    unsigned ta, tb;
    PICK(ta, 0, false);
    STAMP(1);
    {
        const unsigned x0 = XOFF(ta), w0 = WOFF(ta);
        STAGE(SB(0, 0), w0); STAGE(SA(0, 0), x0);
        STAGE(SB(0, 1), w0 + HALF_B); STAGE(SA(0, 1), x0 + HALF_B);
        PICK(tb, 1, false);
        const unsigned x1 = XOFF(tb), w1 = WOFF(tb);
        if (wr == 1) BAR;
        WAIT_V(4); BAR;
        STAGE(SB(1, 0), w1); STAGE(SA(1, 0), x1); STAGE(SB(1, 1), w1 + HALF_B);
        WAIT_V(6); BAR;
    }
#define WV_A WAIT_V(10)
#define WV_B WAIT_V(10)
    for (int t = 0; t < NT - 2; t += 2) {
        STAMP(8 + (t >> 1));
        const bool fast = t >= FAST_FROM;
        LDB(B0, 0, 0); SCHED; LDA(At, 0, 0); STAGE(SA(1, 1), XOFF(tb) + HALF_B);
        WV_A; WAIT_L(8); BAR; WAIT_L(0); MMA(0, 0, At, B0); BAR; SCHED;
        PICK(ta, 2, fast);
        const unsigned x2 = XOFF(ta), w2 = WOFF(ta);
        LDB(B1, 0, 1); STAGE(SB(0, 0), w2);
        WV_A; BAR; WAIT_L(0); MMA(0, 1, At, B1); BAR;
        LDA(At, 0, 1); STAGE(SA(0, 0), x2);
        WV_A; BAR; WAIT_L(0); MMA(1, 0, At, B0); BAR; SCHED;
        STAGE(SB(0, 1), w2 + HALF_B);
        WV_B; BAR; MMA(1, 1, At, B1); BAR;
        LDB(B0, 1, 0); SCHED; LDA(At, 1, 0); STAGE(SA(0, 1), x2 + HALF_B);
        WV_A; WAIT_L(8); BAR; WAIT_L(0); MMA(0, 0, At, B0); BAR; SCHED;
        PICK(tb, 3, fast);
        const unsigned x3 = XOFF(tb), w3 = WOFF(tb);
        LDB(B1, 1, 1); STAGE(SB(1, 0), w3);
        WV_A; BAR; WAIT_L(0); MMA(0, 1, At, B1); BAR;
        LDA(At, 1, 1); STAGE(SA(1, 0), x3);
        WV_A; BAR; WAIT_L(0); MMA(1, 0, At, B0); BAR; SCHED;
        STAGE(SB(1, 1), w3 + HALF_B);
        WV_B; BAR; MMA(1, 1, At, B1); BAR;
    }
    STAMP(2);
    {
        const unsigned xl = XOFF(tb);
        LDB(B0, 0, 0); WAIT_V(0); LDA(At, 0, 0); STAGE(SA(1, 1), xl + HALF_B);
        BAR; WAIT_L(0); MMA(0, 0, At, B0); BAR;
        LDB(B1, 0, 1); BAR; WAIT_L(0); MMA(0, 1, At, B1); BAR;
        LDA(At, 0, 1); WAIT_V(4); BAR; WAIT_L(0); MMA(1, 0, At, B0); MMA(1, 1, At, B1); BAR;
    }
    {
        LDB(B0, 1, 0); LDA(At, 1, 0); WAIT_V(2); BAR; WAIT_L(0); MMA(0, 0, At, B0); BAR;
        LDB(B1, 1, 1); WAIT_V(0); BAR; WAIT_L(0); MMA(0, 1, At, B1); BAR;
        LDA(At, 1, 1); BAR; WAIT_L(0); MMA(1, 0, At, B0); MMA(1, 1, At, B1);
    }
    volatile LAS float* red = (volatile LAS float*)(lds + LDS_STAGE + 128);
    {
        float am2 = 0.f, am3 = 0.f;
        HALF_MAX(0, am0, am1);
        HALF_MAX(1, am2, am3);
        const float am = wave_max(fmaxf(fmaxf(am0, am1), fmaxf(am2, am3)));
        if (lane == 0) red[wid] = am;
    }
    WAIT_L(0); BAR;
    if (wr == 0) BAR;
    asm volatile("" ::: "memory");
    STAMP(3);

    const int gid = rb * NCB + cb;
    if (wid == 0) {
        const float a = wave_max(red[lane & 7]);
        if (lane < NR)
            __hip_atomic_store((unsigned long long*)(peer_ws + AM_OFF) + rank * 128 + gid,
                               ((unsigned long long)call << 32) | __float_as_uint(a), __ATOMIC_RELAXED, __HIP_MEMORY_SCOPE_SYSTEM);
    }
    unsigned char* po[NR - 1];
#pragma unroll
    for (int j = 0; j < NR - 1; ++j) { po[j] = uni(p.out[(rank + 1 + (gid + wc + j) % 7) & (NR - 1)]); asm volatile("" : "+s"(po[j])); }
    {
        const unsigned long long t0 = now();
        float g; bool bd;
        for (;;) {
            const unsigned tmv = ld_tmo(tmo);
            const u32x4 v = __builtin_amdgcn_raw_buffer_load_b128(rws, AM_OFF + tid16, 0, AMP_AUX);
            g = fmaxf(__uint_as_float(v.x), __uint_as_float(v.z));
            bd = tmv == call;
            if (__all(v.y == call && v.w == call)) break;
            if (give_up(tmo, tmv, call, t0)) { bd = true; break; }
            __builtin_amdgcn_s_sleep(AMP_SLEEP);
        }
        g = wave_max(g);
        if (lane == 0) red[8 + wid] = bd ? __builtin_nanf("") : g;
    }
    WAIT_L(0); BAR; asm volatile("" ::: "memory");
    const unsigned long long tout = now();
    if (gid == 0 && wid == 7 && lane == 0) __hip_atomic_store((unsigned long long*)(myws + TS_OFF), tout, __ATOMIC_RELAXED, __HIP_MEMORY_SCOPE_AGENT);
    float gmax = 0.f; bool bad = false;
#pragma unroll
    for (int i = 0; i < 8; ++i) { const float r = red[8 + i]; bad |= r != r; gmax = fmaxf(gmax, r); }
    STAMP(4);

    const float scale = gmax / 448.0f;
    const float inv = 1.0f / scale;
#define QUANT(O, ai, m) do { \
        const f32x4 a_ = acc[ai][0][m][0] * inv, b_ = acc[ai][0][m][1] * inv, c_ = acc[ai][1][m][0] * inv, e_ = acc[ai][1][m][1] * inv; \
        _Pragma("unroll") for (int r = 0; r < 4; ++r) { \
            int pk = __builtin_amdgcn_cvt_pk_fp8_f32(a_[r], b_[r], 0, false); \
            pk = __builtin_amdgcn_cvt_pk_fp8_f32(c_[r], e_[r], pk, true); \
            O[r] = bad ? 0x7f7f7f7fu : (unsigned)pk; \
        } } while (0)
    const unsigned ooff = (unsigned)(rb * 256 + wr * 64 + (lane & 15)) * N_ + rank * NB + cb * 256 + wc * 64 + (lane >> 4) * 16;
    const int rbi = (rb & 3) * 4 + (rb >> 2);
    const bool pusher = wr == 0 && rbi * 4 < NPUSH1;
    const unsigned dl = (unsigned)((rbi * 8 + cb) * 4 + wc) * PACE_FINE / 1000;
    const __amdgpu_buffer_rsrc_t rl = mk_rsrc(p.myout, OUT_BYTES);
    u32x4 q0;
    QUANT(q0, 0, 0);
    ST128(q0, rl, ooff, 0, AUX_SYS);
    asm volatile("" ::: "memory");
    const bool early = pusher && (unsigned)(now() - tout) >= dl;
#define PUSH_FIRST() do { _Pragma("unroll") for (int j = 0; j < NR - 1; ++j) \
        ST128(q0, mk_rsrc(po[j], OUT_BYTES), ooff, 0, AUX_SYS); } while (0)
    if (early) { PUSH_FIRST(); WAIT_V(7); } else { WAIT_V(0); }
    BAR; asm volatile("" ::: "memory");
    const int lq = ((lane & 7) - rank) & (NR - 1);
#define ANNOUNCE(CTR, N, WORD) do { unsigned n_ = 0; \
        if (lane == 0) n_ = __hip_atomic_fetch_add((unsigned*)(myws + OC_OFF + (CTR) * 128), 1u, __ATOMIC_RELAXED, __HIP_MEMORY_SCOPE_AGENT); \
        if (__builtin_amdgcn_readfirstlane(n_) + 1 == call * (N) && lane < NR && lq != 0) \
            __hip_atomic_store((unsigned long long*)(peer_ws + OF_OFF) + lq * 64 + (WORD), (unsigned long long)call * 0x100000001ull, __ATOMIC_RELAXED, __HIP_MEMORY_SCOPE_SYSTEM); } while (0)
    if (wid == 7) {
        if (GROUP_OF1) ANNOUNCE(rb, 8u, rb * 4);
        else if (lane < NR && lq != 0)
            __hip_atomic_store((unsigned*)(peer_ws + OF_OFF) + lq * 128 + gid, call, __ATOMIC_RELAXED, __HIP_MEMORY_SCOPE_SYSTEM);
    }
    STAMP(7);
    if (pusher && !early) { wait_until(tout, dl); PUSH_FIRST(); }
    wait_until(tout, (unsigned)(STAGE2_DELAY + wid * STAGE2_STEP));
#pragma unroll
    for (int i = 1; i < 8; ++i) {
        u32x4 qi;
        QUANT(qi, i >> 2, i & 3);
        ST128(qi, rl, ooff, (unsigned)((i >> 2) * 128 + (i & 3) * 16) * N_, AUX_SYS);
    }
    WAIT_V(0);
    if (pusher && lane < NR)
        __hip_atomic_store((unsigned*)(peer_ws + DN_OFF + (unsigned)(gid * 8 + wid) * 32) + lq, call, __ATOMIC_RELAXED, __HIP_MEMORY_SCOPE_SYSTEM);
    BAR; asm volatile("" ::: "memory");
    if (wid == 7) ANNOUNCE(16, 128u, NR * 64);
    STAMP(5);
    if (pusher) {
        const unsigned dn = DN_OFF + (unsigned)(gid * 8 + wid) * 32;
        const unsigned long long t0 = now();
        for (;;) {
            const unsigned tmv = ld_tmo(tmo);
            const unsigned v = __builtin_amdgcn_raw_buffer_load_b32(rws, (lane & 7) * 4, dn, AUX_SYS);
            if (__all(v == call)) break;
            if (give_up(tmo, tmv, call, t0)) { if (lane == 0) p.myout[ooff] = 0x7f; break; }
            __builtin_amdgcn_s_sleep(2);
        }
    }
    STAMPMAX(6);
}

__global__ void __launch_bounds__(512, 2) fused_kernel(const Params p) {
    extern __shared__ __attribute__((aligned(16))) unsigned char smem[];
    const int b = blockIdx.x, xcd = b & 7, slot = b >> 3;
    const int wid = __builtin_amdgcn_readfirstlane(threadIdx.x >> 6), lane = threadIdx.x & 63;
    unsigned char* myws = p.myws;
    unsigned* ctr = (unsigned*)(myws + CT_OFF) + b * 8 + wid;
    STAMP(0);
    if (slot < 16) {
        const int q = slot & 7, wgi = xcd * 2 + (slot >> 3);
        if (q != 0 && wid < LINK_WAVES) { if (wid * 16 + wgi < (NW > NWO ? NW : NWO)) link_role(p, q, wid * 16 + wgi, wid, lane, ctr); }
        else export_role(p, q == 0 ? wid * 16 + wgi : 128 + (wid - LINK_WAVES) * 112 + (q - 1) * 16 + wgi,
                         (wid == 7 && wgi < NPUSH0) ? q : 0, wgi, wid, lane, ctr);
    } else {
        const int g = slot - 16;
        const unsigned call = __builtin_amdgcn_readfirstlane(__hip_atomic_load(ctr, __ATOMIC_RELAXED, __HIP_MEMORY_SCOPE_AGENT)) + 1;
        if (lane == 0) __hip_atomic_store(ctr, call, __ATOMIC_RELAXED, __HIP_MEMORY_SCOPE_AGENT);
        if (XCDMAP) gemm_role(p, (LAS unsigned char*)smem, xcd * 2 + (g >> 3), g & 7, wid, lane, call);
        else gemm_role(p, (LAS unsigned char*)smem, (xcd >> 1) * 4 + (g >> 2), (xcd & 1) * 4 + (g & 3), wid, lane, call);
    }
}

}

extern "C" void kernel_launch(int rank, int nranks, ncclComm_t comm, hipStream_t stream,
                              void* const* d_in, const int* in_sizes, int n_in,
                              void* d_out, int out_size,
                              const int* in_dtypes, int out_dtype,
                              void* const* d_out_all,
                              void* const* d_ws_all, size_t ws_bytes) {
    if (nranks != NR || (unsigned)rank >= (unsigned)NR || n_in < 2 || ws_bytes < WS_USED ||
        in_sizes[0] != M_ * KS || in_sizes[1] != KS * N_ || out_size != M_ * N_) {
        fprintf(stderr, "kernel_launch: unexpected problem shape\n");
        return;
    }
    static bool attr_set = false;
    if (!attr_set) {
        HIP_CHECK(hipFuncSetAttribute((const void*)fused_kernel, hipFuncAttributeMaxDynamicSharedMemorySize, LDS_BYTES));
        attr_set = true;
    }
    Params p;
    p.x = (const unsigned short*)d_in[0];
    p.w = (const unsigned short*)d_in[1];
    for (int i = 0; i < NR; ++i) {
        p.out[i] = (unsigned char*)d_out_all[i];
        p.ws[i] = (unsigned char*)d_ws_all[i];
    }
    p.myws = p.ws[rank];
    p.myout = (unsigned char*)d_out;
    p.rank = rank;
    hipLaunchKernelGGL(fused_kernel, dim3(256), dim3(512), LDS_BYTES, stream, p);
    HIP_CHECK(hipGetLastError());
}
```

```cpp
#include <hip/hip_runtime.h>
#include <rccl/rccl.h>
#include <cstdio>
#include <cstdint>

#define HIP_CHECK(expr) do { hipError_t _e = (expr); if (_e != hipSuccess) { \
    fprintf(stderr, "HIP error %s at %s:%d\n", hipGetErrorString(_e), __FILE__, __LINE__); \
    return; } } while (0)

namespace {

#define LAS __attribute__((address_space(3)))
typedef short bf16x8 __attribute__((ext_vector_type(8)));
typedef float f32x4 __attribute__((ext_vector_type(4)));
typedef unsigned u32x4 __attribute__((ext_vector_type(4)));
typedef unsigned u32x2 __attribute__((ext_vector_type(2)));

constexpr int NR = 8;
constexpr int M_ = 4096, K_ = 4096, N_ = 16384;
constexpr int KS = K_ / NR;
constexpr int NB = N_ / NR;
constexpr int NRB = 16, NCB = 8;
constexpr int NKC = 8;
constexpr int NT = 64;
constexpr unsigned TILE_B = 32768, HALF_B = 16384;
constexpr int NUNIT = NKC * 96;
constexpr int NOUT = 1024;

constexpr unsigned XS_OFF = 0;
constexpr unsigned WS_OFF = XS_OFF + 64u * NRB * TILE_B;
constexpr unsigned EW_OFF = WS_OFF + 64u * NCB * TILE_B;
constexpr unsigned FX_OFF = EW_OFF + 64u * NCB * TILE_B;
constexpr unsigned FW_OFF = FX_OFF + NRB * 64 * 16;
constexpr unsigned EF_OFF = FW_OFF + NCB * 64 * 16;
constexpr unsigned OF_OFF = EF_OFF + NR * NUNIT * 4;
constexpr unsigned AM_OFF = OF_OFF + 2 * NR * 512;
constexpr unsigned DN_OFF = AM_OFF + NR * 128 * 8;
constexpr unsigned CT_OFF = DN_OFF + 1024 * 32;
constexpr unsigned TK_OFF = CT_OFF + 256 * 8 * 4;
constexpr unsigned EC_OFF = TK_OFF + 2 * NR * 128;
constexpr unsigned OC_OFF = EC_OFF + NKC * 128;
constexpr unsigned TM_OFF = OC_OFF + 17 * 128;
constexpr unsigned DB_OFF = TM_OFF + 256;
constexpr unsigned UL_OFF = DB_OFF + 512 * 64 * 8;
constexpr unsigned WS_USED = UL_OFF + 8 * 4096 * 8;
constexpr unsigned OUT_BYTES = (unsigned)M_ * N_;

constexpr unsigned long long DEADLINE = 100000000ull;
constexpr int AUX_SYS = 17, AUX_DEV = 16;

#define NW 18
#define NWO 14
#define PACE_IN 1340
#define PACE_OUT 1340
#define NPUSH0 16
#define NPUSH1 12
#define PICK_SLEEP 20
#define FAST_FROM 54
constexpr int TT_START = 100;
constexpr int PACE_FINE = 1800;
constexpr int EXP_LEAD = 900;
constexpr int FLAG_NOW = 128;
constexpr int STAGE2_DELAY = 450, STAGE2_STEP = 60;
constexpr int STAGE2_AT = 100;
constexpr int LINK_WAVES = 2;
constexpr int NEXP = 112 * (8 - LINK_WAVES) + 128;
static_assert(NPUSH0 % 4 == 0 && NPUSH0 <= 16 && NPUSH1 % 4 == 0 && NPUSH1 <= 64 && NW >= 8 && NW <= 32 && NWO >= 8 && NWO <= 32, "");

constexpr int LDS_STAGE = 8 * HALF_B;
constexpr int LDS_BYTES = LDS_STAGE + 256;

struct Params {
    const unsigned short* x;
    const unsigned short* w;
    unsigned char* out[NR];
    unsigned char* ws[NR];
    unsigned char* myws;
    unsigned char* myout;
    int rank;
};

__device__ __forceinline__ unsigned char* uni(const void* p) {
    const unsigned long long v = (unsigned long long)p;
    const unsigned lo = __builtin_amdgcn_readfirstlane((unsigned)v), hi = __builtin_amdgcn_readfirstlane((unsigned)(v >> 32));
    return (unsigned char*)(((unsigned long long)hi << 32) | lo);
}
__device__ __forceinline__ __amdgpu_buffer_rsrc_t mk_rsrc(const void* p, unsigned bytes) {
    return __builtin_amdgcn_make_buffer_rsrc((void*)uni(p), 0, (int)bytes, 0x00020000);
}
#define ST128(V, R, VO, SO, AUX) do { const u32x4 st_v_ = (V); __builtin_amdgcn_raw_buffer_store_b128(st_v_, (R), (VO), (SO), (AUX)); \
        asm volatile("s_nop 1" :: "v"(st_v_)); } while (0)
__device__ __forceinline__ float wave_max(float f) {
    unsigned v = __float_as_uint(f);
    v = max(v, (unsigned)__builtin_amdgcn_update_dpp(0, (int)v, 0x111, 0xf, 0xf, false));
    v = max(v, (unsigned)__builtin_amdgcn_update_dpp(0, (int)v, 0x112, 0xf, 0xf, false));
    v = max(v, (unsigned)__builtin_amdgcn_update_dpp(0, (int)v, 0x114, 0xf, 0xf, false));
    v = max(v, (unsigned)__builtin_amdgcn_update_dpp(0, (int)v, 0x118, 0xf, 0xf, false));
    v = max(v, (unsigned)__builtin_amdgcn_update_dpp(0, (int)v, 0x142, 0xa, 0xf, false));
    v = max(v, (unsigned)__builtin_amdgcn_update_dpp(0, (int)v, 0x143, 0xc, 0xf, false));
    return __uint_as_float((unsigned)__builtin_amdgcn_readlane((int)v, 63));
}
__device__ __forceinline__ unsigned long long now() { return __builtin_amdgcn_s_memrealtime(); }

#define STAMP(i) do { } while (0)
#define STAMPMAX(i) do { } while (0)

#define ARR(r) do { } while (0)

__device__ __forceinline__ unsigned ld_tmo(unsigned* tmo) { return __hip_atomic_load(tmo, __ATOMIC_RELAXED, __HIP_MEMORY_SCOPE_AGENT); }
__device__ __forceinline__ bool give_up(unsigned* tmo, unsigned tmv, unsigned call, unsigned long long t0) {
    bool g = tmv == call || now() - t0 > DEADLINE;
    g = __builtin_amdgcn_readfirstlane(g ? 1 : 0) != 0;
    if (g) __hip_atomic_store(tmo, call, __ATOMIC_RELAXED, __HIP_MEMORY_SCOPE_AGENT);
    return g;
}

__device__ __forceinline__ void wait_until(unsigned long long t0, unsigned dl) {
    for (;;) {
        const int left = (int)(dl + (unsigned)t0 - (unsigned)now());
        if (left <= 0) break;
        if (left > 30) __builtin_amdgcn_s_sleep(8); else __builtin_amdgcn_s_sleep(1);
    }
}

__device__ __forceinline__ void load_w(u32x4 (&buf)[8], __amdgpu_buffer_rsrc_t rw, int kc, unsigned c0, int lane) {
    const unsigned woff = (unsigned)((lane >> 4) * 8 * N_ + (lane & 15) * 4) * 2, sb = ((unsigned)(kc * 64) * N_ + c0) * 2;
#pragma unroll
    for (int i = 0; i < 16; ++i) {
        const u32x2 v = __builtin_amdgcn_raw_buffer_load_b64(rw, woff, sb + (unsigned)(((i >> 3) * 32 + (i & 7)) * N_) * 2, 0);
        buf[i >> 1][(i & 1) * 2] = v.x; buf[i >> 1][(i & 1) * 2 + 1] = v.y;
    }
}
__device__ __forceinline__ void store_w(const u32x4 (&buf)[8], __amdgpu_buffer_rsrc_t rd, unsigned dbase, int lane) {
#pragma unroll
    for (int kh = 0; kh < 2; ++kh)
#pragma unroll
        for (int f = 0; f < 4; ++f) {
            u32x4 o;
#pragma unroll
            for (int dd = 0; dd < 4; ++dd)
                o[dd] = __builtin_amdgcn_perm(buf[kh * 4 + dd][2 + (f >> 1)], buf[kh * 4 + dd][f >> 1], (f & 1) ? 0x07060302u : 0x05040100u);
            ST128(o, rd, lane * 16, dbase + (f >> 1) * HALF_B + ((f & 1) * 2 + kh) * 1024, AUX_SYS);
        }
}

__device__ __forceinline__ void export_role(const Params& p, int eidx, int push_q, int push_u, int wid, int lane, unsigned* ctr) {
    const unsigned long long tstart = now();
    const int rank = p.rank;
    unsigned char* myws = p.myws;
    const unsigned cprev = __hip_atomic_load(ctr, __ATOMIC_RELAXED, __HIP_MEMORY_SCOPE_AGENT);
    unsigned char* peer = p.ws[(rank + lane) & (NR - 1)];
    const __amdgpu_buffer_rsrc_t rme = mk_rsrc(myws, WS_USED);
    const __amdgpu_buffer_rsrc_t rx = mk_rsrc(p.x, (unsigned)M_ * KS * 2);
    const __amdgpu_buffer_rsrc_t rw = mk_rsrc(p.w, (unsigned)KS * N_ * 2);
    u32x4 buf[8];
    if (push_q != 0) load_w(buf, rw, 0, (unsigned)(((rank + push_q) & (NR - 1)) * NB + push_u * 64), lane);
    const unsigned call = __builtin_amdgcn_readfirstlane(cprev) + 1;
    if (lane == 0) __hip_atomic_store(ctr, call, __ATOMIC_RELAXED, __HIP_MEMORY_SCOPE_AGENT);
    if (push_q != 0) {
        const __amdgpu_buffer_rsrc_t rd = mk_rsrc(p.ws[(rank + push_q) & (NR - 1)], WS_USED);
        wait_until(tstart, TT_START + (unsigned)push_u * 144 / 10);
        store_w(buf, rd, WS_OFF + (unsigned)(rank * NKC * NCB + (push_u >> 2)) * TILE_B + (push_u & 3) * 4096, lane);
        asm volatile("s_waitcnt vmcnt(0)" ::: "memory");
        __builtin_amdgcn_raw_buffer_store_b32((call & 0xffffu) * 0x10001u, rd, lane == 0 ? 0u : 0x80000000u,
                                              FW_OFF + (unsigned)(((push_u >> 2) * 64 + rank) * 4 + (push_u & 3)) * 4, AUX_SYS);
    }
    for (int e = eidx; e < NKC * 320; e += NEXP) {
        const int kc = e / 320, r = e - kc * 320;
        unsigned foff; int flane;
        if (kc > 0) wait_until(tstart, (unsigned)(kc * 96 * PACE_IN / 100 - EXP_LEAD + r * 2));
        else if (r >= 256) wait_until(tstart, 150u);
        if (r < 256) {
            const int j = r >> 3, q = r & 7, cb = j >> 2, wc = j & 3;
            if (kc == 0 && j < NPUSH0 && q != 0) continue;
            load_w(buf, rw, kc, (unsigned)(((rank + q) & (NR - 1)) * NB + j * 64), lane);
            store_w(buf, rme, (q == 0 ? WS_OFF + (unsigned)(rank * NKC * NCB) * TILE_B : EW_OFF + (unsigned)(q * NKC * NCB) * TILE_B)
                              + (unsigned)(kc * NCB + cb) * TILE_B + wc * 4096, lane);
            flane = q;
            foff = q == 0 ? FW_OFF + (unsigned)((cb * 64 + kc * 8 + rank) * 4 + wc) * 4 : EF_OFF + (unsigned)(q * NUNIT + j) * 4;
        } else {
            const int rb = (r - 256) >> 2, uq = r & 3;
            const unsigned xoff = (unsigned)((lane & 15) * KS + (lane >> 4) * 8) * 2, sb = (unsigned)((rb * 256 + uq * 64) * KS + kc * 64) * 2;
            const unsigned dbase = XS_OFF + (unsigned)((rank * NKC + kc) * NRB + rb) * TILE_B + uq * 8192;
#pragma unroll
            for (int g = 0; g < 8; ++g)
                buf[g] = __builtin_amdgcn_raw_buffer_load_b128(rx, xoff, sb + (unsigned)((g >> 1) * 16 * KS + (g & 1) * 32) * 2, 0);
#pragma unroll
            for (int g = 0; g < 8; ++g)
                ST128(buf[g], rme, lane * 16, dbase + g * 1024, AUX_SYS);
            flane = -1;
            foff = lane == 0 ? FX_OFF + (unsigned)((rb * 64 + kc * 8 + rank) * 4 + uq) * 4
                             : EF_OFF + (unsigned)(lane * NUNIT + 32 + ((rb - 2 * lane) & 15) * 4 + uq) * 4;
        }
        asm volatile("s_waitcnt vmcnt(0)" ::: "memory");
        if ((flane < 0 ? lane < NR : lane == flane) && (lane == 0 || kc == 0))
            __hip_atomic_store((unsigned*)(peer + foff), lane == 0 ? (call & 0xffffu) * 0x10001u : call, __ATOMIC_RELAXED, __HIP_MEMORY_SCOPE_SYSTEM);
        if (kc > 0) {
            unsigned n = 0;
            if (lane == 0) n = __hip_atomic_fetch_add((unsigned*)(myws + EC_OFF + kc * 128), 1u, __ATOMIC_RELAXED, __HIP_MEMORY_SCOPE_AGENT);
            if (__builtin_amdgcn_readfirstlane(n) + 1 == call * 320u && lane > 0 && lane < NR)
                __hip_atomic_store((unsigned*)(peer + EF_OFF) + lane * NUNIT + kc * 96, call, __ATOMIC_RELAXED, __HIP_MEMORY_SCOPE_SYSTEM);
        }
    }
    STAMPMAX(3);
}

__device__ __forceinline__ void link_role(const Params& p, int q, int widx, int wid, int lane, unsigned* ctr) {
    unsigned long long tstart = now();
    const int rank = p.rank;
    const int src = (rank - q) & (NR - 1);
    unsigned char* myws = p.myws;
    unsigned* tmo = (unsigned*)(myws + TM_OFF);
    const unsigned cprev = __hip_atomic_load(ctr, __ATOMIC_RELAXED, __HIP_MEMORY_SCOPE_AGENT);
    const __amdgpu_buffer_rsrc_t rme = mk_rsrc(myws, WS_USED);
    const unsigned l0 = lane == 0 ? 0u : 0x80000000u;
    const unsigned call = __builtin_amdgcn_readfirstlane(cprev) + 1;
    if (lane == 0) __hip_atomic_store(ctr, call, __ATOMIC_RELAXED, __HIP_MEMORY_SCOPE_AGENT);
    u32x4 bufa[4], bufb[4];
    const unsigned char* tk = uni(myws + TK_OFF + q * 128);
    unsigned tick;
#define DRAW() asm volatile("s_mov_b64 exec, 1\n\tglobal_atomic_add %0, %1, %2, %3 sc0\n\ts_mov_b64 exec, -1" : "=&v"(tick) : "v"(0u), "v"(1u), "s"(tk) : "memory")
#define ULOG_TOP
#define ULOG_PUT(I, T, E)
#define DUE(T0, DL) ((int)((DL) + (unsigned)(T0) - (unsigned)now()) <= 0)

    if (widx < NW) {
        const __amdgpu_buffer_rsrc_t rs = mk_rsrc(p.ws[src], WS_USED);
#define WAIT_EXPORTED(FO, COND) do { const unsigned long long t0_ = now(); \
            for (;;) { \
                const unsigned tmv = ld_tmo(tmo); \
                const u32x2 v = __builtin_amdgcn_raw_buffer_load_b64(rme, (FO), EF_OFF + (unsigned)(q * NUNIT) * 4, AUX_SYS); \
                if (__all(COND)) break; \
                if (give_up(tmo, tmv, call, t0_)) break; \
                __builtin_amdgcn_s_sleep(2); \
            } } while (0)
#define IN_ADDR(H, SB, DB, FO) do { const int kc_ = ((H) >> 1) / 96, j_ = ((H) >> 1) - kc_ * 96, i_ = (j_ & 3) * 2 + ((H) & 1); \
            if (j_ < 32) { \
                const unsigned o_ = (unsigned)(kc_ * NCB + (j_ >> 2)) * TILE_B + i_ * 1024; \
                SB = EW_OFF + (unsigned)(q * NKC * NCB) * TILE_B + o_; DB = WS_OFF + (unsigned)(src * NKC * NCB) * TILE_B + o_; \
                FO = FW_OFF + (unsigned)(((j_ >> 2) * 64 + kc_ * 8 + src) * 8 + i_) * 2; \
            } else { \
                const int rb_ = (((j_ - 32) >> 2) + 2 * q) & 15; \
                SB = DB = XS_OFF + (unsigned)((src * NKC + kc_) * NRB + rb_) * TILE_B + i_ * 1024; \
                FO = FX_OFF + (unsigned)((rb_ * 64 + kc_ * 8 + src) * 8 + i_) * 2; \
            } } while (0)
#define IN_LOAD(BUF, SB) do { _Pragma("unroll") for (int g = 0; g < 4; ++g) \
            BUF[g] = __builtin_amdgcn_raw_buffer_load_b128(rs, lane * 16, (SB) + g * 8192, AUX_SYS); } while (0)
#define FLAG(FO) __builtin_amdgcn_raw_buffer_store_b16((unsigned short)call, rme, l0, (FO), AUX_DEV)
        DRAW();
        WAIT_EXPORTED((unsigned)(NPUSH0 + (lane < (32 - NPUSH0) / 2 ? lane * 2 : 0)) * 4, v.x == call && v.y == call);
        {
            const int late = (int)((unsigned)(now() - tstart) - (TT_START + (unsigned)NPUSH0 * PACE_IN / 100));
            if (late > 0) tstart += (unsigned)late;
        }
        const unsigned tb = (call - 1) * (unsigned)(2 * (NUNIT - NPUSH0)) - (2 * NPUSH0 + NW);
        int u_ok = 32;
        unsigned fprev = 0;
        unsigned sb, db, fo, dbn, fon;
        IN_ADDR(2 * NPUSH0 + widx, sb, db, fo);
        wait_until(tstart, TT_START + (unsigned)(2 * NPUSH0 + widx) * PACE_IN / 200);
        IN_LOAD(bufa, sb);
#define IN_STEP(CUR, NXT) { \
              \
            asm volatile("s_waitcnt vmcnt(0)" : "+v"(tick), "+v"(CUR[0]), "+v"(CUR[1]), "+v"(CUR[2]), "+v"(CUR[3]) :: "memory"); \
            ULOG_TOP; \
            const unsigned hr = __builtin_amdgcn_readfirstlane(tick) - tb; \
            const bool valid = hr < 2u * NUNIT; \
            const int hn = valid ? (int)hr : 2 * NUNIT - 1; \
            while (valid && (hn >> 1) >= u_ok) {                             \
                if (u_ok > 32) { WAIT_EXPORTED((unsigned)u_ok * 4, v.x == call); u_ok += 96; } \
                else { WAIT_EXPORTED((unsigned)(32 + (lane & 31) * 2) * 4, v.x == call && v.y == call); u_ok = 96; } \
            } \
            IN_ADDR(hn, sb, dbn, fon); \
            const unsigned dl = TT_START + (unsigned)hn * PACE_IN / 200; \
            const bool early = valid && DUE(tstart, dl); \
            if (early) IN_LOAD(NXT, sb); \
            asm volatile("" ::: "memory"); \
            fprev = __builtin_amdgcn_readfirstlane(fprev); db = __builtin_amdgcn_readfirstlane(db); fo = __builtin_amdgcn_readfirstlane(fo); \
            if (fprev) FLAG(fprev); \
            _Pragma("unroll") for (int g = 0; g < 4; ++g) ST128(CUR[g], rme, lane * 16, db + g * 8192, AUX_DEV); \
            fprev = fo; \
            if (valid) DRAW(); \
            if (hn >= 2 * NUNIT - FLAG_NOW && !early) {                      \
                asm volatile("s_waitcnt vmcnt(0)" : "+v"(tick) :: "memory"); \
                FLAG(fprev); fprev = 0; \
            } \
            if (!valid) break; \
            if (!early) { wait_until(tstart, dl); IN_LOAD(NXT, sb); } \
            ULOG_PUT(hn, ttop, early); \
            db = dbn; fo = fon; }
        for (;;) {
            IN_STEP(bufa, bufb)
            IN_STEP(bufb, bufa)
        }
#undef WAIT_EXPORTED
    }
    STAMPMAX(1);

    if (widx < NWO) {
        const __amdgpu_buffer_rsrc_t rd = mk_rsrc(p.myout, OUT_BYTES);
        bool bad;
#define WAIT_TILES(STAGE, MASK) do { const unsigned long long t0_ = now(); \
            for (;;) { \
                const unsigned tmv = ld_tmo(tmo); \
                const u32x2 v = __builtin_amdgcn_raw_buffer_load_b64(rme, lane * 8, OF_OFF + ((STAGE) * NR + q) * 512, AUX_SYS); \
                bad = __builtin_amdgcn_readfirstlane(tmv) == call; \
                seen = __ballot(v.x == call && v.y == call); \
                if ((seen & (MASK)) == (MASK)) break; \
                if (give_up(tmo, tmv, call, t0_)) { bad = true; seen = ~0ull; break; } \
                __builtin_amdgcn_s_sleep(3); \
            } \
              \
            bm = __builtin_amdgcn_readfirstlane(bad ? 0x7f7f7f7f : 0); \
            rs = mk_rsrc(p.out[src], bm ? 0u : OUT_BYTES); } while (0)
#define OUT_RB(H) ((((H) >> 3) & 3) * 4 + (((H) >> 5) & 3))
#define RB_MASK(K) (1ull << (OUT_RB(2 * NPUSH1 + (K)) * 4))
#define OUT_ROW(H) (OUT_RB(H) * 256 + ((H) >> 10) * 128 + (((H) >> 7) & 1) * 64 + (((H) >> 8) & 3) * 16 + ((H) & 7) * 2)
#define OUT_OFF(K) ((unsigned)OUT_ROW(2 * NPUSH1 + (K)) * N_ + src * NB)
#define OFRAG(g) (((g) >> 1) * N_ + ((g) & 1) * 1024)
#define OUT_LOAD(BUF, OB) do { _Pragma("unroll") for (int g = 0; g < 4; ++g) \
            BUF[g] = __builtin_amdgcn_raw_buffer_load_b128(rs, lane * 16, (OB) + OFRAG(g), AUX_SYS); } while (0)
#define OUT_SLOT(K) ((unsigned)(K) * PACE_OUT / 200)
        unsigned bm;
        unsigned long long seen;
        __amdgpu_buffer_rsrc_t rs;
        tk += NR * 128;
        DRAW();
        WAIT_TILES(0, RB_MASK(widx));
        const unsigned long long tf = now();
        STAMPMAX(2);
        const unsigned tb = (call - 1) * (unsigned)(2 * (NOUT - NPUSH1)) - NWO;
        bool stage2 = false;
        unsigned ob = OUT_OFF(widx);
        wait_until(tf, OUT_SLOT(widx));
        OUT_LOAD(bufa, ob);
#define OUT_STEP(CUR, NXT) { \
            asm volatile("s_waitcnt vmcnt(0)" : "+v"(tick), "+v"(CUR[0]), "+v"(CUR[1]), "+v"(CUR[2]), "+v"(CUR[3]) :: "memory"); \
            ULOG_TOP; \
            const unsigned kr = __builtin_amdgcn_readfirstlane(tick) - tb; \
            const bool valid = kr < 2u * (NOUT - NPUSH1); \
            const int kn = valid ? (int)kr : 2 * (NOUT - NPUSH1) - 1; \
            const unsigned bmc = bm; \
            if (!stage2) { \
                if (NPUSH1 + (kn >> 1) >= STAGE2_AT) { WAIT_TILES(1, 1ull); stage2 = true; } \
                else if ((seen & RB_MASK(kn)) != RB_MASK(kn)) WAIT_TILES(0, RB_MASK(kn)); \
            } \
            const unsigned obn = OUT_OFF(kn), dl = OUT_SLOT(kn); \
            const bool early = valid && DUE(tf, dl); \
            if (early) OUT_LOAD(NXT, obn); \
            asm volatile("" ::: "memory"); \
            ob = __builtin_amdgcn_readfirstlane(ob); \
            _Pragma("unroll") for (int g = 0; g < 4; ++g) ST128(CUR[g] | bmc, rd, lane * 16, ob + OFRAG(g), AUX_DEV); \
            if (!valid) break; \
            DRAW(); \
            if (!early) { wait_until(tf, dl); OUT_LOAD(NXT, obn); } \
            ULOG_PUT(2048 + kn, ttop, early); \
            ob = obn; }
        for (;;) {
            OUT_STEP(bufa, bufb)
            OUT_STEP(bufb, bufa)
        }
        asm volatile("s_waitcnt vmcnt(0)" ::: "memory");
    }
    STAMPMAX(6);
}

#define WAIT_V(n) asm volatile("s_waitcnt vmcnt(" #n ")" ::: "memory")
#define WAIT_L(n) asm volatile("s_waitcnt lgkmcnt(" #n ")" ::: "memory")
#define BAR __builtin_amdgcn_s_barrier()
#define SCHED __builtin_amdgcn_sched_barrier(0)

__device__ __forceinline__ void gemm_role(const Params& p, LAS unsigned char* lds, int rb, int cb, int wid, int lane, unsigned call) {
    const int rank = p.rank;
    unsigned char* myws = p.myws;
    unsigned* tmo = (unsigned*)(myws + TM_OFF);
    const __amdgpu_buffer_rsrc_t rws = mk_rsrc(myws, WS_USED);
    const int wr = wid >> 2, wc = wid & 3;
    const unsigned tid16 = (unsigned)(wid * 64 + lane) * 16;
    volatile LAS int* sync_word = (volatile LAS int*)(lds + LDS_STAGE);
    unsigned char* peer_ws = p.ws[lane & 7];

    int tiv;
    {
        int s = rank, kc = lane;
        if (lane >= 8) {
            const int j = lane - 8, i = j % 7, q0 = rb >> 1;
            kc = j / 7;
            const int q = (i < q0) ? q0 - i : 7 - (i - q0);
            s = (rank - q) & (NR - 1);
        }
        tiv = s * 8 + kc;
    }
    const unsigned xtile0 = XS_OFF + rb * TILE_B, wtile0 = WS_OFF + cb * TILE_B;
    unsigned long long avail = 0, taken = 0;

    const unsigned fi = (unsigned)((tiv & 7) * 8 + (tiv >> 3)) * 16;
    const unsigned fxo = FX_OFF + rb * 1024 + fi, fwo = FW_OFF + cb * 1024 + fi;
    const unsigned cc = (call & 0xffffu) * 0x10001u;
#define FLAGS_OK(fx, fw) ((((fx.x ^ cc) | (fx.y ^ cc) | (fx.z ^ cc) | (fx.w ^ cc)) | ((fw.x ^ cc) | (fw.y ^ cc) | (fw.z ^ cc) | (fw.w ^ cc))) == 0)
#define PICK(T, SLOT, FAST) do { \
    if (avail == 0) { \
        if (wid == 0) { \
            unsigned long long r; \
            const unsigned long long t0 = now(); \
            for (;;) { \
                const unsigned tmv = ld_tmo(tmo); \
                const u32x4 fx = __builtin_amdgcn_raw_buffer_load_b128(rws, fxo, 0, AUX_DEV); \
                const u32x4 fw = __builtin_amdgcn_raw_buffer_load_b128(rws, fwo, 0, AUX_DEV); \
                r = __ballot(FLAGS_OK(fx, fw)) & ~taken; \
                if (r) break; \
                if (give_up(tmo, tmv, call, t0)) { r = ~taken; break; } \
                if (FAST) __builtin_amdgcn_s_sleep(2); else __builtin_amdgcn_s_sleep(PICK_SLEEP); \
            } \
            ARR(r); \
            if (lane == 0) { sync_word[2 * (SLOT)] = (int)(unsigned)r; sync_word[2 * (SLOT) + 1] = (int)(unsigned)(r >> 32); } \
            asm volatile("s_waitcnt lgkmcnt(0)" ::: "memory"); \
        } \
        asm volatile("" ::: "memory"); BAR; asm volatile("" ::: "memory"); \
        const unsigned lo_ = __builtin_amdgcn_readfirstlane(sync_word[2 * (SLOT)]), hi_ = __builtin_amdgcn_readfirstlane(sync_word[2 * (SLOT) + 1]); \
        avail = ((unsigned long long)hi_ << 32) | lo_; \
        asm volatile("s_waitcnt lgkmcnt(0)" ::: "memory"); \
    } \
    { const int i_ = __builtin_ctzll(avail); const unsigned long long b_ = 1ull << i_; avail &= ~b_; taken |= b_; T = (unsigned)__builtin_amdgcn_readlane(tiv, i_); } \
    } while (0)

#define SA(b, h) (((b) * 2 + (h)) * HALF_B)
#define SB(b, h) ((4 + (b) * 2 + (h)) * HALF_B)
#define XOFF(T) (xtile0 + (T) * (NRB * TILE_B))
#define WOFF(T) (wtile0 + (T) * (NCB * TILE_B))
#define STAGE(P, SRC) do { \
        __builtin_amdgcn_raw_ptr_buffer_load_lds(rws, (LAS void*)(lds + (P) + wid * 1024), 16, tid16, (SRC), 0, AUX_DEV); \
        __builtin_amdgcn_raw_ptr_buffer_load_lds(rws, (LAS void*)(lds + (P) + 8192 + wid * 1024), 16, tid16, (SRC) + 8192, 0, AUX_DEV); } while (0)
#define LDA(dst, b, h) _Pragma("unroll") for (int m = 0; m < 4; ++m) _Pragma("unroll") for (int k = 0; k < 2; ++k) \
        dst[m][k] = *(const LAS bf16x8*)(abase + SA(b, h) + (m * 2 + k) * 1024)
#define LDB(dst, b, h) _Pragma("unroll") for (int n = 0; n < 2; ++n) _Pragma("unroll") for (int k = 0; k < 2; ++k) \
        dst[n][k] = *(const LAS bf16x8*)(bbase + SB(b, h) - 4 * HALF_B + (n * 2 + k) * 1024)
#define MMA(ai, bj, At, Bt) do { __builtin_amdgcn_s_setprio(1); \
        _Pragma("unroll") for (int m = 0; m < 4; ++m) _Pragma("unroll") for (int n = 0; n < 2; ++n) _Pragma("unroll") for (int k = 0; k < 2; ++k) \
            acc[ai][bj][m][n] = __builtin_amdgcn_mfma_f32_16x16x32_bf16(Bt[n][k], At[m][k], acc[ai][bj][m][n], 0, 0, 0); \
        __builtin_amdgcn_s_setprio(0); } while (0)

#define HALF_MAX(ai, A, B) do { _Pragma("unroll") for (int bj = 0; bj < 2; ++bj) _Pragma("unroll") for (int m = 0; m < 4; ++m) { \
        const f32x4 u_ = acc[ai][bj][m][0], v_ = acc[ai][bj][m][1]; \
        A = fmaxf(fmaxf(A, fabsf(u_[0])), fabsf(u_[1])); B = fmaxf(fmaxf(B, fabsf(v_[0])), fabsf(v_[1])); \
        A = fmaxf(fmaxf(A, fabsf(u_[2])), fabsf(u_[3])); B = fmaxf(fmaxf(B, fabsf(v_[2])), fabsf(v_[3])); } } while (0)
    float am0 = 0.f, am1 = 0.f;
    const LAS unsigned char* abase = lds + wr * 8192 + lane * 16;
    const LAS unsigned char* bbase = lds + 4 * HALF_B + wc * 4096 + lane * 16;
    f32x4 acc[2][2][4][2] = {};
    bf16x8 At[4][2], B0[2][2], B1[2][2];

    unsigned ta, tb;
    PICK(ta, 0, false);
    STAMP(1);
    {
        const unsigned x0 = XOFF(ta), w0 = WOFF(ta);
        STAGE(SB(0, 0), w0); STAGE(SA(0, 0), x0);
        STAGE(SB(0, 1), w0 + HALF_B); STAGE(SA(0, 1), x0 + HALF_B);
        PICK(tb, 1, false);
        const unsigned x1 = XOFF(tb), w1 = WOFF(tb);
        if (wr == 1) BAR;
        WAIT_V(4); BAR;
        STAGE(SB(1, 0), w1); STAGE(SA(1, 0), x1); STAGE(SB(1, 1), w1 + HALF_B);
        WAIT_V(6); BAR;
    }
    for (int t = 0; t < NT - 2; t += 2) {
        STAMP(8 + (t >> 1));
        const bool fast = t >= FAST_FROM;
        LDB(B0, 0, 0); SCHED; LDA(At, 0, 0); STAGE(SA(1, 1), XOFF(tb) + HALF_B);
        WAIT_V(10); WAIT_L(8); BAR; WAIT_L(0); MMA(0, 0, At, B0); BAR; SCHED;
        PICK(ta, 2, fast);
        const unsigned x2 = XOFF(ta), w2 = WOFF(ta);
        LDB(B1, 0, 1); STAGE(SB(0, 0), w2);
        WAIT_V(10); BAR; WAIT_L(0); MMA(0, 1, At, B1); BAR;
        LDA(At, 0, 1); STAGE(SA(0, 0), x2);
        WAIT_V(10); BAR; WAIT_L(0); MMA(1, 0, At, B0); BAR; SCHED;
        STAGE(SB(0, 1), w2 + HALF_B);
        WAIT_V(10); BAR; MMA(1, 1, At, B1); BAR;
        LDB(B0, 1, 0); SCHED; LDA(At, 1, 0); STAGE(SA(0, 1), x2 + HALF_B);
        WAIT_V(10); WAIT_L(8); BAR; WAIT_L(0); MMA(0, 0, At, B0); BAR; SCHED;
        PICK(tb, 3, fast);
        const unsigned x3 = XOFF(tb), w3 = WOFF(tb);
        LDB(B1, 1, 1); STAGE(SB(1, 0), w3);
        WAIT_V(10); BAR; WAIT_L(0); MMA(0, 1, At, B1); BAR;
        LDA(At, 1, 1); STAGE(SA(1, 0), x3);
        WAIT_V(10); BAR; WAIT_L(0); MMA(1, 0, At, B0); BAR; SCHED;
        STAGE(SB(1, 1), w3 + HALF_B);
        WAIT_V(10); BAR; MMA(1, 1, At, B1); BAR;
    }
    STAMP(2);
    {
        const unsigned xl = XOFF(tb);
        LDB(B0, 0, 0); WAIT_V(0); LDA(At, 0, 0); STAGE(SA(1, 1), xl + HALF_B);
        BAR; WAIT_L(0); MMA(0, 0, At, B0); BAR;
        LDB(B1, 0, 1); BAR; WAIT_L(0); MMA(0, 1, At, B1); BAR;
        LDA(At, 0, 1); WAIT_V(4); BAR; WAIT_L(0); MMA(1, 0, At, B0); MMA(1, 1, At, B1); BAR;
    }
    {
        LDB(B0, 1, 0); LDA(At, 1, 0); WAIT_V(2); BAR; WAIT_L(0); MMA(0, 0, At, B0); BAR;
        LDB(B1, 1, 1); WAIT_V(0); BAR; WAIT_L(0); MMA(0, 1, At, B1); BAR;
        LDA(At, 1, 1); BAR; WAIT_L(0); MMA(1, 0, At, B0); MMA(1, 1, At, B1);
    }
    volatile LAS float* red = (volatile LAS float*)(lds + LDS_STAGE + 128);
    {
        float am2 = 0.f, am3 = 0.f;
        HALF_MAX(0, am0, am1);
        HALF_MAX(1, am2, am3);
        const float am = wave_max(fmaxf(fmaxf(am0, am1), fmaxf(am2, am3)));
        if (lane == 0) red[wid] = am;
    }
    WAIT_L(0); BAR;
    if (wr == 0) BAR;
    asm volatile("" ::: "memory");
    STAMP(3);

    const int gid = rb * NCB + cb;
    if (wid == 0) {
        const float a = wave_max(red[lane & 7]);
        if (lane < NR)
            __hip_atomic_store((unsigned long long*)(peer_ws + AM_OFF) + rank * 128 + gid,
                               ((unsigned long long)call << 32) | __float_as_uint(a), __ATOMIC_RELAXED, __HIP_MEMORY_SCOPE_SYSTEM);
    }
    unsigned char* po[NR - 1];
#pragma unroll
    for (int j = 0; j < NR - 1; ++j) { po[j] = uni(p.out[(rank + 1 + (gid + wc + j) % 7) & (NR - 1)]); asm volatile("" : "+s"(po[j])); }
    {
        const unsigned long long t0 = now();
        float g; bool bd;
        for (;;) {
            const unsigned tmv = ld_tmo(tmo);
            const u32x4 v = __builtin_amdgcn_raw_buffer_load_b128(rws, AM_OFF + tid16, 0, AUX_SYS);
            g = fmaxf(__uint_as_float(v.x), __uint_as_float(v.z));
            bd = tmv == call;
            if (__all(v.y == call && v.w == call)) break;
            if (give_up(tmo, tmv, call, t0)) { bd = true; break; }
            __builtin_amdgcn_s_sleep(1);
        }
        g = wave_max(g);
        if (lane == 0) red[8 + wid] = bd ? __builtin_nanf("") : g;
    }
    WAIT_L(0); BAR; asm volatile("" ::: "memory");
    const unsigned long long tout = now();
    float gmax = 0.f; bool bad = false;
#pragma unroll
    for (int i = 0; i < 8; ++i) { const float r = red[8 + i]; bad |= r != r; gmax = fmaxf(gmax, r); }
    STAMP(4);

    const float scale = gmax / 448.0f;
    const float inv = 1.0f / scale;
#define QUANT(O, ai, m) do { \
        const f32x4 a_ = acc[ai][0][m][0] * inv, b_ = acc[ai][0][m][1] * inv, c_ = acc[ai][1][m][0] * inv, e_ = acc[ai][1][m][1] * inv; \
        _Pragma("unroll") for (int r = 0; r < 4; ++r) { \
            int pk = __builtin_amdgcn_cvt_pk_fp8_f32(a_[r], b_[r], 0, false); \
            pk = __builtin_amdgcn_cvt_pk_fp8_f32(c_[r], e_[r], pk, true); \
            O[r] = bad ? 0x7f7f7f7fu : (unsigned)pk; \
        } } while (0)
    const unsigned ooff = (unsigned)(rb * 256 + wr * 64 + (lane & 15)) * N_ + rank * NB + cb * 256 + wc * 64 + (lane >> 4) * 16;
    const int rbi = (rb & 3) * 4 + (rb >> 2);
    const bool pusher = wr == 0 && rbi * 4 < NPUSH1;
    const unsigned dl = (unsigned)((rbi * 8 + cb) * 4 + wc) * PACE_FINE / 1000;
    const __amdgpu_buffer_rsrc_t rl = mk_rsrc(p.myout, OUT_BYTES);
    u32x4 q0;
    QUANT(q0, 0, 0);
    ST128(q0, rl, ooff, 0, AUX_SYS);
    asm volatile("" ::: "memory");
    const bool early = pusher && (unsigned)(now() - tout) >= dl;
#define PUSH_FIRST() do { _Pragma("unroll") for (int j = 0; j < NR - 1; ++j) \
        ST128(q0, mk_rsrc(po[j], OUT_BYTES), ooff, 0, AUX_SYS); } while (0)
    if (early) { PUSH_FIRST(); WAIT_V(7); } else { WAIT_V(0); }
    BAR; asm volatile("" ::: "memory");
    const int lq = ((lane & 7) - rank) & (NR - 1);
#define ANNOUNCE(CTR, N, WORD) do { unsigned n_ = 0; \
        if (lane == 0) n_ = __hip_atomic_fetch_add((unsigned*)(myws + OC_OFF + (CTR) * 128), 1u, __ATOMIC_RELAXED, __HIP_MEMORY_SCOPE_AGENT); \
        if (__builtin_amdgcn_readfirstlane(n_) + 1 == call * (N) && lane < NR && lq != 0) \
            __hip_atomic_store((unsigned long long*)(peer_ws + OF_OFF) + lq * 64 + (WORD), (unsigned long long)call * 0x100000001ull, __ATOMIC_RELAXED, __HIP_MEMORY_SCOPE_SYSTEM); } while (0)
    if (wid == 7) ANNOUNCE(rb, 8u, rb * 4);
    STAMP(7);
    if (pusher && !early) { wait_until(tout, dl); PUSH_FIRST(); }
    wait_until(tout, (unsigned)(STAGE2_DELAY + wid * STAGE2_STEP));
#pragma unroll
    for (int i = 1; i < 8; ++i) {
        u32x4 qi;
        QUANT(qi, i >> 2, i & 3);
        ST128(qi, rl, ooff, (unsigned)((i >> 2) * 128 + (i & 3) * 16) * N_, AUX_SYS);
    }
    WAIT_V(0);
    if (pusher && lane < NR)
        __hip_atomic_store((unsigned*)(peer_ws + DN_OFF + (unsigned)(gid * 8 + wid) * 32) + lq, call, __ATOMIC_RELAXED, __HIP_MEMORY_SCOPE_SYSTEM);
    BAR; asm volatile("" ::: "memory");
    if (wid == 7) ANNOUNCE(16, 128u, NR * 64);
    STAMP(5);
    if (pusher) {
        const unsigned dn = DN_OFF + (unsigned)(gid * 8 + wid) * 32;
        const unsigned long long t0 = now();
        for (;;) {
            const unsigned tmv = ld_tmo(tmo);
            const unsigned v = __builtin_amdgcn_raw_buffer_load_b32(rws, (lane & 7) * 4, dn, AUX_SYS);
            if (__all(v == call)) break;
            if (give_up(tmo, tmv, call, t0)) { if (lane == 0) p.myout[ooff] = 0x7f; break; }
            __builtin_amdgcn_s_sleep(2);
        }
    }
    STAMPMAX(6);
}

__global__ void __launch_bounds__(512, 2) fused_kernel(const Params p) {
    extern __shared__ __attribute__((aligned(16))) unsigned char smem[];
    const int b = blockIdx.x, xcd = b & 7, slot = b >> 3;
    const int wid = __builtin_amdgcn_readfirstlane(threadIdx.x >> 6), lane = threadIdx.x & 63;
    unsigned char* myws = p.myws;
    unsigned* ctr = (unsigned*)(myws + CT_OFF) + b * 8 + wid;
    STAMP(0);
    if (slot < 16) {
        const int q = slot & 7, wgi = xcd * 2 + (slot >> 3);
        if (q != 0 && wid < LINK_WAVES) { if (wid * 16 + wgi < (NW > NWO ? NW : NWO)) link_role(p, q, wid * 16 + wgi, wid, lane, ctr); }
        else export_role(p, q == 0 ? wid * 16 + wgi : 128 + (wid - LINK_WAVES) * 112 + (q - 1) * 16 + wgi,
                         (wid == 7 && wgi < NPUSH0) ? q : 0, wgi, wid, lane, ctr);
    } else {
        const int g = slot - 16;
        const unsigned call = __builtin_amdgcn_readfirstlane(__hip_atomic_load(ctr, __ATOMIC_RELAXED, __HIP_MEMORY_SCOPE_AGENT)) + 1;
        if (lane == 0) __hip_atomic_store(ctr, call, __ATOMIC_RELAXED, __HIP_MEMORY_SCOPE_AGENT);
        gemm_role(p, (LAS unsigned char*)smem, (xcd >> 1) * 4 + (g >> 2), (xcd & 1) * 4 + (g & 3), wid, lane, call);
    }
}

}

extern "C" void kernel_launch(int rank, int nranks, ncclComm_t comm, hipStream_t stream,
                              void* const* d_in, const int* in_sizes, int n_in,
                              void* d_out, int out_size,
                              const int* in_dtypes, int out_dtype,
                              void* const* d_out_all,
                              void* const* d_ws_all, size_t ws_bytes) {
    if (nranks != NR || (unsigned)rank >= (unsigned)NR || n_in < 2 || ws_bytes < WS_USED ||
        in_sizes[0] != M_ * KS || in_sizes[1] != KS * N_ || out_size != M_ * N_) {
        fprintf(stderr, "kernel_launch: unexpected problem shape\n");
        return;
    }
    static bool attr_set = false;
    if (!attr_set) {
        HIP_CHECK(hipFuncSetAttribute((const void*)fused_kernel, hipFuncAttributeMaxDynamicSharedMemorySize, LDS_BYTES));
        attr_set = true;
    }
    Params p;
    p.x = (const unsigned short*)d_in[0];
    p.w = (const unsigned short*)d_in[1];
    for (int i = 0; i < NR; ++i) {
        p.out[i] = (unsigned char*)d_out_all[i];
        p.ws[i] = (unsigned char*)d_ws_all[i];
    }
    p.myws = p.ws[rank];
    p.myout = (unsigned char*)d_out;
    p.rank = rank;
    hipLaunchKernelGGL(fused_kernel, dim3(256), dim3(512), LDS_BYTES, stream, p);
    HIP_CHECK(hipGetLastError());
}
```

```cpp
#include <hip/hip_runtime.h>
#include <rccl/rccl.h>
#include <cstdio>
#include <cstdint>

#define HIP_CHECK(expr) do { hipError_t _e = (expr); if (_e != hipSuccess) { \
    fprintf(stderr, "HIP error %s at %s:%d\n", hipGetErrorString(_e), __FILE__, __LINE__); \
    return; } } while (0)

namespace {

#define LAS __attribute__((address_space(3)))
typedef short bf16x8 __attribute__((ext_vector_type(8)));
typedef float f32x4 __attribute__((ext_vector_type(4)));
typedef unsigned u32x4 __attribute__((ext_vector_type(4)));
typedef unsigned u32x2 __attribute__((ext_vector_type(2)));

constexpr int NR = 8;
constexpr int M_ = 4096, K_ = 4096, N_ = 16384;
constexpr int KS = K_ / NR;
constexpr int NB = N_ / NR;
constexpr int NRB = 16, NCB = 8;
constexpr int NKC = 8;
constexpr int NT = 64;
constexpr unsigned TILE_B = 32768, HALF_B = 16384;
constexpr int NUNIT = NKC * 96;
constexpr int NOUT = 1024;

constexpr unsigned XS_OFF = 0;
constexpr unsigned WS_OFF = XS_OFF + 64u * NRB * TILE_B;
constexpr unsigned EW_OFF = WS_OFF + 64u * NCB * TILE_B;
constexpr unsigned FX_OFF = EW_OFF + 64u * NCB * TILE_B;
constexpr unsigned FW_OFF = FX_OFF + NRB * 64 * 16;
constexpr unsigned EF_OFF = FW_OFF + NCB * 64 * 16;
constexpr unsigned OF_OFF = EF_OFF + NR * NUNIT * 4;
constexpr unsigned AM_OFF = OF_OFF + 2 * NR * 512;
constexpr unsigned DN_OFF = AM_OFF + NR * 128 * 8;
constexpr unsigned CT_OFF = DN_OFF + 1024 * 32;
constexpr unsigned TK_OFF = CT_OFF + 256 * 8 * 4;
constexpr unsigned EC_OFF = TK_OFF + 2 * NR * 128;
constexpr unsigned AL_OFF = EC_OFF + 9 * 128;
constexpr unsigned OC_OFF = AL_OFF + 128;
constexpr unsigned TM_OFF = OC_OFF + 17 * 128;
constexpr unsigned DB_OFF = TM_OFF + 256;
constexpr unsigned UL_OFF = DB_OFF + 512 * 64 * 8;
constexpr unsigned WS_USED = UL_OFF + 8 * 4096 * 8;
constexpr unsigned OUT_BYTES = (unsigned)M_ * N_;

constexpr unsigned long long DEADLINE = 100000000ull;
constexpr int AUX_SYS = 17, AUX_DEV = 16;

#define NW 18
#define NWO 14
#define PACE_IN 1340
#define PACE_OUT 1340
#define NPUSH0 16
#define NPUSH1 12
#define EARLY_PUSH 1
#define EARLY_LEAD 0
#define PICK_SLEEP 20
#define FAST_FROM 54
constexpr int TT_START = 100;
constexpr int PACE_FINE = 1800;
constexpr int EXP_LEAD = 900;
constexpr int FLAG_NOW = 128;
constexpr int STAGE2_DELAY = 450, STAGE2_STEP = 60;
constexpr int STAGE2_AT = 100;
constexpr int LINK_WAVES = 2;
constexpr int NEXP = 112 * 4 + 128;
constexpr int NEXP_W0 = 256 - 7 * NPUSH0;
static_assert(NPUSH0 % 4 == 0 && NPUSH0 <= 16 && NPUSH1 % 4 == 0 && NPUSH1 <= 64 && NW >= 8 && NW <= 32 && NWO >= 8 && NWO <= 32, "");

constexpr int LDS_STAGE = 8 * HALF_B;
constexpr int LDS_BYTES = LDS_STAGE + 256;

struct Params {
    const unsigned short* x;
    const unsigned short* w;
    unsigned char* out[NR];
    unsigned char* ws[NR];
    unsigned char* myws;
    unsigned char* myout;
    int rank;
};

__device__ __forceinline__ unsigned char* uni(const void* p) {
    const unsigned long long v = (unsigned long long)p;
    const unsigned lo = __builtin_amdgcn_readfirstlane((unsigned)v), hi = __builtin_amdgcn_readfirstlane((unsigned)(v >> 32));
    return (unsigned char*)(((unsigned long long)hi << 32) | lo);
}
__device__ __forceinline__ __amdgpu_buffer_rsrc_t mk_rsrc(const void* p, unsigned bytes) {
    return __builtin_amdgcn_make_buffer_rsrc((void*)uni(p), 0, (int)bytes, 0x00020000);
}
#define ST128(V, R, VO, SO, AUX) do { const u32x4 st_v_ = (V); __builtin_amdgcn_raw_buffer_store_b128(st_v_, (R), (VO), (SO), (AUX)); \
        asm volatile("s_nop 1" :: "v"(st_v_)); } while (0)
__device__ __forceinline__ float wave_max(float f) {
    unsigned v = __float_as_uint(f);
    v = max(v, (unsigned)__builtin_amdgcn_update_dpp(0, (int)v, 0x111, 0xf, 0xf, false));
    v = max(v, (unsigned)__builtin_amdgcn_update_dpp(0, (int)v, 0x112, 0xf, 0xf, false));
    v = max(v, (unsigned)__builtin_amdgcn_update_dpp(0, (int)v, 0x114, 0xf, 0xf, false));
    v = max(v, (unsigned)__builtin_amdgcn_update_dpp(0, (int)v, 0x118, 0xf, 0xf, false));
    v = max(v, (unsigned)__builtin_amdgcn_update_dpp(0, (int)v, 0x142, 0xa, 0xf, false));
    v = max(v, (unsigned)__builtin_amdgcn_update_dpp(0, (int)v, 0x143, 0xc, 0xf, false));
    return __uint_as_float((unsigned)__builtin_amdgcn_readlane((int)v, 63));
}
__device__ __forceinline__ unsigned long long now() { return __builtin_amdgcn_s_memrealtime(); }

#define STAMP(i) do { } while (0)
#define STAMPMAX(i) do { } while (0)

#define ARR(r) do { } while (0)

__device__ __forceinline__ unsigned ld_tmo(unsigned* tmo) { return __hip_atomic_load(tmo, __ATOMIC_RELAXED, __HIP_MEMORY_SCOPE_AGENT); }
__device__ __forceinline__ bool give_up(unsigned* tmo, unsigned tmv, unsigned call, unsigned long long t0) {
    bool g = tmv == call || now() - t0 > DEADLINE;
    g = __builtin_amdgcn_readfirstlane(g ? 1 : 0) != 0;
    if (g) __hip_atomic_store(tmo, call, __ATOMIC_RELAXED, __HIP_MEMORY_SCOPE_AGENT);
    return g;
}

__device__ __forceinline__ void wait_until(unsigned long long t0, unsigned dl) {
    for (;;) {
        const int left = (int)(dl + (unsigned)t0 - (unsigned)now());
        if (left <= 0) break;
        if (left > 30) __builtin_amdgcn_s_sleep(8); else __builtin_amdgcn_s_sleep(1);
    }
}

__device__ __forceinline__ void load_w(u32x4 (&buf)[8], __amdgpu_buffer_rsrc_t rw, int kc, unsigned c0, int lane) {
    const unsigned woff = (unsigned)((lane >> 4) * 8 * N_ + (lane & 15) * 4) * 2, sb = ((unsigned)(kc * 64) * N_ + c0) * 2;
#pragma unroll
    for (int i = 0; i < 16; ++i) {
        const u32x2 v = __builtin_amdgcn_raw_buffer_load_b64(rw, woff, sb + (unsigned)(((i >> 3) * 32 + (i & 7)) * N_) * 2, 0);
        buf[i >> 1][(i & 1) * 2] = v.x; buf[i >> 1][(i & 1) * 2 + 1] = v.y;
    }
}
__device__ __forceinline__ void store_w(const u32x4 (&buf)[8], __amdgpu_buffer_rsrc_t rd, unsigned dbase, int lane) {
#pragma unroll
    for (int kh = 0; kh < 2; ++kh)
#pragma unroll
        for (int f = 0; f < 4; ++f) {
            u32x4 o;
#pragma unroll
            for (int dd = 0; dd < 4; ++dd)
                o[dd] = __builtin_amdgcn_perm(buf[kh * 4 + dd][2 + (f >> 1)], buf[kh * 4 + dd][f >> 1], (f & 1) ? 0x07060302u : 0x05040100u);
            ST128(o, rd, lane * 16, dbase + (f >> 1) * HALF_B + ((f & 1) * 2 + kh) * 1024, AUX_SYS);
        }
}

__device__ __forceinline__ void in_addr(int h, int q, int src, unsigned& sb, unsigned& db, unsigned& fo) {
    const int kc = (h >> 1) / 96, j = (h >> 1) - kc * 96, i = (j & 3) * 2 + (h & 1);
    if (j < 32) {
        const unsigned o = (unsigned)(kc * NCB + (j >> 2)) * TILE_B + i * 1024;
        sb = EW_OFF + (unsigned)(q * NKC * NCB) * TILE_B + o; db = WS_OFF + (unsigned)(src * NKC * NCB) * TILE_B + o;
        fo = FW_OFF + (unsigned)(((j >> 2) * 64 + kc * 8 + src) * 8 + i) * 2;
    } else {
        const int rb = (((j - 32) >> 2) + 2 * q) & 15;
        sb = db = XS_OFF + (unsigned)((src * NKC + kc) * NRB + rb) * TILE_B + i * 1024;
        fo = FX_OFF + (unsigned)((rb * 64 + kc * 8 + src) * 8 + i) * 2;
    }
}

__device__ __forceinline__ void early_role(const Params& p, int q, int k, int wid, int lane, unsigned* ctr) {
    const unsigned long long tstart = now();
    const int rank = p.rank, d = (rank + q) & (NR - 1);
    unsigned char* myws = p.myws;
    unsigned* tmo = (unsigned*)(myws + TM_OFF);
    const unsigned cprev = __hip_atomic_load(ctr, __ATOMIC_RELAXED, __HIP_MEMORY_SCOPE_AGENT);
    const __amdgpu_buffer_rsrc_t rme = mk_rsrc(myws, WS_USED);
    unsigned char* dws = uni(p.ws[d]);
    const __amdgpu_buffer_rsrc_t rd = mk_rsrc(dws, WS_USED);
    const unsigned l0 = lane == 0 ? 0u : 0x80000000u;
    u32x4 buf[8];
    if (k < NPUSH0) load_w(buf, mk_rsrc(p.w, (unsigned)KS * N_ * 2), 0, (unsigned)(d * NB + k * 64), lane);
    const unsigned call = __builtin_amdgcn_readfirstlane(cprev) + 1;
    if (lane == 0) __hip_atomic_store(ctr, call, __ATOMIC_RELAXED, __HIP_MEMORY_SCOPE_AGENT);
    if (k < NPUSH0) {
        wait_until(tstart, TT_START + (unsigned)k * 144 / 10);
        store_w(buf, rd, WS_OFF + (unsigned)(rank * NKC * NCB + (k >> 2)) * TILE_B + (k & 3) * 4096, lane);
        asm volatile("s_waitcnt vmcnt(0)" ::: "memory");
        __builtin_amdgcn_raw_buffer_store_b32((call & 0xffffu) * 0x10001u, rd, l0, FW_OFF + (unsigned)(((k >> 2) * 64 + rank) * 4 + (k & 3)) * 4, AUX_SYS);
    }
    if (!EARLY_PUSH) return;
    int g_ok = -1;
    for (int m = k < NPUSH0 ? k + 32 : k; m < NUNIT; m += 32) {
        wait_until(tstart, (unsigned)(TT_START - EARLY_LEAD + m * 144 / 10));
        if (__builtin_amdgcn_readfirstlane(__builtin_amdgcn_raw_buffer_load_b32(rme, 0, AL_OFF + q * 4, AUX_SYS)) == (int)call) break;
        unsigned t = 0;
        if (lane == 0) t = __hip_atomic_fetch_add((unsigned*)(dws + TK_OFF + q * 128), 2u, __ATOMIC_RELAXED, __HIP_MEMORY_SCOPE_SYSTEM);
        const unsigned n = __builtin_amdgcn_readfirstlane(t) - (((call - 1) & 0xffffu) << 16);
        if (n >= 2u * (NUNIT - NPUSH0)) break;
        const int h = 2 * NPUSH0 + (int)n, hl = n + 1 < 2u * (NUNIT - NPUSH0) ? h + 1 : h;
        const int ul = hl >> 1, g = ul < 32 ? 0 : ul < 96 ? 1 : ul / 96 + 1;
        while (g_ok < g) {
            ++g_ok;
            const unsigned need = call * (g_ok == 0 ? (unsigned)NEXP_W0 : g_ok == 1 ? 64u : 320u);
            const unsigned long long t0 = now();
            for (;;) {
                const unsigned tmv = ld_tmo(tmo);
                const unsigned c = __builtin_amdgcn_raw_buffer_load_b32(rme, 0, EC_OFF + (g_ok == 0 ? 0 : g_ok == 1 ? 8 : g_ok - 1) * 128, AUX_DEV);
                if (__builtin_amdgcn_readfirstlane(c) == (int)need) break;
                if (give_up(tmo, tmv, call, t0)) break;
                __builtin_amdgcn_s_sleep(4);
            }
        }
        unsigned sb0, db0, fo0, sb1, db1, fo1;
        in_addr(h, q, rank, sb0, db0, fo0); in_addr(hl, q, rank, sb1, db1, fo1);
#pragma unroll
        for (int i = 0; i < 4; ++i) {
            buf[i] = __builtin_amdgcn_raw_buffer_load_b128(rme, lane * 16, sb0 + i * 8192, AUX_DEV);
            buf[4 + i] = __builtin_amdgcn_raw_buffer_load_b128(rme, lane * 16, sb1 + i * 8192, AUX_DEV);
        }
#pragma unroll
        for (int i = 0; i < 4; ++i) { ST128(buf[i], rd, lane * 16, db0 + i * 8192, AUX_SYS); ST128(buf[4 + i], rd, lane * 16, db1 + i * 8192, AUX_SYS); }
        asm volatile("s_waitcnt vmcnt(0)" ::: "memory");
        __builtin_amdgcn_raw_buffer_store_b16((unsigned short)call, rd, l0, fo0, AUX_SYS);
        __builtin_amdgcn_raw_buffer_store_b16((unsigned short)call, rd, l0, fo1, AUX_SYS);
    }
}

__device__ __forceinline__ void export_role(const Params& p, int eidx, int wid, int lane, unsigned* ctr) {
    const unsigned long long tstart = now();
    const int rank = p.rank;
    unsigned char* myws = p.myws;
    const unsigned cprev = __hip_atomic_load(ctr, __ATOMIC_RELAXED, __HIP_MEMORY_SCOPE_AGENT);
    unsigned char* peer = p.ws[(rank + lane) & (NR - 1)];
    const __amdgpu_buffer_rsrc_t rme = mk_rsrc(myws, WS_USED);
    const __amdgpu_buffer_rsrc_t rx = mk_rsrc(p.x, (unsigned)M_ * KS * 2);
    const __amdgpu_buffer_rsrc_t rw = mk_rsrc(p.w, (unsigned)KS * N_ * 2);
    u32x4 buf[8];
    const unsigned call = __builtin_amdgcn_readfirstlane(cprev) + 1;
    if (lane == 0) __hip_atomic_store(ctr, call, __ATOMIC_RELAXED, __HIP_MEMORY_SCOPE_AGENT);
    for (int e = eidx; e < NKC * 320; e += NEXP) {
        const int kc = e / 320, r = e - kc * 320;
        unsigned foff; int flane;
        if (kc > 0) wait_until(tstart, (unsigned)(kc * 96 * PACE_IN / 100 - EXP_LEAD + r * 2));
        else if (r >= 256) wait_until(tstart, 150u);
        if (r < 256) {
            const int j = r >> 3, q = r & 7, cb = j >> 2, wc = j & 3;
            if (kc == 0 && j < NPUSH0 && q != 0) continue;
            load_w(buf, rw, kc, (unsigned)(((rank + q) & (NR - 1)) * NB + j * 64), lane);
            store_w(buf, rme, (q == 0 ? WS_OFF + (unsigned)(rank * NKC * NCB) * TILE_B : EW_OFF + (unsigned)(q * NKC * NCB) * TILE_B)
                              + (unsigned)(kc * NCB + cb) * TILE_B + wc * 4096, lane);
            flane = q;
            foff = q == 0 ? FW_OFF + (unsigned)((cb * 64 + kc * 8 + rank) * 4 + wc) * 4 : EF_OFF + (unsigned)(q * NUNIT + j) * 4;
        } else {
            const int rb = (r - 256) >> 2, uq = r & 3;
            const unsigned xoff = (unsigned)((lane & 15) * KS + (lane >> 4) * 8) * 2, sb = (unsigned)((rb * 256 + uq * 64) * KS + kc * 64) * 2;
            const unsigned dbase = XS_OFF + (unsigned)((rank * NKC + kc) * NRB + rb) * TILE_B + uq * 8192;
#pragma unroll
            for (int g = 0; g < 8; ++g)
                buf[g] = __builtin_amdgcn_raw_buffer_load_b128(rx, xoff, sb + (unsigned)((g >> 1) * 16 * KS + (g & 1) * 32) * 2, 0);
#pragma unroll
            for (int g = 0; g < 8; ++g)
                ST128(buf[g], rme, lane * 16, dbase + g * 1024, AUX_SYS);
            flane = -1;
            foff = lane == 0 ? FX_OFF + (unsigned)((rb * 64 + kc * 8 + rank) * 4 + uq) * 4
                             : EF_OFF + (unsigned)(lane * NUNIT + 32 + ((rb - 2 * lane) & 15) * 4 + uq) * 4;
        }
        asm volatile("s_waitcnt vmcnt(0)" ::: "memory");
        if ((flane < 0 ? lane < NR : lane == flane) && (lane == 0 || kc == 0))
            __hip_atomic_store((unsigned*)(peer + foff), lane == 0 ? (call & 0xffffu) * 0x10001u : call, __ATOMIC_RELAXED, __HIP_MEMORY_SCOPE_SYSTEM);
        {
            unsigned n = 0;
            if (lane == 0) n = __hip_atomic_fetch_add((unsigned*)(myws + EC_OFF + (kc == 0 && r >= 256 ? 8 : kc) * 128), 1u, __ATOMIC_RELAXED, __HIP_MEMORY_SCOPE_AGENT);
            if (kc > 0 && __builtin_amdgcn_readfirstlane(n) + 1 == call * 320u && lane > 0 && lane < NR)
                __hip_atomic_store((unsigned*)(peer + EF_OFF) + lane * NUNIT + kc * 96, call, __ATOMIC_RELAXED, __HIP_MEMORY_SCOPE_SYSTEM);
        }
    }
    STAMPMAX(3);
}

__device__ __forceinline__ void link_role(const Params& p, int q, int widx, int wid, int lane, unsigned* ctr) {
    unsigned long long tstart = now();
    const int rank = p.rank;
    const int src = (rank - q) & (NR - 1);
    unsigned char* myws = p.myws;
    unsigned* tmo = (unsigned*)(myws + TM_OFF);
    const unsigned cprev = __hip_atomic_load(ctr, __ATOMIC_RELAXED, __HIP_MEMORY_SCOPE_AGENT);
    const __amdgpu_buffer_rsrc_t rme = mk_rsrc(myws, WS_USED);
    const unsigned l0 = lane == 0 ? 0u : 0x80000000u;
    const unsigned call = __builtin_amdgcn_readfirstlane(cprev) + 1;
    if (lane == 0) __hip_atomic_store(ctr, call, __ATOMIC_RELAXED, __HIP_MEMORY_SCOPE_AGENT);
    u32x4 bufa[4], bufb[4];
    const unsigned char* tk = uni(myws + TK_OFF + q * 128);
    unsigned tick;
#define DRAW() asm volatile("s_mov_b64 exec, 1\n\tglobal_atomic_add %0, %1, %2, %3 sc0 sc1\n\ts_mov_b64 exec, -1" : "=&v"(tick) : "v"(0u), "v"(1u), "s"(tk) : "memory")
#define ULOG_TOP
#define ULOG_PUT(I, T, E)
#define DUE(T0, DL) ((int)((DL) + (unsigned)(T0) - (unsigned)now()) <= 0)

    if (widx < NW) {
        const __amdgpu_buffer_rsrc_t rs = mk_rsrc(p.ws[src], WS_USED);
#define WAIT_EXPORTED(FO, COND) do { const unsigned long long t0_ = now(); \
            for (;;) { \
                const unsigned tmv = ld_tmo(tmo); \
                const u32x2 v = __builtin_amdgcn_raw_buffer_load_b64(rme, (FO), EF_OFF + (unsigned)(q * NUNIT) * 4, AUX_SYS); \
                if (__all(COND)) break; \
                if (give_up(tmo, tmv, call, t0_)) break; \
                __builtin_amdgcn_s_sleep(2); \
            } } while (0)
#define IN_ADDR(H, SB, DB, FO) in_addr((H), q, src, SB, DB, FO)
#define IN_LOAD(BUF, SB) do { _Pragma("unroll") for (int g = 0; g < 4; ++g) \
            BUF[g] = __builtin_amdgcn_raw_buffer_load_b128(rs, lane * 16, (SB) + g * 8192, AUX_SYS); } while (0)
#define FLAG(FO) __builtin_amdgcn_raw_buffer_store_b16((unsigned short)call, rme, l0, (FO), AUX_DEV)
#define NEED_EXPORTED(U) while ((U) >= u_ok) {                               \
                if (u_ok > 32) { WAIT_EXPORTED((unsigned)u_ok * 4, v.x == call); u_ok += 96; } \
                else { WAIT_EXPORTED((unsigned)(32 + (lane & 31) * 2) * 4, v.x == call && v.y == call); u_ok = 96; } \
            }
        DRAW();
        if (widx == 0) __builtin_amdgcn_raw_buffer_store_b32(call, rs, l0, AL_OFF + q * 4, AUX_SYS);
        WAIT_EXPORTED((unsigned)(NPUSH0 + (lane < (32 - NPUSH0) / 2 ? lane * 2 : 0)) * 4, v.x == call && v.y == call);
        asm volatile("s_waitcnt vmcnt(0)" : "+v"(tick) :: "memory");
        const unsigned tb = (((call - 1) & 0xffffu) << 16) - 2 * NPUSH0;
        const unsigned h0 = __builtin_amdgcn_readfirstlane(tick) - tb;
        if (h0 < 2u * NUNIT) {
        int u_ok = 32;
        unsigned fprev = 0;
        unsigned sb, db, fo, dbn, fon;
        DRAW();
        NEED_EXPORTED((int)h0 >> 1);
        IN_ADDR((int)h0, sb, db, fo);
        {
            const int late = (int)((unsigned)(now() - tstart) - (TT_START + (unsigned)NPUSH0 * PACE_IN / 100));
            tstart += (unsigned)((late > 0 ? late : 0) + (2 * NPUSH0 + widx - (int)h0) * PACE_IN / 200);
        }
        wait_until(tstart, TT_START + h0 * PACE_IN / 200);
        IN_LOAD(bufa, sb);
#define IN_STEP(CUR, NXT) { \
              \
            asm volatile("s_waitcnt vmcnt(0)" : "+v"(tick), "+v"(CUR[0]), "+v"(CUR[1]), "+v"(CUR[2]), "+v"(CUR[3]) :: "memory"); \
            ULOG_TOP; \
            const unsigned hr = __builtin_amdgcn_readfirstlane(tick) - tb; \
            const bool valid = hr < 2u * NUNIT; \
            const int hn = valid ? (int)hr : 2 * NUNIT - 1; \
            if (valid) NEED_EXPORTED(hn >> 1); \
            IN_ADDR(hn, sb, dbn, fon); \
            const unsigned dl = TT_START + (unsigned)hn * PACE_IN / 200; \
            const bool early = valid && DUE(tstart, dl); \
            if (early) IN_LOAD(NXT, sb); \
            asm volatile("" ::: "memory"); \
            fprev = __builtin_amdgcn_readfirstlane(fprev); db = __builtin_amdgcn_readfirstlane(db); fo = __builtin_amdgcn_readfirstlane(fo); \
            if (fprev) FLAG(fprev); \
            _Pragma("unroll") for (int g = 0; g < 4; ++g) ST128(CUR[g], rme, lane * 16, db + g * 8192, AUX_DEV); \
            fprev = fo; \
            if (valid) DRAW(); \
            if (hn >= 2 * NUNIT - FLAG_NOW && !early) {                      \
                asm volatile("s_waitcnt vmcnt(0)" : "+v"(tick) :: "memory"); \
                FLAG(fprev); fprev = 0; \
            } \
            if (!valid) break; \
            if (!early) { wait_until(tstart, dl); IN_LOAD(NXT, sb); } \
            ULOG_PUT(hn, ttop, early); \
            db = dbn; fo = fon; }
        for (;;) {
            IN_STEP(bufa, bufb)
            IN_STEP(bufb, bufa)
        }
        }
#undef WAIT_EXPORTED
    }
    STAMPMAX(1);

    if (widx < NWO) {
        const __amdgpu_buffer_rsrc_t rd = mk_rsrc(p.myout, OUT_BYTES);
        bool bad;
#define WAIT_TILES(STAGE, MASK) do { const unsigned long long t0_ = now(); \
            for (;;) { \
                const unsigned tmv = ld_tmo(tmo); \
                const u32x2 v = __builtin_amdgcn_raw_buffer_load_b64(rme, lane * 8, OF_OFF + ((STAGE) * NR + q) * 512, AUX_SYS); \
                bad = __builtin_amdgcn_readfirstlane(tmv) == call; \
                seen = __ballot(v.x == call && v.y == call); \
                if ((seen & (MASK)) == (MASK)) break; \
                if (give_up(tmo, tmv, call, t0_)) { bad = true; seen = ~0ull; break; } \
                __builtin_amdgcn_s_sleep(3); \
            } \
              \
            bm = __builtin_amdgcn_readfirstlane(bad ? 0x7f7f7f7f : 0); \
            rs = mk_rsrc(p.out[src], bm ? 0u : OUT_BYTES); } while (0)
#define OUT_RB(H) ((((H) >> 3) & 3) * 4 + (((H) >> 5) & 3))
#define RB_MASK(K) (1ull << (OUT_RB(2 * NPUSH1 + (K)) * 4))
#define OUT_ROW(H) (OUT_RB(H) * 256 + ((H) >> 10) * 128 + (((H) >> 7) & 1) * 64 + (((H) >> 8) & 3) * 16 + ((H) & 7) * 2)
#define OUT_OFF(K) ((unsigned)OUT_ROW(2 * NPUSH1 + (K)) * N_ + src * NB)
#define OFRAG(g) (((g) >> 1) * N_ + ((g) & 1) * 1024)
#define OUT_LOAD(BUF, OB) do { _Pragma("unroll") for (int g = 0; g < 4; ++g) \
            BUF[g] = __builtin_amdgcn_raw_buffer_load_b128(rs, lane * 16, (OB) + OFRAG(g), AUX_SYS); } while (0)
#define OUT_SLOT(K) ((unsigned)(K) * PACE_OUT / 200)
        unsigned bm;
        unsigned long long seen;
        __amdgpu_buffer_rsrc_t rs;
        tk += NR * 128;
        DRAW();
        WAIT_TILES(0, RB_MASK(widx));
        const unsigned long long tf = now();
        STAMPMAX(2);
        if (widx == 0) __builtin_amdgcn_raw_buffer_store_b32((call & 0xffffu) << 16, rme, l0, TK_OFF + q * 128, AUX_SYS);
        const unsigned tb = (call - 1) * (unsigned)(2 * (NOUT - NPUSH1)) - NWO;
        bool stage2 = false;
        unsigned ob = OUT_OFF(widx);
        wait_until(tf, OUT_SLOT(widx));
        OUT_LOAD(bufa, ob);
#define OUT_STEP(CUR, NXT) { \
            asm volatile("s_waitcnt vmcnt(0)" : "+v"(tick), "+v"(CUR[0]), "+v"(CUR[1]), "+v"(CUR[2]), "+v"(CUR[3]) :: "memory"); \
            ULOG_TOP; \
            const unsigned kr = __builtin_amdgcn_readfirstlane(tick) - tb; \
            const bool valid = kr < 2u * (NOUT - NPUSH1); \
            const int kn = valid ? (int)kr : 2 * (NOUT - NPUSH1) - 1; \
            const unsigned bmc = bm; \
            if (!stage2) { \
                if (NPUSH1 + (kn >> 1) >= STAGE2_AT) { WAIT_TILES(1, 1ull); stage2 = true; } \
                else if ((seen & RB_MASK(kn)) != RB_MASK(kn)) WAIT_TILES(0, RB_MASK(kn)); \
            } \
            const unsigned obn = OUT_OFF(kn), dl = OUT_SLOT(kn); \
            const bool early = valid && DUE(tf, dl); \
            if (early) OUT_LOAD(NXT, obn); \
            asm volatile("" ::: "memory"); \
            ob = __builtin_amdgcn_readfirstlane(ob); \
            _Pragma("unroll") for (int g = 0; g < 4; ++g) ST128(CUR[g] | bmc, rd, lane * 16, ob + OFRAG(g), AUX_DEV); \
            if (!valid) break; \
            DRAW(); \
            if (!early) { wait_until(tf, dl); OUT_LOAD(NXT, obn); } \
            ULOG_PUT(2048 + kn, ttop, early); \
            ob = obn; }
        for (;;) {
            OUT_STEP(bufa, bufb)
            OUT_STEP(bufb, bufa)
        }
        asm volatile("s_waitcnt vmcnt(0)" ::: "memory");
    }
    STAMPMAX(6);
}

#define WAIT_V(n) asm volatile("s_waitcnt vmcnt(" #n ")" ::: "memory")
#define WAIT_L(n) asm volatile("s_waitcnt lgkmcnt(" #n ")" ::: "memory")
#define BAR __builtin_amdgcn_s_barrier()
#define SCHED __builtin_amdgcn_sched_barrier(0)

__device__ __forceinline__ void gemm_role(const Params& p, LAS unsigned char* lds, int rb, int cb, int wid, int lane, unsigned call) {
    const int rank = p.rank;
    unsigned char* myws = p.myws;
    unsigned* tmo = (unsigned*)(myws + TM_OFF);
    const __amdgpu_buffer_rsrc_t rws = mk_rsrc(myws, WS_USED);
    const int wr = wid >> 2, wc = wid & 3;
    const unsigned tid16 = (unsigned)(wid * 64 + lane) * 16;
    volatile LAS int* sync_word = (volatile LAS int*)(lds + LDS_STAGE);
    unsigned char* peer_ws = p.ws[lane & 7];

    int tiv;
    {
        int s = rank, kc = lane;
        if (lane >= 8) {
            const int j = lane - 8, i = j % 7, q0 = rb >> 1;
            kc = j / 7;
            const int q = (i < q0) ? q0 - i : 7 - (i - q0);
            s = (rank - q) & (NR - 1);
        }
        tiv = s * 8 + kc;
    }
    const unsigned xtile0 = XS_OFF + rb * TILE_B, wtile0 = WS_OFF + cb * TILE_B;
    unsigned long long avail = 0, taken = 0;

    const unsigned fi = (unsigned)((tiv & 7) * 8 + (tiv >> 3)) * 16;
    const unsigned fxo = FX_OFF + rb * 1024 + fi, fwo = FW_OFF + cb * 1024 + fi;
    const unsigned cc = (call & 0xffffu) * 0x10001u;
#define FLAGS_OK(fx, fw) ((((fx.x ^ cc) | (fx.y ^ cc) | (fx.z ^ cc) | (fx.w ^ cc)) | ((fw.x ^ cc) | (fw.y ^ cc) | (fw.z ^ cc) | (fw.w ^ cc))) == 0)
#define PICK(T, SLOT, FAST) do { \
    if (avail == 0) { \
        if (wid == 0) { \
            unsigned long long r; \
            const unsigned long long t0 = now(); \
            for (;;) { \
                const unsigned tmv = ld_tmo(tmo); \
                const u32x4 fx = __builtin_amdgcn_raw_buffer_load_b128(rws, fxo, 0, AUX_DEV); \
                const u32x4 fw = __builtin_amdgcn_raw_buffer_load_b128(rws, fwo, 0, AUX_DEV); \
                r = __ballot(FLAGS_OK(fx, fw)) & ~taken; \
                if (r) break; \
                if (give_up(tmo, tmv, call, t0)) { r = ~taken; break; } \
                if (FAST) __builtin_amdgcn_s_sleep(2); else __builtin_amdgcn_s_sleep(PICK_SLEEP); \
            } \
            ARR(r); \
            if (lane == 0) { sync_word[2 * (SLOT)] = (int)(unsigned)r; sync_word[2 * (SLOT) + 1] = (int)(unsigned)(r >> 32); } \
            asm volatile("s_waitcnt lgkmcnt(0)" ::: "memory"); \
        } \
        asm volatile("" ::: "memory"); BAR; asm volatile("" ::: "memory"); \
        const unsigned lo_ = __builtin_amdgcn_readfirstlane(sync_word[2 * (SLOT)]), hi_ = __builtin_amdgcn_readfirstlane(sync_word[2 * (SLOT) + 1]); \
        avail = ((unsigned long long)hi_ << 32) | lo_; \
        asm volatile("s_waitcnt lgkmcnt(0)" ::: "memory"); \
    } \
    { const int i_ = __builtin_ctzll(avail); const unsigned long long b_ = 1ull << i_; avail &= ~b_; taken |= b_; T = (unsigned)__builtin_amdgcn_readlane(tiv, i_); } \
    } while (0)

#define SA(b, h) (((b) * 2 + (h)) * HALF_B)
#define SB(b, h) ((4 + (b) * 2 + (h)) * HALF_B)
#define XOFF(T) (xtile0 + (T) * (NRB * TILE_B))
#define WOFF(T) (wtile0 + (T) * (NCB * TILE_B))
#define STAGE(P, SRC) do { \
        __builtin_amdgcn_raw_ptr_buffer_load_lds(rws, (LAS void*)(lds + (P) + wid * 1024), 16, tid16, (SRC), 0, AUX_DEV); \
        __builtin_amdgcn_raw_ptr_buffer_load_lds(rws, (LAS void*)(lds + (P) + 8192 + wid * 1024), 16, tid16, (SRC) + 8192, 0, AUX_DEV); } while (0)
#define LDA(dst, b, h) _Pragma("unroll") for (int m = 0; m < 4; ++m) _Pragma("unroll") for (int k = 0; k < 2; ++k) \
        dst[m][k] = *(const LAS bf16x8*)(abase + SA(b, h) + (m * 2 + k) * 1024)
#define LDB(dst, b, h) _Pragma("unroll") for (int n = 0; n < 2; ++n) _Pragma("unroll") for (int k = 0; k < 2; ++k) \
        dst[n][k] = *(const LAS bf16x8*)(bbase + SB(b, h) - 4 * HALF_B + (n * 2 + k) * 1024)
#define MMA(ai, bj, At, Bt) do { __builtin_amdgcn_s_setprio(1); \
        _Pragma("unroll") for (int m = 0; m < 4; ++m) _Pragma("unroll") for (int n = 0; n < 2; ++n) _Pragma("unroll") for (int k = 0; k < 2; ++k) \
            acc[ai][bj][m][n] = __builtin_amdgcn_mfma_f32_16x16x32_bf16(Bt[n][k], At[m][k], acc[ai][bj][m][n], 0, 0, 0); \
        __builtin_amdgcn_s_setprio(0); } while (0)

#define HALF_MAX(ai, A, B) do { _Pragma("unroll") for (int bj = 0; bj < 2; ++bj) _Pragma("unroll") for (int m = 0; m < 4; ++m) { \
        const f32x4 u_ = acc[ai][bj][m][0], v_ = acc[ai][bj][m][1]; \
        A = fmaxf(fmaxf(A, fabsf(u_[0])), fabsf(u_[1])); B = fmaxf(fmaxf(B, fabsf(v_[0])), fabsf(v_[1])); \
        A = fmaxf(fmaxf(A, fabsf(u_[2])), fabsf(u_[3])); B = fmaxf(fmaxf(B, fabsf(v_[2])), fabsf(v_[3])); } } while (0)
    float am0 = 0.f, am1 = 0.f;
    const LAS unsigned char* abase = lds + wr * 8192 + lane * 16;
    const LAS unsigned char* bbase = lds + 4 * HALF_B + wc * 4096 + lane * 16;
    f32x4 acc[2][2][4][2] = {};
    bf16x8 At[4][2], B0[2][2], B1[2][2];

    unsigned ta, tb;
    PICK(ta, 0, false);
    STAMP(1);
    {
        const unsigned x0 = XOFF(ta), w0 = WOFF(ta);
        STAGE(SB(0, 0), w0); STAGE(SA(0, 0), x0);
        STAGE(SB(0, 1), w0 + HALF_B); STAGE(SA(0, 1), x0 + HALF_B);
        PICK(tb, 1, false);
        const unsigned x1 = XOFF(tb), w1 = WOFF(tb);
        if (wr == 1) BAR;
        WAIT_V(4); BAR;
        STAGE(SB(1, 0), w1); STAGE(SA(1, 0), x1); STAGE(SB(1, 1), w1 + HALF_B);
        WAIT_V(6); BAR;
    }
    for (int t = 0; t < NT - 2; t += 2) {
        STAMP(8 + (t >> 1));
        const bool fast = t >= FAST_FROM;
        LDB(B0, 0, 0); SCHED; LDA(At, 0, 0); STAGE(SA(1, 1), XOFF(tb) + HALF_B);
        WAIT_V(10); WAIT_L(8); BAR; WAIT_L(0); MMA(0, 0, At, B0); BAR; SCHED;
        PICK(ta, 2, fast);
        const unsigned x2 = XOFF(ta), w2 = WOFF(ta);
        LDB(B1, 0, 1); STAGE(SB(0, 0), w2);
        WAIT_V(10); BAR; WAIT_L(0); MMA(0, 1, At, B1); BAR;
        LDA(At, 0, 1); STAGE(SA(0, 0), x2);
        WAIT_V(10); BAR; WAIT_L(0); MMA(1, 0, At, B0); BAR; SCHED;
        STAGE(SB(0, 1), w2 + HALF_B);
        WAIT_V(10); BAR; MMA(1, 1, At, B1); BAR;
        LDB(B0, 1, 0); SCHED; LDA(At, 1, 0); STAGE(SA(0, 1), x2 + HALF_B);
        WAIT_V(10); WAIT_L(8); BAR; WAIT_L(0); MMA(0, 0, At, B0); BAR; SCHED;
        PICK(tb, 3, fast);
        const unsigned x3 = XOFF(tb), w3 = WOFF(tb);
        LDB(B1, 1, 1); STAGE(SB(1, 0), w3);
        WAIT_V(10); BAR; WAIT_L(0); MMA(0, 1, At, B1); BAR;
        LDA(At, 1, 1); STAGE(SA(1, 0), x3);
        WAIT_V(10); BAR; WAIT_L(0); MMA(1, 0, At, B0); BAR; SCHED;
        STAGE(SB(1, 1), w3 + HALF_B);
        WAIT_V(10); BAR; MMA(1, 1, At, B1); BAR;
    }
    STAMP(2);
    {
        const unsigned xl = XOFF(tb);
        LDB(B0, 0, 0); WAIT_V(0); LDA(At, 0, 0); STAGE(SA(1, 1), xl + HALF_B);
        BAR; WAIT_L(0); MMA(0, 0, At, B0); BAR;
        LDB(B1, 0, 1); BAR; WAIT_L(0); MMA(0, 1, At, B1); BAR;
        LDA(At, 0, 1); WAIT_V(4); BAR; WAIT_L(0); MMA(1, 0, At, B0); MMA(1, 1, At, B1); BAR;
    }
    {
        LDB(B0, 1, 0); LDA(At, 1, 0); WAIT_V(2); BAR; WAIT_L(0); MMA(0, 0, At, B0); BAR;
        LDB(B1, 1, 1); WAIT_V(0); BAR; WAIT_L(0); MMA(0, 1, At, B1); BAR;
        LDA(At, 1, 1); BAR; WAIT_L(0); MMA(1, 0, At, B0); MMA(1, 1, At, B1);
    }
    volatile LAS float* red = (volatile LAS float*)(lds + LDS_STAGE + 128);
    {
        float am2 = 0.f, am3 = 0.f;
        HALF_MAX(0, am0, am1);
        HALF_MAX(1, am2, am3);
        const float am = wave_max(fmaxf(fmaxf(am0, am1), fmaxf(am2, am3)));
        if (lane == 0) red[wid] = am;
    }
    WAIT_L(0); BAR;
    if (wr == 0) BAR;
    asm volatile("" ::: "memory");
    STAMP(3);

    const int gid = rb * NCB + cb;
    if (wid == 0) {
        const float a = wave_max(red[lane & 7]);
        if (lane < NR)
            __hip_atomic_store((unsigned long long*)(peer_ws + AM_OFF) + rank * 128 + gid,
                               ((unsigned long long)call << 32) | __float_as_uint(a), __ATOMIC_RELAXED, __HIP_MEMORY_SCOPE_SYSTEM);
    }
    unsigned char* po[NR - 1];
#pragma unroll
    for (int j = 0; j < NR - 1; ++j) { po[j] = uni(p.out[(rank + 1 + (gid + wc + j) % 7) & (NR - 1)]); asm volatile("" : "+s"(po[j])); }
    {
        const unsigned long long t0 = now();
        float g; bool bd;
        for (;;) {
            const unsigned tmv = ld_tmo(tmo);
            const u32x4 v = __builtin_amdgcn_raw_buffer_load_b128(rws, AM_OFF + tid16, 0, AUX_SYS);
            g = fmaxf(__uint_as_float(v.x), __uint_as_float(v.z));
            bd = tmv == call;
            if (__all(v.y == call && v.w == call)) break;
            if (give_up(tmo, tmv, call, t0)) { bd = true; break; }
            __builtin_amdgcn_s_sleep(1);
        }
        g = wave_max(g);
        if (lane == 0) red[8 + wid] = bd ? __builtin_nanf("") : g;
    }
    WAIT_L(0); BAR; asm volatile("" ::: "memory");
    const unsigned long long tout = now();
    float gmax = 0.f; bool bad = false;
#pragma unroll
    for (int i = 0; i < 8; ++i) { const float r = red[8 + i]; bad |= r != r; gmax = fmaxf(gmax, r); }
    STAMP(4);

    const float scale = gmax / 448.0f;
    const float inv = 1.0f / scale;
#define QUANT(O, ai, m) do { \
        const f32x4 a_ = acc[ai][0][m][0] * inv, b_ = acc[ai][0][m][1] * inv, c_ = acc[ai][1][m][0] * inv, e_ = acc[ai][1][m][1] * inv; \
        _Pragma("unroll") for (int r = 0; r < 4; ++r) { \
            int pk = __builtin_amdgcn_cvt_pk_fp8_f32(a_[r], b_[r], 0, false); \
            pk = __builtin_amdgcn_cvt_pk_fp8_f32(c_[r], e_[r], pk, true); \
            O[r] = bad ? 0x7f7f7f7fu : (unsigned)pk; \
        } } while (0)
    const unsigned ooff = (unsigned)(rb * 256 + wr * 64 + (lane & 15)) * N_ + rank * NB + cb * 256 + wc * 64 + (lane >> 4) * 16;
    const int rbi = (rb & 3) * 4 + (rb >> 2);
    const bool pusher = wr == 0 && rbi * 4 < NPUSH1;
    const unsigned dl = (unsigned)((rbi * 8 + cb) * 4 + wc) * PACE_FINE / 1000;
    const __amdgpu_buffer_rsrc_t rl = mk_rsrc(p.myout, OUT_BYTES);
    u32x4 q0;
    QUANT(q0, 0, 0);
    ST128(q0, rl, ooff, 0, AUX_SYS);
    asm volatile("" ::: "memory");
    const bool early = pusher && (unsigned)(now() - tout) >= dl;
#define PUSH_FIRST() do { _Pragma("unroll") for (int j = 0; j < NR - 1; ++j) \
        ST128(q0, mk_rsrc(po[j], OUT_BYTES), ooff, 0, AUX_SYS); } while (0)
    if (early) { PUSH_FIRST(); WAIT_V(7); } else { WAIT_V(0); }
    BAR; asm volatile("" ::: "memory");
    const int lq = ((lane & 7) - rank) & (NR - 1);
#define ANNOUNCE(CTR, N, WORD) do { unsigned n_ = 0; \
        if (lane == 0) n_ = __hip_atomic_fetch_add((unsigned*)(myws + OC_OFF + (CTR) * 128), 1u, __ATOMIC_RELAXED, __HIP_MEMORY_SCOPE_AGENT); \
        if (__builtin_amdgcn_readfirstlane(n_) + 1 == call * (N) && lane < NR && lq != 0) \
            __hip_atomic_store((unsigned long long*)(peer_ws + OF_OFF) + lq * 64 + (WORD), (unsigned long long)call * 0x100000001ull, __ATOMIC_RELAXED, __HIP_MEMORY_SCOPE_SYSTEM); } while (0)
    if (wid == 7) ANNOUNCE(rb, 8u, rb * 4);
    STAMP(7);
    if (pusher && !early) { wait_until(tout, dl); PUSH_FIRST(); }
    wait_until(tout, (unsigned)(STAGE2_DELAY + wid * STAGE2_STEP));
#pragma unroll
    for (int i = 1; i < 8; ++i) {
        u32x4 qi;
        QUANT(qi, i >> 2, i & 3);
        ST128(qi, rl, ooff, (unsigned)((i >> 2) * 128 + (i & 3) * 16) * N_, AUX_SYS);
    }
    WAIT_V(0);
    if (pusher && lane < NR)
        __hip_atomic_store((unsigned*)(peer_ws + DN_OFF + (unsigned)(gid * 8 + wid) * 32) + lq, call, __ATOMIC_RELAXED, __HIP_MEMORY_SCOPE_SYSTEM);
    BAR; asm volatile("" ::: "memory");
    if (wid == 7) ANNOUNCE(16, 128u, NR * 64);
    STAMP(5);
    if (pusher) {
        const unsigned dn = DN_OFF + (unsigned)(gid * 8 + wid) * 32;
        const unsigned long long t0 = now();
        for (;;) {
            const unsigned tmv = ld_tmo(tmo);
            const unsigned v = __builtin_amdgcn_raw_buffer_load_b32(rws, (lane & 7) * 4, dn, AUX_SYS);
            if (__all(v == call)) break;
            if (give_up(tmo, tmv, call, t0)) { if (lane == 0) p.myout[ooff] = 0x7f; break; }
            __builtin_amdgcn_s_sleep(2);
        }
    }
    STAMPMAX(6);
}

__global__ void __launch_bounds__(512, 2) fused_kernel(const Params p) {
    extern __shared__ __attribute__((aligned(16))) unsigned char smem[];
    const int b = blockIdx.x, xcd = b & 7, slot = b >> 3;
    const int wid = __builtin_amdgcn_readfirstlane(threadIdx.x >> 6), lane = threadIdx.x & 63;
    unsigned char* myws = p.myws;
    unsigned* ctr = (unsigned*)(myws + CT_OFF) + b * 8 + wid;
    STAMP(0);
    if (slot < 16) {
        const int q = slot & 7, wgi = xcd * 2 + (slot >> 3);
        if (q != 0 && wid < LINK_WAVES) { if (wid * 16 + wgi < (NW > NWO ? NW : NWO)) link_role(p, q, wid * 16 + wgi, wid, lane, ctr); }
        else if (q != 0 && wid >= 6) early_role(p, q, (7 - wid) * 16 + wgi, wid, lane, ctr);
        else export_role(p, q == 0 ? wid * 16 + wgi : 128 + (wid - LINK_WAVES) * 112 + (q - 1) * 16 + wgi, wid, lane, ctr);
    } else {
        const int g = slot - 16;
        const unsigned call = __builtin_amdgcn_readfirstlane(__hip_atomic_load(ctr, __ATOMIC_RELAXED, __HIP_MEMORY_SCOPE_AGENT)) + 1;
        if (lane == 0) __hip_atomic_store(ctr, call, __ATOMIC_RELAXED, __HIP_MEMORY_SCOPE_AGENT);
        gemm_role(p, (LAS unsigned char*)smem, (xcd >> 1) * 4 + (g >> 2), (xcd & 1) * 4 + (g & 3), wid, lane, call);
    }
}

}

extern "C" void kernel_launch(int rank, int nranks, ncclComm_t comm, hipStream_t stream,
                              void* const* d_in, const int* in_sizes, int n_in,
                              void* d_out, int out_size,
                              const int* in_dtypes, int out_dtype,
                              void* const* d_out_all,
                              void* const* d_ws_all, size_t ws_bytes) {
    if (nranks != NR || (unsigned)rank >= (unsigned)NR || n_in < 2 || ws_bytes < WS_USED ||
        in_sizes[0] != M_ * KS || in_sizes[1] != KS * N_ || out_size != M_ * N_) {
        fprintf(stderr, "kernel_launch: unexpected problem shape\n");
        return;
    }
    static bool attr_set = false;
    if (!attr_set) {
        HIP_CHECK(hipFuncSetAttribute((const void*)fused_kernel, hipFuncAttributeMaxDynamicSharedMemorySize, LDS_BYTES));
        attr_set = true;
    }
    Params p;
    p.x = (const unsigned short*)d_in[0];
    p.w = (const unsigned short*)d_in[1];
    for (int i = 0; i < NR; ++i) {
        p.out[i] = (unsigned char*)d_out_all[i];
        p.ws[i] = (unsigned char*)d_ws_all[i];
    }
    p.myws = p.ws[rank];
    p.myout = (unsigned char*)d_out;
    p.rank = rank;
    hipLaunchKernelGGL(fused_kernel, dim3(256), dim3(512), LDS_BYTES, stream, p);
    HIP_CHECK(hipGetLastError());
}
```

```cpp
#include <hip/hip_runtime.h>
#include <rccl/rccl.h>
#include <cstdio>
#include <cstdint>

#define HIP_CHECK(expr) do { hipError_t _e = (expr); if (_e != hipSuccess) { \
    fprintf(stderr, "HIP error %s at %s:%d\n", hipGetErrorString(_e), __FILE__, __LINE__); \
    return; } } while (0)

namespace {

#define LAS __attribute__((address_space(3)))
typedef short bf16x8 __attribute__((ext_vector_type(8)));
typedef float f32x4 __attribute__((ext_vector_type(4)));
typedef unsigned u32x4 __attribute__((ext_vector_type(4)));
typedef unsigned u32x2 __attribute__((ext_vector_type(2)));

constexpr int NR = 8;
constexpr int M_ = 4096, K_ = 4096, N_ = 16384;
constexpr int KS = K_ / NR;
constexpr int NB = N_ / NR;
constexpr int NRB = 16, NCB = 8;
constexpr int NKC = 8;
constexpr int NT = 64;
constexpr unsigned TILE_B = 32768, HALF_B = 16384;
constexpr int NUNIT = NKC * 96;
constexpr int NOUT = 1024;

constexpr unsigned XS_OFF = 0;
constexpr unsigned WS_OFF = XS_OFF + 64u * NRB * TILE_B;
constexpr unsigned EW_OFF = WS_OFF + 64u * NCB * TILE_B;
constexpr unsigned FX_OFF = EW_OFF + 64u * NCB * TILE_B;
constexpr unsigned FW_OFF = FX_OFF + NRB * 64 * 16;
constexpr unsigned EF_OFF = FW_OFF + NCB * 64 * 16;
constexpr unsigned OF_OFF = EF_OFF + NR * NUNIT * 4;
constexpr unsigned AM_OFF = OF_OFF + 2 * NR * 512;
constexpr unsigned DN_OFF = AM_OFF + NR * 128 * 8;
constexpr unsigned CT_OFF = DN_OFF + 1024 * 32;
constexpr unsigned TK_OFF = CT_OFF + 256 * 8 * 4;
constexpr unsigned EC_OFF = TK_OFF + 2 * NR * 128;
constexpr unsigned AL_OFF = EC_OFF + 9 * 128;
constexpr unsigned OC_OFF = AL_OFF + 128;
constexpr unsigned TM_OFF = OC_OFF + 17 * 128;
constexpr unsigned DB_OFF = TM_OFF + 256;
constexpr unsigned UL_OFF = DB_OFF + 512 * 64 * 8;
constexpr unsigned WS_USED = UL_OFF + 8 * 4096 * 8;
constexpr unsigned OUT_BYTES = (unsigned)M_ * N_;

constexpr unsigned long long DEADLINE = 100000000ull;
constexpr int AUX_SYS = 17, AUX_DEV = 16;

#define NW 18
#define NWO 14
#define PACE_IN 1300
#define PACE_OUT 1320
#define NPUSH0 16
#define NPUSH1 12
#define EARLY_PUSH 1
#define EARLY_LEAD 0
#define PICK_SLEEP 20
#define FAST_FROM 54
constexpr int TT_START = 100;
constexpr int PACE_FINE = 1800;
constexpr int EXP_LEAD = 900;
#define PEEK_AHEAD 40
constexpr int FLAG_NOW = 128;
constexpr int STAGE2_DELAY = 450, STAGE2_STEP = 60;
constexpr int STAGE2_AT = 100;
constexpr int LINK_WAVES = 2;
constexpr int NEXP = 112 * 4 + 128;
constexpr int NEXP_W0 = 256 - 7 * NPUSH0;
static_assert(NPUSH0 % 4 == 0 && NPUSH0 <= 16 && NPUSH1 % 4 == 0 && NPUSH1 <= 64 && NW >= 8 && NW <= 32 && NWO >= 8 && NWO <= 32, "");

constexpr int LDS_STAGE = 8 * HALF_B;
constexpr int LDS_BYTES = LDS_STAGE + 256;

struct Params {
    const unsigned short* x;
    const unsigned short* w;
    unsigned char* out[NR];
    unsigned char* ws[NR];
    unsigned char* myws;
    unsigned char* myout;
    int rank;
};

__device__ __forceinline__ unsigned char* uni(const void* p) {
    const unsigned long long v = (unsigned long long)p;
    const unsigned lo = __builtin_amdgcn_readfirstlane((unsigned)v), hi = __builtin_amdgcn_readfirstlane((unsigned)(v >> 32));
    return (unsigned char*)(((unsigned long long)hi << 32) | lo);
}
__device__ __forceinline__ __amdgpu_buffer_rsrc_t mk_rsrc(const void* p, unsigned bytes) {
    return __builtin_amdgcn_make_buffer_rsrc((void*)uni(p), 0, (int)bytes, 0x00020000);
}
#define ST128(V, R, VO, SO, AUX) do { const u32x4 st_v_ = (V); __builtin_amdgcn_raw_buffer_store_b128(st_v_, (R), (VO), (SO), (AUX)); \
        asm volatile("s_nop 1" :: "v"(st_v_)); } while (0)
__device__ __forceinline__ float wave_max(float f) {
    unsigned v = __float_as_uint(f);
    v = max(v, (unsigned)__builtin_amdgcn_update_dpp(0, (int)v, 0x111, 0xf, 0xf, false));
    v = max(v, (unsigned)__builtin_amdgcn_update_dpp(0, (int)v, 0x112, 0xf, 0xf, false));
    v = max(v, (unsigned)__builtin_amdgcn_update_dpp(0, (int)v, 0x114, 0xf, 0xf, false));
    v = max(v, (unsigned)__builtin_amdgcn_update_dpp(0, (int)v, 0x118, 0xf, 0xf, false));
    v = max(v, (unsigned)__builtin_amdgcn_update_dpp(0, (int)v, 0x142, 0xa, 0xf, false));
    v = max(v, (unsigned)__builtin_amdgcn_update_dpp(0, (int)v, 0x143, 0xc, 0xf, false));
    return __uint_as_float((unsigned)__builtin_amdgcn_readlane((int)v, 63));
}
__device__ __forceinline__ unsigned long long now() { return __builtin_amdgcn_s_memrealtime(); }

#define STAMP(i) do { } while (0)
#define STAMPMAX(i) do { } while (0)

#define ARR(r) do { } while (0)

__device__ __forceinline__ unsigned ld_tmo(unsigned* tmo) { return __hip_atomic_load(tmo, __ATOMIC_RELAXED, __HIP_MEMORY_SCOPE_AGENT); }
__device__ __forceinline__ bool give_up(unsigned* tmo, unsigned tmv, unsigned call, unsigned long long t0) {
    bool g = tmv == call || now() - t0 > DEADLINE;
    g = __builtin_amdgcn_readfirstlane(g ? 1 : 0) != 0;
    if (g) __hip_atomic_store(tmo, call, __ATOMIC_RELAXED, __HIP_MEMORY_SCOPE_AGENT);
    return g;
}

__device__ __forceinline__ void wait_until(unsigned long long t0, unsigned dl) {
    for (;;) {
        const int left = (int)(dl + (unsigned)t0 - (unsigned)now());
        if (left <= 0) break;
        if (left > 30) __builtin_amdgcn_s_sleep(8); else __builtin_amdgcn_s_sleep(1);
    }
}

__device__ __forceinline__ void load_w(u32x4 (&buf)[8], __amdgpu_buffer_rsrc_t rw, int kc, unsigned c0, int lane) {
    const unsigned woff = (unsigned)((lane >> 4) * 8 * N_ + (lane & 15) * 4) * 2, sb = ((unsigned)(kc * 64) * N_ + c0) * 2;
#pragma unroll
    for (int i = 0; i < 16; ++i) {
        const u32x2 v = __builtin_amdgcn_raw_buffer_load_b64(rw, woff, sb + (unsigned)(((i >> 3) * 32 + (i & 7)) * N_) * 2, 0);
        buf[i >> 1][(i & 1) * 2] = v.x; buf[i >> 1][(i & 1) * 2 + 1] = v.y;
    }
}
__device__ __forceinline__ void store_w(const u32x4 (&buf)[8], __amdgpu_buffer_rsrc_t rd, unsigned dbase, int lane) {
#pragma unroll
    for (int kh = 0; kh < 2; ++kh)
#pragma unroll
        for (int f = 0; f < 4; ++f) {
            u32x4 o;
#pragma unroll
            for (int dd = 0; dd < 4; ++dd)
                o[dd] = __builtin_amdgcn_perm(buf[kh * 4 + dd][2 + (f >> 1)], buf[kh * 4 + dd][f >> 1], (f & 1) ? 0x07060302u : 0x05040100u);
            ST128(o, rd, lane * 16, dbase + (f >> 1) * HALF_B + ((f & 1) * 2 + kh) * 1024, AUX_SYS);
        }
}

__device__ __forceinline__ void in_addr(int h, int q, int src, unsigned& sb, unsigned& db, unsigned& fo) {
    const int kc = (h >> 1) / 96, j = (h >> 1) - kc * 96, i = (j & 3) * 2 + (h & 1);
    if (j < 32) {
        const unsigned o = (unsigned)(kc * NCB + (j >> 2)) * TILE_B + i * 1024;
        sb = EW_OFF + (unsigned)(q * NKC * NCB) * TILE_B + o; db = WS_OFF + (unsigned)(src * NKC * NCB) * TILE_B + o;
        fo = FW_OFF + (unsigned)(((j >> 2) * 64 + kc * 8 + src) * 8 + i) * 2;
    } else {
        const int rb = (((j - 32) >> 2) + 2 * q) & 15;
        sb = db = XS_OFF + (unsigned)((src * NKC + kc) * NRB + rb) * TILE_B + i * 1024;
        fo = FX_OFF + (unsigned)((rb * 64 + kc * 8 + src) * 8 + i) * 2;
    }
}

__device__ __forceinline__ void early_role(const Params& p, int q, int k, int wid, int lane, unsigned* ctr) {
    const unsigned long long tstart = now();
    const int rank = p.rank, d = (rank + q) & (NR - 1);
    unsigned char* myws = p.myws;
    unsigned* tmo = (unsigned*)(myws + TM_OFF);
    const unsigned cprev = __hip_atomic_load(ctr, __ATOMIC_RELAXED, __HIP_MEMORY_SCOPE_AGENT);
    const __amdgpu_buffer_rsrc_t rme = mk_rsrc(myws, WS_USED);
    unsigned char* dws = uni(p.ws[d]);
    const __amdgpu_buffer_rsrc_t rd = mk_rsrc(dws, WS_USED);
    const unsigned l0 = lane == 0 ? 0u : 0x80000000u;
    u32x4 buf[8];
    if (k < NPUSH0) load_w(buf, mk_rsrc(p.w, (unsigned)KS * N_ * 2), 0, (unsigned)(d * NB + k * 64), lane);
    const unsigned call = __builtin_amdgcn_readfirstlane(cprev) + 1;
    if (lane == 0) __hip_atomic_store(ctr, call, __ATOMIC_RELAXED, __HIP_MEMORY_SCOPE_AGENT);
    if (k < NPUSH0) {
        wait_until(tstart, TT_START + (unsigned)k * 144 / 10);
        store_w(buf, rd, WS_OFF + (unsigned)(rank * NKC * NCB + (k >> 2)) * TILE_B + (k & 3) * 4096, lane);
        asm volatile("s_waitcnt vmcnt(0)" ::: "memory");
        __builtin_amdgcn_raw_buffer_store_b32((call & 0xffffu) * 0x10001u, rd, l0, FW_OFF + (unsigned)(((k >> 2) * 64 + rank) * 4 + (k & 3)) * 4, AUX_SYS);
    }
    if (!EARLY_PUSH) return;
    int g_ok = -1;
    for (int m = k < NPUSH0 ? k + 32 : k; m < NUNIT; m += 32) {
        wait_until(tstart, (unsigned)(TT_START - EARLY_LEAD + m * 144 / 10));
        if (__builtin_amdgcn_readfirstlane(__builtin_amdgcn_raw_buffer_load_b32(rme, 0, AL_OFF + q * 4, AUX_SYS)) == (int)call) break;
        unsigned t = 0;
        if (lane == 0) t = __hip_atomic_fetch_add((unsigned*)(dws + TK_OFF + q * 128), 2u, __ATOMIC_RELAXED, __HIP_MEMORY_SCOPE_SYSTEM);
        const unsigned n = __builtin_amdgcn_readfirstlane(t) - (((call - 1) & 0xffffu) << 16);
        if (n >= 2u * (NUNIT - NPUSH0)) break;
        const int h = 2 * NPUSH0 + (int)n, hl = n + 1 < 2u * (NUNIT - NPUSH0) ? h + 1 : h;
        const int ul = hl >> 1, g = ul < 32 ? 0 : ul < 96 ? 1 : ul / 96 + 1;
        while (g_ok < g) {
            ++g_ok;
            const unsigned need = call * (g_ok == 0 ? (unsigned)NEXP_W0 : g_ok == 1 ? 64u : 320u);
            const unsigned long long t0 = now();
            for (;;) {
                const unsigned tmv = ld_tmo(tmo);
                const unsigned c = __builtin_amdgcn_raw_buffer_load_b32(rme, 0, EC_OFF + (g_ok == 0 ? 0 : g_ok == 1 ? 8 : g_ok - 1) * 128, AUX_DEV);
                if (__builtin_amdgcn_readfirstlane(c) == (int)need) break;
                if (give_up(tmo, tmv, call, t0)) break;
                __builtin_amdgcn_s_sleep(4);
            }
        }
        unsigned sb0, db0, fo0, sb1, db1, fo1;
        in_addr(h, q, rank, sb0, db0, fo0); in_addr(hl, q, rank, sb1, db1, fo1);
#pragma unroll
        for (int i = 0; i < 4; ++i) {
            buf[i] = __builtin_amdgcn_raw_buffer_load_b128(rme, lane * 16, sb0 + i * 8192, AUX_DEV);
            buf[4 + i] = __builtin_amdgcn_raw_buffer_load_b128(rme, lane * 16, sb1 + i * 8192, AUX_DEV);
        }
#pragma unroll
        for (int i = 0; i < 4; ++i) { ST128(buf[i], rd, lane * 16, db0 + i * 8192, AUX_SYS); ST128(buf[4 + i], rd, lane * 16, db1 + i * 8192, AUX_SYS); }
        asm volatile("s_waitcnt vmcnt(0)" ::: "memory");
        __builtin_amdgcn_raw_buffer_store_b16((unsigned short)call, rd, l0, fo0, AUX_SYS);
        __builtin_amdgcn_raw_buffer_store_b16((unsigned short)call, rd, l0, fo1, AUX_SYS);
    }
}

__device__ __forceinline__ void export_role(const Params& p, int eidx, int wid, int lane, unsigned* ctr) {
    const unsigned long long tstart = now();
    const int rank = p.rank;
    unsigned char* myws = p.myws;
    const unsigned cprev = __hip_atomic_load(ctr, __ATOMIC_RELAXED, __HIP_MEMORY_SCOPE_AGENT);
    unsigned char* peer = p.ws[(rank + lane) & (NR - 1)];
    const __amdgpu_buffer_rsrc_t rme = mk_rsrc(myws, WS_USED);
    const __amdgpu_buffer_rsrc_t rx = mk_rsrc(p.x, (unsigned)M_ * KS * 2);
    const __amdgpu_buffer_rsrc_t rw = mk_rsrc(p.w, (unsigned)KS * N_ * 2);
    u32x4 buf[8];
    const unsigned call = __builtin_amdgcn_readfirstlane(cprev) + 1;
    if (lane == 0) __hip_atomic_store(ctr, call, __ATOMIC_RELAXED, __HIP_MEMORY_SCOPE_AGENT);
    for (int e = eidx; e < NKC * 320; e += NEXP) {
        const int kc = e / 320, r = e - kc * 320;
        unsigned foff; int flane;
        if (kc > 0) wait_until(tstart, (unsigned)(kc * 96 * PACE_IN / 100 - EXP_LEAD + r * 2));
        else if (r >= 256) wait_until(tstart, 150u);
        if (r < 256) {
            const int j = r >> 3, q = r & 7, cb = j >> 2, wc = j & 3;
            if (kc == 0 && j < NPUSH0 && q != 0) continue;
            load_w(buf, rw, kc, (unsigned)(((rank + q) & (NR - 1)) * NB + j * 64), lane);
            store_w(buf, rme, (q == 0 ? WS_OFF + (unsigned)(rank * NKC * NCB) * TILE_B : EW_OFF + (unsigned)(q * NKC * NCB) * TILE_B)
                              + (unsigned)(kc * NCB + cb) * TILE_B + wc * 4096, lane);
            flane = q;
            foff = q == 0 ? FW_OFF + (unsigned)((cb * 64 + kc * 8 + rank) * 4 + wc) * 4 : EF_OFF + (unsigned)(q * NUNIT + j) * 4;
        } else {
            const int rb = (r - 256) >> 2, uq = r & 3;
            const unsigned xoff = (unsigned)((lane & 15) * KS + (lane >> 4) * 8) * 2, sb = (unsigned)((rb * 256 + uq * 64) * KS + kc * 64) * 2;
            const unsigned dbase = XS_OFF + (unsigned)((rank * NKC + kc) * NRB + rb) * TILE_B + uq * 8192;
#pragma unroll
            for (int g = 0; g < 8; ++g)
                buf[g] = __builtin_amdgcn_raw_buffer_load_b128(rx, xoff, sb + (unsigned)((g >> 1) * 16 * KS + (g & 1) * 32) * 2, 0);
#pragma unroll
            for (int g = 0; g < 8; ++g)
                ST128(buf[g], rme, lane * 16, dbase + g * 1024, AUX_SYS);
            flane = -1;
            foff = lane == 0 ? FX_OFF + (unsigned)((rb * 64 + kc * 8 + rank) * 4 + uq) * 4
                             : EF_OFF + (unsigned)(lane * NUNIT + 32 + ((rb - 2 * lane) & 15) * 4 + uq) * 4;
        }
        asm volatile("s_waitcnt vmcnt(0)" ::: "memory");
        if ((flane < 0 ? lane < NR : lane == flane) && (lane == 0 || kc == 0))
            __hip_atomic_store((unsigned*)(peer + foff), lane == 0 ? (call & 0xffffu) * 0x10001u : call, __ATOMIC_RELAXED, __HIP_MEMORY_SCOPE_SYSTEM);
        {
            unsigned n = 0;
            if (lane == 0) n = __hip_atomic_fetch_add((unsigned*)(myws + EC_OFF + (kc == 0 && r >= 256 ? 8 : kc) * 128), 1u, __ATOMIC_RELAXED, __HIP_MEMORY_SCOPE_AGENT);
            if (kc > 0 && __builtin_amdgcn_readfirstlane(n) + 1 == call * 320u && lane > 0 && lane < NR)
                __hip_atomic_store((unsigned*)(peer + EF_OFF) + lane * NUNIT + kc * 96, call, __ATOMIC_RELAXED, __HIP_MEMORY_SCOPE_SYSTEM);
        }
    }
    STAMPMAX(3);
}

__device__ __forceinline__ void link_role(const Params& p, int q, int widx, int wid, int lane, unsigned* ctr) {
    unsigned long long tstart = now();
    const int rank = p.rank;
    const int src = (rank - q) & (NR - 1);
    unsigned char* myws = p.myws;
    unsigned* tmo = (unsigned*)(myws + TM_OFF);
    const unsigned cprev = __hip_atomic_load(ctr, __ATOMIC_RELAXED, __HIP_MEMORY_SCOPE_AGENT);
    const __amdgpu_buffer_rsrc_t rme = mk_rsrc(myws, WS_USED);
    const unsigned l0 = lane == 0 ? 0u : 0x80000000u;
    const unsigned call = __builtin_amdgcn_readfirstlane(cprev) + 1;
    if (lane == 0) __hip_atomic_store(ctr, call, __ATOMIC_RELAXED, __HIP_MEMORY_SCOPE_AGENT);
    u32x4 bufa[4], bufb[4];
    const unsigned char* tk = uni(myws + TK_OFF + q * 128);
    unsigned tick;
#define DRAW() asm volatile("s_mov_b64 exec, 1\n\tglobal_atomic_add %0, %1, %2, %3 sc0 sc1\n\ts_mov_b64 exec, -1" : "=&v"(tick) : "v"(0u), "v"(1u), "s"(tk) : "memory")
#define ULOG_TOP
#define ULOG_PUT(I, T, E)
    const unsigned char* wsu = uni(myws);
    u32x2 pk = {0u, 0u};
    bool peeked = false;
#define PEEK(VO) do { asm volatile("global_load_dwordx2 %0, %1, %2 sc0 sc1" : "+v"(pk) : "v"(VO), "s"(wsu) : "memory"); peeked = true; } while (0)
#define DUE(T0, DL) ((int)((DL) + (unsigned)(T0) - (unsigned)now()) <= 0)

    if (widx < NW) {
        const __amdgpu_buffer_rsrc_t rs = mk_rsrc(p.ws[src], WS_USED);
#define WAIT_EXPORTED(FO, COND) do { const unsigned long long t0_ = now(); \
            for (;;) { \
                const unsigned tmv = ld_tmo(tmo); \
                const u32x2 v = __builtin_amdgcn_raw_buffer_load_b64(rme, (FO), EF_OFF + (unsigned)(q * NUNIT) * 4, AUX_SYS); \
                if (__all(COND)) break; \
                if (give_up(tmo, tmv, call, t0_)) break; \
                __builtin_amdgcn_s_sleep(2); \
            } } while (0)
#define IN_ADDR(H, SB, DB, FO) in_addr((H), q, src, SB, DB, FO)
#define IN_LOAD(BUF, SB) do { _Pragma("unroll") for (int g = 0; g < 4; ++g) \
            BUF[g] = __builtin_amdgcn_raw_buffer_load_b128(rs, lane * 16, (SB) + g * 8192, AUX_SYS); } while (0)
#define FLAG(FO) __builtin_amdgcn_raw_buffer_store_b16((unsigned short)call, rme, l0, (FO), AUX_DEV)
#define NEED_EXPORTED(U) while ((U) >= u_ok) {                               \
                if (u_ok > 32) { WAIT_EXPORTED((unsigned)u_ok * 4, v.x == call); u_ok += 96; } \
                else { WAIT_EXPORTED((unsigned)(32 + (lane & 31) * 2) * 4, v.x == call && v.y == call); u_ok = 96; } \
            }
        DRAW();
        if (widx == 0) __builtin_amdgcn_raw_buffer_store_b32(call, rs, l0, AL_OFF + q * 4, AUX_SYS);
        WAIT_EXPORTED((unsigned)(NPUSH0 + (lane < (32 - NPUSH0) / 2 ? lane * 2 : 0)) * 4, v.x == call && v.y == call);
        asm volatile("s_waitcnt vmcnt(0)" : "+v"(tick) :: "memory");
        const unsigned tb = (((call - 1) & 0xffffu) << 16) - 2 * NPUSH0;
        const unsigned h0 = __builtin_amdgcn_readfirstlane(tick) - tb;
        if (h0 < 2u * NUNIT) {
        int u_ok = 32;
        unsigned fprev = 0;
        unsigned sb, db, fo, dbn, fon;
        DRAW();
        NEED_EXPORTED((int)h0 >> 1);
        IN_ADDR((int)h0, sb, db, fo);
        {
            const int late = (int)((unsigned)(now() - tstart) - (TT_START + (unsigned)NPUSH0 * PACE_IN / 100));
            tstart += (unsigned)((late > 0 ? late : 0) + (2 * NPUSH0 + widx - (int)h0) * PACE_IN / 200);
        }
        wait_until(tstart, TT_START + h0 * PACE_IN / 200);
        IN_LOAD(bufa, sb);
#define IN_STEP(CUR, NXT) { \
              \
            asm volatile("s_waitcnt vmcnt(0)" : "+v"(tick), "+v"(pk), "+v"(CUR[0]), "+v"(CUR[1]), "+v"(CUR[2]), "+v"(CUR[3]) :: "memory"); \
            ULOG_TOP; \
            if (peeked) { if (__all(pk.x == call && (u_ok > 32 || pk.y == call))) u_ok = u_ok > 32 ? u_ok + 96 : 96; peeked = false; } \
            const unsigned hr = __builtin_amdgcn_readfirstlane(tick) - tb; \
            const bool valid = hr < 2u * NUNIT; \
            const int hn = valid ? (int)hr : 2 * NUNIT - 1; \
            if (valid) NEED_EXPORTED(hn >> 1); \
            IN_ADDR(hn, sb, dbn, fon); \
            const unsigned dl = TT_START + (unsigned)hn * PACE_IN / 200; \
            const bool early = valid && DUE(tstart, dl); \
            if (early) IN_LOAD(NXT, sb); \
            asm volatile("" ::: "memory"); \
            fprev = __builtin_amdgcn_readfirstlane(fprev); db = __builtin_amdgcn_readfirstlane(db); fo = __builtin_amdgcn_readfirstlane(fo); \
            if (fprev) FLAG(fprev); \
            _Pragma("unroll") for (int g = 0; g < 4; ++g) ST128(CUR[g], rme, lane * 16, db + g * 8192, AUX_DEV); \
            fprev = fo; \
            if (valid) DRAW(); \
            if (valid && u_ok < NUNIT && (hn >> 1) + PEEK_AHEAD >= u_ok && (hn >> 1) < u_ok) \
                PEEK(EF_OFF + (unsigned)(q * NUNIT + (u_ok > 32 ? u_ok : 32 + (lane & 31) * 2)) * 4); \
            if (hn >= 2 * NUNIT - FLAG_NOW && !early) {                      \
                asm volatile("s_waitcnt vmcnt(0)" : "+v"(tick) :: "memory"); \
                FLAG(fprev); fprev = 0; \
            } \
            if (!valid) break; \
            if (!early) { wait_until(tstart, dl); IN_LOAD(NXT, sb); } \
            ULOG_PUT(hn, ttop, early); \
            db = dbn; fo = fon; }
        for (;;) {
            IN_STEP(bufa, bufb)
            IN_STEP(bufb, bufa)
        }
        }
#undef WAIT_EXPORTED
    }
    STAMPMAX(1);

    if (widx < NWO) {
        const __amdgpu_buffer_rsrc_t rd = mk_rsrc(p.myout, OUT_BYTES);
        bool bad;
#define WAIT_TILES(STAGE, MASK) do { const unsigned long long t0_ = now(); \
            for (;;) { \
                const unsigned tmv = ld_tmo(tmo); \
                const u32x2 v = __builtin_amdgcn_raw_buffer_load_b64(rme, lane * 8, OF_OFF + ((STAGE) * NR + q) * 512, AUX_SYS); \
                bad = __builtin_amdgcn_readfirstlane(tmv) == call; \
                seen = __ballot(v.x == call && v.y == call); \
                if ((seen & (MASK)) == (MASK)) break; \
                if (give_up(tmo, tmv, call, t0_)) { bad = true; seen = ~0ull; break; } \
                __builtin_amdgcn_s_sleep(3); \
            } \
              \
            bm = __builtin_amdgcn_readfirstlane(bad ? 0x7f7f7f7f : 0); \
            rs = mk_rsrc(p.out[src], bm ? 0u : OUT_BYTES); } while (0)
#define OUT_RB(H) ((((H) >> 3) & 3) * 4 + (((H) >> 5) & 3))
#define RB_MASK(K) (1ull << (OUT_RB(2 * NPUSH1 + (K)) * 4))
#define OUT_ROW(H) (OUT_RB(H) * 256 + ((H) >> 10) * 128 + (((H) >> 7) & 1) * 64 + (((H) >> 8) & 3) * 16 + ((H) & 7) * 2)
#define OUT_OFF(K) ((unsigned)OUT_ROW(2 * NPUSH1 + (K)) * N_ + src * NB)
#define OFRAG(g) (((g) >> 1) * N_ + ((g) & 1) * 1024)
#define OUT_LOAD(BUF, OB) do { _Pragma("unroll") for (int g = 0; g < 4; ++g) \
            BUF[g] = __builtin_amdgcn_raw_buffer_load_b128(rs, lane * 16, (OB) + OFRAG(g), AUX_SYS); } while (0)
#define OUT_SLOT(K) ((unsigned)(K) * PACE_OUT / 200)
        unsigned bm;
        unsigned long long seen;
        __amdgpu_buffer_rsrc_t rs;
        tk += NR * 128;
        DRAW();
        WAIT_TILES(0, RB_MASK(widx));
        const unsigned long long tf = now();
        STAMPMAX(2);
        if (widx == 0) __builtin_amdgcn_raw_buffer_store_b32((call & 0xffffu) << 16, rme, l0, TK_OFF + q * 128, AUX_SYS);
        const unsigned tb = (call - 1) * (unsigned)(2 * (NOUT - NPUSH1)) - NWO;
        bool stage2 = false;
        unsigned ob = OUT_OFF(widx);
        wait_until(tf, OUT_SLOT(widx));
        OUT_LOAD(bufa, ob);
#define OUT_STEP(CUR, NXT) { \
            asm volatile("s_waitcnt vmcnt(0)" : "+v"(tick), "+v"(pk), "+v"(CUR[0]), "+v"(CUR[1]), "+v"(CUR[2]), "+v"(CUR[3]) :: "memory"); \
            ULOG_TOP; \
            if (peeked) {         \
                seen = __ballot(pk.x == call && pk.y == call); \
                if (seen >> 63) stage2 = true; \
                peeked = false; \
            } \
            const unsigned kr = __builtin_amdgcn_readfirstlane(tick) - tb; \
            const bool valid = kr < 2u * (NOUT - NPUSH1); \
            const int kn = valid ? (int)kr : 2 * (NOUT - NPUSH1) - 1; \
            const unsigned bmc = bm; \
            if (!stage2) { \
                if (NPUSH1 + (kn >> 1) >= STAGE2_AT) { WAIT_TILES(1, 1ull); stage2 = true; } \
                else if ((seen & RB_MASK(kn)) != RB_MASK(kn)) WAIT_TILES(0, RB_MASK(kn)); \
            } \
            const unsigned obn = OUT_OFF(kn), dl = OUT_SLOT(kn); \
            const bool early = valid && DUE(tf, dl); \
            if (early) OUT_LOAD(NXT, obn); \
            asm volatile("" ::: "memory"); \
            ob = __builtin_amdgcn_readfirstlane(ob); \
            _Pragma("unroll") for (int g = 0; g < 4; ++g) ST128(CUR[g] | bmc, rd, lane * 16, ob + OFRAG(g), AUX_DEV); \
            if (!valid) break; \
            DRAW(); \
            if (!stage2 && ((~seen & 0x1111111111111111ull) != 0 || NPUSH1 + (kn >> 1) >= STAGE2_AT - 40)) \
                PEEK(OF_OFF + (unsigned)(q * 512 + (lane == 63 ? NR * 512 : lane * 8))); \
            if (!early) { wait_until(tf, dl); OUT_LOAD(NXT, obn); } \
            ULOG_PUT(2048 + kn, ttop, early); \
            ob = obn; }
        for (;;) {
            OUT_STEP(bufa, bufb)
            OUT_STEP(bufb, bufa)
        }
        asm volatile("s_waitcnt vmcnt(0)" ::: "memory");
    }
    STAMPMAX(6);
}

#define WAIT_V(n) asm volatile("s_waitcnt vmcnt(" #n ")" ::: "memory")
#define WAIT_L(n) asm volatile("s_waitcnt lgkmcnt(" #n ")" ::: "memory")
#define BAR __builtin_amdgcn_s_barrier()
#define SCHED __builtin_amdgcn_sched_barrier(0)

__device__ __forceinline__ void gemm_role(const Params& p, LAS unsigned char* lds, int rb, int cb, int wid, int lane, unsigned call) {
    const int rank = p.rank;
    unsigned char* myws = p.myws;
    unsigned* tmo = (unsigned*)(myws + TM_OFF);
    const __amdgpu_buffer_rsrc_t rws = mk_rsrc(myws, WS_USED);
    const int wr = wid >> 2, wc = wid & 3;
    const unsigned tid16 = (unsigned)(wid * 64 + lane) * 16;
    volatile LAS int* sync_word = (volatile LAS int*)(lds + LDS_STAGE);
    unsigned char* peer_ws = p.ws[lane & 7];

    int tiv;
    {
        int s = rank, kc = lane;
        if (lane >= 8) {
            const int j = lane - 8, i = j % 7, q0 = rb >> 1;
            kc = j / 7;
            const int q = (i < q0) ? q0 - i : 7 - (i - q0);
            s = (rank - q) & (NR - 1);
        }
        tiv = s * 8 + kc;
    }
    const unsigned xtile0 = XS_OFF + rb * TILE_B, wtile0 = WS_OFF + cb * TILE_B;
    unsigned long long avail = 0, taken = 0;

    const unsigned fi = (unsigned)((tiv & 7) * 8 + (tiv >> 3)) * 16;
    const unsigned fxo = FX_OFF + rb * 1024 + fi, fwo = FW_OFF + cb * 1024 + fi;
    const unsigned cc = (call & 0xffffu) * 0x10001u;
#define FLAGS_OK(fx, fw) ((((fx.x ^ cc) | (fx.y ^ cc) | (fx.z ^ cc) | (fx.w ^ cc)) | ((fw.x ^ cc) | (fw.y ^ cc) | (fw.z ^ cc) | (fw.w ^ cc))) == 0)
#define PICK(T, SLOT, FAST) do { \
    if (avail == 0) { \
        if (wid == 0) { \
            unsigned long long r; \
            const unsigned long long t0 = now(); \
            for (;;) { \
                const unsigned tmv = ld_tmo(tmo); \
                const u32x4 fx = __builtin_amdgcn_raw_buffer_load_b128(rws, fxo, 0, AUX_DEV); \
                const u32x4 fw = __builtin_amdgcn_raw_buffer_load_b128(rws, fwo, 0, AUX_DEV); \
                r = __ballot(FLAGS_OK(fx, fw)) & ~taken; \
                if (r) break; \
                if (give_up(tmo, tmv, call, t0)) { r = ~taken; break; } \
                if (FAST) __builtin_amdgcn_s_sleep(2); else __builtin_amdgcn_s_sleep(PICK_SLEEP); \
            } \
            ARR(r); \
            if (lane == 0) { sync_word[2 * (SLOT)] = (int)(unsigned)r; sync_word[2 * (SLOT) + 1] = (int)(unsigned)(r >> 32); } \
            asm volatile("s_waitcnt lgkmcnt(0)" ::: "memory"); \
        } \
        asm volatile("" ::: "memory"); BAR; asm volatile("" ::: "memory"); \
        const unsigned lo_ = __builtin_amdgcn_readfirstlane(sync_word[2 * (SLOT)]), hi_ = __builtin_amdgcn_readfirstlane(sync_word[2 * (SLOT) + 1]); \
        avail = ((unsigned long long)hi_ << 32) | lo_; \
        asm volatile("s_waitcnt lgkmcnt(0)" ::: "memory"); \
    } \
    { const int i_ = __builtin_ctzll(avail); const unsigned long long b_ = 1ull << i_; avail &= ~b_; taken |= b_; T = (unsigned)__builtin_amdgcn_readlane(tiv, i_); } \
    } while (0)

#define SA(b, h) (((b) * 2 + (h)) * HALF_B)
#define SB(b, h) ((4 + (b) * 2 + (h)) * HALF_B)
#define XOFF(T) (xtile0 + (T) * (NRB * TILE_B))
#define WOFF(T) (wtile0 + (T) * (NCB * TILE_B))
#define STAGE(P, SRC) do { \
        __builtin_amdgcn_raw_ptr_buffer_load_lds(rws, (LAS void*)(lds + (P) + wid * 1024), 16, tid16, (SRC), 0, AUX_DEV); \
        __builtin_amdgcn_raw_ptr_buffer_load_lds(rws, (LAS void*)(lds + (P) + 8192 + wid * 1024), 16, tid16, (SRC) + 8192, 0, AUX_DEV); } while (0)
#define LDA(dst, b, h) _Pragma("unroll") for (int m = 0; m < 4; ++m) _Pragma("unroll") for (int k = 0; k < 2; ++k) \
        dst[m][k] = *(const LAS bf16x8*)(abase + SA(b, h) + (m * 2 + k) * 1024)
#define LDB(dst, b, h) _Pragma("unroll") for (int n = 0; n < 2; ++n) _Pragma("unroll") for (int k = 0; k < 2; ++k) \
        dst[n][k] = *(const LAS bf16x8*)(bbase + SB(b, h) - 4 * HALF_B + (n * 2 + k) * 1024)
#define MMA(ai, bj, At, Bt) do { __builtin_amdgcn_s_setprio(1); \
        _Pragma("unroll") for (int m = 0; m < 4; ++m) _Pragma("unroll") for (int n = 0; n < 2; ++n) _Pragma("unroll") for (int k = 0; k < 2; ++k) \
            acc[ai][bj][m][n] = __builtin_amdgcn_mfma_f32_16x16x32_bf16(Bt[n][k], At[m][k], acc[ai][bj][m][n], 0, 0, 0); \
        __builtin_amdgcn_s_setprio(0); } while (0)

#define HALF_MAX(ai, A, B) do { _Pragma("unroll") for (int bj = 0; bj < 2; ++bj) _Pragma("unroll") for (int m = 0; m < 4; ++m) { \
        const f32x4 u_ = acc[ai][bj][m][0], v_ = acc[ai][bj][m][1]; \
        A = fmaxf(fmaxf(A, fabsf(u_[0])), fabsf(u_[1])); B = fmaxf(fmaxf(B, fabsf(v_[0])), fabsf(v_[1])); \
        A = fmaxf(fmaxf(A, fabsf(u_[2])), fabsf(u_[3])); B = fmaxf(fmaxf(B, fabsf(v_[2])), fabsf(v_[3])); } } while (0)
    float am0 = 0.f, am1 = 0.f;
    const LAS unsigned char* abase = lds + wr * 8192 + lane * 16;
    const LAS unsigned char* bbase = lds + 4 * HALF_B + wc * 4096 + lane * 16;
    f32x4 acc[2][2][4][2] = {};
    bf16x8 At[4][2], B0[2][2], B1[2][2];

    unsigned ta, tb;
    PICK(ta, 0, false);
    STAMP(1);
    {
        const unsigned x0 = XOFF(ta), w0 = WOFF(ta);
        STAGE(SB(0, 0), w0); STAGE(SA(0, 0), x0);
        STAGE(SB(0, 1), w0 + HALF_B); STAGE(SA(0, 1), x0 + HALF_B);
        PICK(tb, 1, false);
        const unsigned x1 = XOFF(tb), w1 = WOFF(tb);
        if (wr == 1) BAR;
        WAIT_V(4); BAR;
        STAGE(SB(1, 0), w1); STAGE(SA(1, 0), x1); STAGE(SB(1, 1), w1 + HALF_B);
        WAIT_V(6); BAR;
    }
    for (int t = 0; t < NT - 2; t += 2) {
        STAMP(8 + (t >> 1));
        const bool fast = t >= FAST_FROM;
        LDB(B0, 0, 0); SCHED; LDA(At, 0, 0); STAGE(SA(1, 1), XOFF(tb) + HALF_B);
        WAIT_V(10); WAIT_L(8); BAR; WAIT_L(0); MMA(0, 0, At, B0); BAR; SCHED;
        PICK(ta, 2, fast);
        const unsigned x2 = XOFF(ta), w2 = WOFF(ta);
        LDB(B1, 0, 1); STAGE(SB(0, 0), w2);
        WAIT_V(10); BAR; WAIT_L(0); MMA(0, 1, At, B1); BAR;
        LDA(At, 0, 1); STAGE(SA(0, 0), x2);
        WAIT_V(10); BAR; WAIT_L(0); MMA(1, 0, At, B0); BAR; SCHED;
        STAGE(SB(0, 1), w2 + HALF_B);
        WAIT_V(10); BAR; MMA(1, 1, At, B1); BAR;
        LDB(B0, 1, 0); SCHED; LDA(At, 1, 0); STAGE(SA(0, 1), x2 + HALF_B);
        WAIT_V(10); WAIT_L(8); BAR; WAIT_L(0); MMA(0, 0, At, B0); BAR; SCHED;
        PICK(tb, 3, fast);
        const unsigned x3 = XOFF(tb), w3 = WOFF(tb);
        LDB(B1, 1, 1); STAGE(SB(1, 0), w3);
        WAIT_V(10); BAR; WAIT_L(0); MMA(0, 1, At, B1); BAR;
        LDA(At, 1, 1); STAGE(SA(1, 0), x3);
        WAIT_V(10); BAR; WAIT_L(0); MMA(1, 0, At, B0); BAR; SCHED;
        STAGE(SB(1, 1), w3 + HALF_B);
        WAIT_V(10); BAR; MMA(1, 1, At, B1); BAR;
    }
    STAMP(2);
    {
        const unsigned xl = XOFF(tb);
        LDB(B0, 0, 0); WAIT_V(0); LDA(At, 0, 0); STAGE(SA(1, 1), xl + HALF_B);
        BAR; WAIT_L(0); MMA(0, 0, At, B0); BAR;
        LDB(B1, 0, 1); BAR; WAIT_L(0); MMA(0, 1, At, B1); BAR;
        LDA(At, 0, 1); WAIT_V(4); BAR; WAIT_L(0); MMA(1, 0, At, B0); MMA(1, 1, At, B1); BAR;
    }
    {
        LDB(B0, 1, 0); LDA(At, 1, 0); WAIT_V(2); BAR; WAIT_L(0); MMA(0, 0, At, B0); BAR;
        LDB(B1, 1, 1); WAIT_V(0); BAR; WAIT_L(0); MMA(0, 1, At, B1); BAR;
        LDA(At, 1, 1); BAR; WAIT_L(0); MMA(1, 0, At, B0); MMA(1, 1, At, B1);
    }
    volatile LAS float* red = (volatile LAS float*)(lds + LDS_STAGE + 128);
    {
        float am2 = 0.f, am3 = 0.f;
        HALF_MAX(0, am0, am1);
        HALF_MAX(1, am2, am3);
        const float am = wave_max(fmaxf(fmaxf(am0, am1), fmaxf(am2, am3)));
        if (lane == 0) red[wid] = am;
    }
    WAIT_L(0); BAR;
    if (wr == 0) BAR;
    asm volatile("" ::: "memory");
    STAMP(3);

    const int gid = rb * NCB + cb;
    if (wid == 0) {
        const float a = wave_max(red[lane & 7]);
        if (lane < NR)
            __hip_atomic_store((unsigned long long*)(peer_ws + AM_OFF) + rank * 128 + gid,
                               ((unsigned long long)call << 32) | __float_as_uint(a), __ATOMIC_RELAXED, __HIP_MEMORY_SCOPE_SYSTEM);
    }
    unsigned char* po[NR - 1];
#pragma unroll
    for (int j = 0; j < NR - 1; ++j) { po[j] = uni(p.out[(rank + 1 + (gid + wc + j) % 7) & (NR - 1)]); asm volatile("" : "+s"(po[j])); }
    {
        const unsigned long long t0 = now();
        float g; bool bd;
        for (;;) {
            const unsigned tmv = ld_tmo(tmo);
            const u32x4 v = __builtin_amdgcn_raw_buffer_load_b128(rws, AM_OFF + tid16, 0, AUX_SYS);
            g = fmaxf(__uint_as_float(v.x), __uint_as_float(v.z));
            bd = tmv == call;
            if (__all(v.y == call && v.w == call)) break;
            if (give_up(tmo, tmv, call, t0)) { bd = true; break; }
            __builtin_amdgcn_s_sleep(1);
        }
        g = wave_max(g);
        if (lane == 0) red[8 + wid] = bd ? __builtin_nanf("") : g;
    }
    WAIT_L(0); BAR; asm volatile("" ::: "memory");
    const unsigned long long tout = now();
    float gmax = 0.f; bool bad = false;
#pragma unroll
    for (int i = 0; i < 8; ++i) { const float r = red[8 + i]; bad |= r != r; gmax = fmaxf(gmax, r); }
    STAMP(4);

    const float scale = gmax / 448.0f;
    const float inv = 1.0f / scale;
#define QUANT(O, ai, m) do { \
        const f32x4 a_ = acc[ai][0][m][0] * inv, b_ = acc[ai][0][m][1] * inv, c_ = acc[ai][1][m][0] * inv, e_ = acc[ai][1][m][1] * inv; \
        _Pragma("unroll") for (int r = 0; r < 4; ++r) { \
            int pk = __builtin_amdgcn_cvt_pk_fp8_f32(a_[r], b_[r], 0, false); \
            pk = __builtin_amdgcn_cvt_pk_fp8_f32(c_[r], e_[r], pk, true); \
            O[r] = bad ? 0x7f7f7f7fu : (unsigned)pk; \
        } } while (0)
    const unsigned ooff = (unsigned)(rb * 256 + wr * 64 + (lane & 15)) * N_ + rank * NB + cb * 256 + wc * 64 + (lane >> 4) * 16;
    const int rbi = (rb & 3) * 4 + (rb >> 2);
    const bool pusher = wr == 0 && rbi * 4 < NPUSH1;
    const unsigned dl = (unsigned)((rbi * 8 + cb) * 4 + wc) * PACE_FINE / 1000;
    const __amdgpu_buffer_rsrc_t rl = mk_rsrc(p.myout, OUT_BYTES);
    u32x4 q0;
    QUANT(q0, 0, 0);
    ST128(q0, rl, ooff, 0, AUX_SYS);
    asm volatile("" ::: "memory");
    const bool early = pusher && (unsigned)(now() - tout) >= dl;
#define PUSH_FIRST() do { _Pragma("unroll") for (int j = 0; j < NR - 1; ++j) \
        ST128(q0, mk_rsrc(po[j], OUT_BYTES), ooff, 0, AUX_SYS); } while (0)
    if (early) { PUSH_FIRST(); WAIT_V(7); } else { WAIT_V(0); }
    BAR; asm volatile("" ::: "memory");
    const int lq = ((lane & 7) - rank) & (NR - 1);
#define ANNOUNCE(CTR, N, WORD) do { unsigned n_ = 0; \
        if (lane == 0) n_ = __hip_atomic_fetch_add((unsigned*)(myws + OC_OFF + (CTR) * 128), 1u, __ATOMIC_RELAXED, __HIP_MEMORY_SCOPE_AGENT); \
        if (__builtin_amdgcn_readfirstlane(n_) + 1 == call * (N) && lane < NR && lq != 0) \
            __hip_atomic_store((unsigned long long*)(peer_ws + OF_OFF) + lq * 64 + (WORD), (unsigned long long)call * 0x100000001ull, __ATOMIC_RELAXED, __HIP_MEMORY_SCOPE_SYSTEM); } while (0)
    if (wid == 7) ANNOUNCE(rb, 8u, rb * 4);
    STAMP(7);
    if (pusher && !early) { wait_until(tout, dl); PUSH_FIRST(); }
    wait_until(tout, (unsigned)(STAGE2_DELAY + wid * STAGE2_STEP));
#pragma unroll
    for (int i = 1; i < 8; ++i) {
        u32x4 qi;
        QUANT(qi, i >> 2, i & 3);
        ST128(qi, rl, ooff, (unsigned)((i >> 2) * 128 + (i & 3) * 16) * N_, AUX_SYS);
    }
    WAIT_V(0);
    if (pusher && lane < NR)
        __hip_atomic_store((unsigned*)(peer_ws + DN_OFF + (unsigned)(gid * 8 + wid) * 32) + lq, call, __ATOMIC_RELAXED, __HIP_MEMORY_SCOPE_SYSTEM);
    BAR; asm volatile("" ::: "memory");
    if (wid == 7) ANNOUNCE(16, 128u, NR * 64);
    STAMP(5);
    if (pusher) {
        const unsigned dn = DN_OFF + (unsigned)(gid * 8 + wid) * 32;
        const unsigned long long t0 = now();
        for (;;) {
            const unsigned tmv = ld_tmo(tmo);
            const unsigned v = __builtin_amdgcn_raw_buffer_load_b32(rws, (lane & 7) * 4, dn, AUX_SYS);
            if (__all(v == call)) break;
            if (give_up(tmo, tmv, call, t0)) { if (lane == 0) p.myout[ooff] = 0x7f; break; }
            __builtin_amdgcn_s_sleep(2);
        }
    }
    STAMPMAX(6);
}

__global__ void __launch_bounds__(512, 2) fused_kernel(const Params p) {
    extern __shared__ __attribute__((aligned(16))) unsigned char smem[];
    const int b = blockIdx.x, xcd = b & 7, slot = b >> 3;
    const int wid = __builtin_amdgcn_readfirstlane(threadIdx.x >> 6), lane = threadIdx.x & 63;
    unsigned char* myws = p.myws;
    unsigned* ctr = (unsigned*)(myws + CT_OFF) + b * 8 + wid;
    STAMP(0);
    if (slot < 16) {
        const int q = slot & 7, wgi = xcd * 2 + (slot >> 3);
        if (q != 0 && wid < LINK_WAVES) { if (wid * 16 + wgi < (NW > NWO ? NW : NWO)) link_role(p, q, wid * 16 + wgi, wid, lane, ctr); }
        else if (q != 0 && wid >= 6) early_role(p, q, (7 - wid) * 16 + wgi, wid, lane, ctr);
        else export_role(p, q == 0 ? wid * 16 + wgi : 128 + (wid - LINK_WAVES) * 112 + (q - 1) * 16 + wgi, wid, lane, ctr);
    } else {
        const int g = slot - 16;
        const unsigned call = __builtin_amdgcn_readfirstlane(__hip_atomic_load(ctr, __ATOMIC_RELAXED, __HIP_MEMORY_SCOPE_AGENT)) + 1;
        if (lane == 0) __hip_atomic_store(ctr, call, __ATOMIC_RELAXED, __HIP_MEMORY_SCOPE_AGENT);
        gemm_role(p, (LAS unsigned char*)smem, (xcd >> 1) * 4 + (g >> 2), (xcd & 1) * 4 + (g & 3), wid, lane, call);
    }
}

}

extern "C" void kernel_launch(int rank, int nranks, ncclComm_t comm, hipStream_t stream,
                              void* const* d_in, const int* in_sizes, int n_in,
                              void* d_out, int out_size,
                              const int* in_dtypes, int out_dtype,
                              void* const* d_out_all,
                              void* const* d_ws_all, size_t ws_bytes) {
    if (nranks != NR || (unsigned)rank >= (unsigned)NR || n_in < 2 || ws_bytes < WS_USED ||
        in_sizes[0] != M_ * KS || in_sizes[1] != KS * N_ || out_size != M_ * N_) {
        fprintf(stderr, "kernel_launch: unexpected problem shape\n");
        return;
    }
    static bool attr_set = false;
    if (!attr_set) {
        HIP_CHECK(hipFuncSetAttribute((const void*)fused_kernel, hipFuncAttributeMaxDynamicSharedMemorySize, LDS_BYTES));
        attr_set = true;
    }
    Params p;
    p.x = (const unsigned short*)d_in[0];
    p.w = (const unsigned short*)d_in[1];
    for (int i = 0; i < NR; ++i) {
        p.out[i] = (unsigned char*)d_out_all[i];
        p.ws[i] = (unsigned char*)d_ws_all[i];
    }
    p.myws = p.ws[rank];
    p.myout = (unsigned char*)d_out;
    p.rank = rank;
    hipLaunchKernelGGL(fused_kernel, dim3(256), dim3(512), LDS_BYTES, stream, p);
    HIP_CHECK(hipGetLastError());
}
```
